```python
import numpy as np

kernel_name = "gpu_a2a_gemm_m4096_k4096_n4096_e5m2e5m2_dequant"


ABSMAX_RTOL = 0.05

COMPUTE_DTYPE = "float32"
EPILOGUE = "dequant"
IN_QUANT = None
OUT_QUANT = None
IN_FP8 = "e5m2e5m2"
IN_DTYPES = {"x": "e5m2", "w_mat": "e5m2",
             "scale_x": "float32", "scale_w": "float32"}
WORLD_SIZE = 8

def _snap_fp8(x, mant_bits, max_abs, min_normal):
    a = np.abs(x.astype(np.float64))
    _m, _e = np.frexp(a)
    q = np.where(a >= min_normal, np.ldexp(1.0, _e - 1 - mant_bits),
                 min_normal * 2.0 ** -mant_bits)
    snapped = np.minimum(np.round(a / q) * q, max_abs)
    return (np.sign(x) * snapped).astype(np.float32)

def setup_inputs():

    x = _snap_fp8(np.random.randn(4096, 4096).astype(np.float32), 2, 57344.0, 2.0 ** -14)
    w_mat = _snap_fp8(np.random.randn(4096, 4096).astype(np.float32), 2, 57344.0, 2.0 ** -14)
    scale_x = (0.02 * (1.0 + np.random.rand(1))).astype(np.float32)
    scale_w = (0.02 * (1.0 + np.random.rand(1))).astype(np.float32)
    return {"x": x, "w_mat": w_mat, "scale_x": scale_x, "scale_w": scale_w}

def reference(x, w_mat, scale_x, scale_w):
    acc = (x @ w_mat).astype(np.float32)
    y = (acc * (scale_x * scale_w)).astype(np.float32)
    return y
```

```cpp
#include <hip/hip_runtime.h>
#include <rccl/rccl.h>
#include <cstdio>

#define HIP_CHECK(expr) do { hipError_t _e = (expr); if (_e != hipSuccess) { \
    fprintf(stderr, "HIP error %s at %s:%d\n", hipGetErrorString(_e), __FILE__, __LINE__); \
    return; } } while (0)

typedef unsigned u32;
typedef unsigned long long u64;
typedef unsigned u32x4 __attribute__((ext_vector_type(4)));
typedef int i32x2 __attribute__((ext_vector_type(2)));
typedef int i32x4 __attribute__((ext_vector_type(4)));
typedef int i32x8 __attribute__((ext_vector_type(8)));
typedef float f32x16 __attribute__((ext_vector_type(16)));
#define LAS __attribute__((address_space(3)))

#define DELTA 150
#define LEAD 2
#define NWS 8
#define NAS 3

namespace {
constexpr int NRANK = 8, GN = 4096, KS = 512;
constexpr size_t OFF_CNT = 0;
constexpr size_t OFF_FLAG = 64 << 10;
constexpr size_t OFF_TMO = 112 << 10;
constexpr size_t OFF_DBG = 128 << 10;
constexpr size_t OFF_SLOT = 1 << 20;
constexpr size_t SLOT_BYTES = 2u << 20;
constexpr size_t WS_NEED = OFF_SLOT + 2 * SLOT_BYTES;
constexpr u64 DEADLINE = 40000000ull;

constexpr u32 LDS_W = 0, LDS_A = NWS * 16384, LDS_CTL = LDS_A + NAS * 8192, LDS_BYTES = LDS_CTL + 64;
constexpr u32 C_FULL = 0  , C_FREE = 16  , C_MASK = 32, C_CALL = 36, C_STAT = 40;

struct Args {
  const unsigned char* x; const unsigned char* w; const float* sx; const float* sw; float* out;
  char* ws[NRANK]; int rank; int pad;
};

template <int N> __device__ __forceinline__ void waitvm() { asm volatile("s_waitcnt vmcnt(%0)" :: "i"(N) : "memory"); }
__device__ __forceinline__ u32 rfl(u32 v) { return __builtin_amdgcn_readfirstlane(v); }
__device__ __forceinline__ u32x4 lds_ld4(u32 addr) { return *(volatile LAS u32x4*)addr; }
__device__ __forceinline__ void lds_st(u32 addr, u32 v) { *(volatile LAS u32*)addr = v; }

#define DMA8(SC) \
  asm volatile("s_mov_b32 m0, %9\n\ts_nop 0\n\t" \
    "global_load_lds_dwordx4 %0, %8" SC "\n\ts_add_u32 m0, m0, 0x400\n\ts_nop 0\n\t" \
    "global_load_lds_dwordx4 %1, %8" SC "\n\ts_add_u32 m0, m0, 0x400\n\ts_nop 0\n\t" \
    "global_load_lds_dwordx4 %2, %8" SC "\n\ts_add_u32 m0, m0, 0x400\n\ts_nop 0\n\t" \
    "global_load_lds_dwordx4 %3, %8" SC "\n\ts_add_u32 m0, m0, 0x400\n\ts_nop 0\n\t" \
    "global_load_lds_dwordx4 %4, %8" SC "\n\ts_add_u32 m0, m0, 0x400\n\ts_nop 0\n\t" \
    "global_load_lds_dwordx4 %5, %8" SC "\n\ts_add_u32 m0, m0, 0x400\n\ts_nop 0\n\t" \
    "global_load_lds_dwordx4 %6, %8" SC "\n\ts_add_u32 m0, m0, 0x400\n\ts_nop 0\n\t" \
    "global_load_lds_dwordx4 %7, %8" SC \
    :: "v"(off[0]), "v"(off[1]), "v"(off[2]), "v"(off[3]), "v"(off[4]), "v"(off[5]), "v"(off[6]), "v"(off[7]), \
       "s"(base), "s"(dst) : "memory", "scc")
__device__ __forceinline__ void dma8(const u32 (&off)[8], const void* base, u32 dst) { DMA8(""); }
__device__ __forceinline__ void dma8_sys(const u32 (&off)[8], const void* base, u32 dst) { DMA8(" sc0 sc1"); }

__device__ __forceinline__ void step_of(int s, int rank, int mt, int& src, int& j) {
  if (s < 4) { src = rank; j = s; return; }
  int t = s - 4; j = t / 7; int i = t - j * 7 + mt; if (i >= 7) i -= 7; if (i >= 7) i -= 7;
  src = (rank + 1 + i) & 7;
}
__device__ __forceinline__ u32 min_free(u32 ctl) {
  u32x4 f = lds_ld4(ctl + C_FREE);
  return rfl(min(min(f.x, f.y), min(f.z, f.w)));
}
__device__ __forceinline__ void wait_oldest(int younger) {
  if (younger == 0) waitvm<0>(); else if (younger == 1) waitvm<8>(); else waitvm<16>();
}

struct Frag { i32x4 a[2][2]; i32x2 b[2][2][4]; };
}

__global__ __launch_bounds__(512, 2) void a2a_gemm_kernel(Args p) {
  extern __shared__ __attribute__((aligned(16))) char smem[];
  const u32 lds0 = (u32)(size_t)(LAS char*)smem;
  const u32 ctl = lds0 + LDS_CTL;
  const int b = blockIdx.x, lane = threadIdx.x & 63;
  const int wave = rfl(threadIdx.x >> 6);
  const int rank = p.rank;
  const int mt = (b >> 3) & 7, n0 = ((b & 7) * 4 + (b >> 6)) * 128;
  char* const myws = p.ws[rank];
#define STAMP(i)

  if (wave < 4) {
    if (wave == 0 && lane < 16) lds_st(ctl + lane * 4, 0u);
    const float scale = *p.sx * *p.sw;
    const int wm = wave >> 1, wn = wave & 1;
    u32 aoff[2][2], boff[2];
    {
      const int r = lane & 31, h = lane >> 5, g = lane >> 4, t = lane & 15;
      for (int ks = 0; ks < 2; ++ks) for (int hf = 0; hf < 2; ++hf)
        aoff[ks][hf] = lds0 + LDS_A + 128 * (32 * wm + r) + 16 * ((4 * ks + 2 * h + hf) ^ ((r >> 1) & 7));
      for (int nt = 0; nt < 2; ++nt)
        boff[nt] = lds0 + LDS_W + 128 * (32 * (g >> 1) + (t >> 1)) + 16 * ((4 * wn + 2 * nt + (g & 1)) ^ (2 * ((t >> 2) & 3))) + 8 * (t & 1);
    }
    asm volatile("s_waitcnt lgkmcnt(0)" ::: "memory");
    __builtin_amdgcn_s_barrier();
    __builtin_amdgcn_s_setprio(2);
    f32x16 acc[2] = {};
    Frag F0, F1;
    u32 as = 0;
    auto full = [&]() { u32x4 f = lds_ld4(ctl + C_FULL); return rfl(min(min(f.x, f.y), f.z)); };
    auto reads = [&](Frag& F, int s) {
      const u32 wo = (u32)(s % NWS) * 16384u, ao = as * 8192u;
      as = (as + 1 == NAS) ? 0 : as + 1;
      #pragma unroll
      for (int ks = 0; ks < 2; ++ks) {
        #pragma unroll
        for (int hf = 0; hf < 2; ++hf) F.a[ks][hf] = *(LAS i32x4*)(aoff[ks][hf] + ao);
        #pragma unroll
        for (int nt = 0; nt < 2; ++nt)
          #pragma unroll
          for (int i = 0; i < 4; ++i)
            F.b[ks][nt][i] = __builtin_amdgcn_ds_read_tr8_b64_v2i32((LAS i32x2*)(boff[nt] + wo + 8192 * ks + 1024 * i));
      }
    };
    auto mfmas = [&](const Frag& F) {
      #pragma unroll
      for (int ks = 0; ks < 2; ++ks) {
        i32x8 a = __builtin_shufflevector(F.a[ks][0], F.a[ks][1], 0, 1, 2, 3, 4, 5, 6, 7);
        #pragma unroll
        for (int nt = 0; nt < 2; ++nt) {
          i32x4 lo = __builtin_shufflevector(F.b[ks][nt][0], F.b[ks][nt][1], 0, 1, 2, 3);
          i32x4 hi = __builtin_shufflevector(F.b[ks][nt][2], F.b[ks][nt][3], 0, 1, 2, 3);
          i32x8 bb = __builtin_shufflevector(lo, hi, 0, 1, 2, 3, 4, 5, 6, 7);
          acc[nt] = __builtin_amdgcn_mfma_scale_f32_32x32x64_f8f6f4(a, bb, acc[nt], 1, 1, 0, 0, 0, 0);
        }
      }
    };
    auto step = [&](Frag& cur, Frag& nxt, int s) {
      if (full() > (u32)(s + 1)) {
        asm volatile("" ::: "memory");
        reads(nxt, s + 1);
        mfmas(cur);
        #pragma unroll
        for (int q = 0; q < 4; ++q) {
          __builtin_amdgcn_sched_group_barrier(0x8, 1, 0);
          __builtin_amdgcn_sched_group_barrier(0x100, 5, 0);
        }
        asm volatile("" ::: "memory");
      } else {
        mfmas(cur);
        __builtin_amdgcn_sched_barrier(0);
        while (full() <= (u32)(s + 1)) __builtin_amdgcn_s_sleep(1);
        asm volatile("" ::: "memory");
        reads(nxt, s + 1);
        asm volatile("" ::: "memory");
      }
      if (lane == 0) lds_st(ctl + C_FREE + wave * 4, (u32)(s + 2));
    };
    while (full() == 0) __builtin_amdgcn_s_sleep(1);
    asm volatile("" ::: "memory");
    if (wave == 0) STAMP(1);
    reads(F0, 0);
    asm volatile("" ::: "memory");
    if (lane == 0) lds_st(ctl + C_FREE + wave * 4, 1u);
    #pragma unroll 1
    for (int s = 0; s < 30; s += 2) {
      step(F0, F1, s);
      step(F1, F0, s + 1);
    }
    step(F0, F1, 30);
    mfmas(F1);
    if (wave == 0) STAMP(2);
    u32 st;
    while ((st = rfl(__hip_atomic_load((LAS u32*)(ctl + C_STAT), __ATOMIC_ACQUIRE, __HIP_MEMORY_SCOPE_WORKGROUP))) == 0) __builtin_amdgcn_s_sleep(1);
    const float sc = (st == 1) ? scale : __builtin_nanf("");
    float* o = p.out + (size_t)(mt * 64 + wm * 32 + 4 * (lane >> 5)) * GN + n0 + wn * 64 + (lane & 31);
    #pragma unroll
    for (int r = 0; r < 16; ++r)
      #pragma unroll
      for (int nt = 0; nt < 2; ++nt)
        o[(size_t)((r & 3) + 8 * (r >> 2)) * GN + nt * 32] = acc[nt][r] * sc;
  } else if (wave < 6) {
    const int h = wave - 4;
    u32 off[8];
    #pragma unroll
    for (int q = 0; q < 8; ++q) off[q] = (u32)(8 * q + (lane >> 3)) * GN + 16 * ((lane & 7) ^ (2 * ((lane >> 4) & 3)));
    const unsigned char* wb = p.w + (size_t)(64 * h) * GN + n0;
    int issued = 0, pub = 0;
    bool bar = false;
    while (pub < 32) {
      bool can = issued < 32 && issued - pub < 3;
      if (can && issued >= NWS) can = min_free(ctl) + NWS > (u32)issued;
      if (can) {
        int src, j; step_of(issued, rank, mt, src, j);
        dma8(off, wb + (size_t)(src * KS + j * 128) * GN, lds0 + LDS_W + (u32)(issued % NWS) * 16384u + h * 8192u);
        ++issued;
      } else if (issued > pub) {
        if (!bar) { __builtin_amdgcn_s_barrier(); bar = true; }
        wait_oldest(issued - pub - 1);
        ++pub;
        if (lane == 0) lds_st(ctl + C_FULL + h * 4, (u32)pub);
      } else {
        __builtin_amdgcn_s_sleep(1);
      }
    }
  } else if (wave == 6) {
    u32 offl[8], offr[8];
    #pragma unroll
    for (int q = 0; q < 8; ++q) {
      offl[q] = (u32)(8 * q + (lane >> 3)) * KS + 16 * ((lane & 7) ^ ((4 * q + (lane >> 4)) & 7));
      offr[q] = (u32)(q * 1024 + lane * 16);
    }
    const unsigned char* xl = p.x + (size_t)(rank * 512 + mt * 64) * KS;
    int issued = 0, pub = 0;
    u32 as = 0, mask = 0;
    bool bar = false;
    const char* slot = nullptr;
    while (pub < 32) {
      bool can = issued < 32 && issued - pub < 3;
      if (can && issued >= NAS) can = min_free(ctl) + NAS > (u32)issued;
      int src = 0, j = 0;
      if (can) {
        step_of(issued, rank, mt, src, j);
        if (issued >= 4 && !((mask >> (src * 4 + j)) & 1)) {
          mask = rfl(__hip_atomic_load((LAS u32*)(ctl + C_MASK), __ATOMIC_ACQUIRE, __HIP_MEMORY_SCOPE_WORKGROUP));
          can = (mask >> (src * 4 + j)) & 1;
          if (can && slot == nullptr) {
            u32 call = rfl(*(volatile LAS u32*)(ctl + C_CALL));
            slot = myws + OFF_SLOT + (size_t)(call & 1) * SLOT_BYTES + (size_t)mt * 8 * 4 * 8192;
          }
        }
      }
      if (can) {
        const u32 dst = lds0 + LDS_A + as * 8192u;
        if (issued < 4) dma8(offl, xl + j * 128, dst);
        else dma8_sys(offr, slot + (size_t)(src * 4 + j) * 8192, dst);
        as = (as + 1 == NAS) ? 0 : as + 1;
        ++issued;
      } else if (issued > pub) {
        if (!bar) { __builtin_amdgcn_s_barrier(); bar = true; }
        wait_oldest(issued - pub - 1);
        ++pub;
        if (lane == 0) lds_st(ctl + C_FULL + 8, (u32)pub);
      } else {
        __builtin_amdgcn_s_sleep(1);
      }
    }
  } else {
    const u64 t0 = __builtin_amdgcn_s_memrealtime();
    u32* cntp = (u32*)(myws + OFF_CNT + (size_t)b * 128);
    const bool pusher = b < 224;
    const int ps = b / 7, pmt = ps & 7, pj = (ps >> 3) & 3;
    const u32 dst = (u32)(rank + 1 + (b - ps * 7)) & 7u;
    u32x4 v[8];
    u32 c;
    {
      const unsigned char* sb = p.x + (size_t)(dst * 512 + pmt * 64) * KS + pj * 128;
      u32 o0 = (u32)(lane >> 3) * KS + 16 * ((lane & 7) ^ (lane >> 4));
      u32 o1 = (u32)(8 + (lane >> 3)) * KS + 16 * ((lane & 7) ^ (4 + (lane >> 4)));
      asm volatile(
        "global_load_dword %8, %9, off sc0 sc1\n\t"
        "global_load_dwordx4 %0, %10, %12\n\t"
        "global_load_dwordx4 %1, %11, %12\n\t"
        "global_load_dwordx4 %2, %10, %13\n\t"
        "global_load_dwordx4 %3, %11, %13\n\t"
        "global_load_dwordx4 %4, %10, %14\n\t"
        "global_load_dwordx4 %5, %11, %14\n\t"
        "global_load_dwordx4 %6, %10, %15\n\t"
        "global_load_dwordx4 %7, %11, %15\n\t"
        "s_barrier\n\t"
        "s_waitcnt vmcnt(0)"
        : "=&v"(v[0]), "=&v"(v[1]), "=&v"(v[2]), "=&v"(v[3]), "=&v"(v[4]), "=&v"(v[5]), "=&v"(v[6]), "=&v"(v[7]), "=&v"(c)
        : "v"(cntp), "v"(o0), "v"(o1), "s"(sb), "s"(sb + 16 * KS), "s"(sb + 32 * KS), "s"(sb + 48 * KS) : "memory");
    }
    const u32 call = rfl(c) + 1;
    if (lane == 0) lds_st(ctl + C_CALL, call);
    u32* tmo = (u32*)(myws + OFF_TMO);
    const char* fl = lane < 32 ? myws + OFF_FLAG + (size_t)(mt * 32 + lane) * 128 : (const char*)tmo;
    const u64 tgo = t0 + 45 + (u64)((ps > LEAD ? ps - LEAD : 0) * DELTA / 10);
    bool todo = pusher && dst < (u32)NRANK;
    u32 m = 0xFu << (rank * 4);
    bool bad = false;
    while (true) {
      if (todo && __builtin_amdgcn_s_memrealtime() >= tgo) {
        char* dws = p.ws[dst];
        char* d = dws + OFF_SLOT + (size_t)(call & 1) * SLOT_BYTES + (size_t)((pmt * 8 + rank) * 4 + pj) * 8192 + lane * 16;
        #pragma unroll
        for (int i = 0; i < 8; ++i)
          asm volatile("global_store_dwordx4 %0, %1, off sc0 sc1\n\ts_nop 1" :: "v"(d + i * 1024), "v"(v[i]) : "memory");
        waitvm<0>();
        if (lane == 0)
          __hip_atomic_store((u32*)(dws + OFF_FLAG + (size_t)(pmt * 32 + rank * 4 + pj) * 128), call, __ATOMIC_RELAXED, __HIP_MEMORY_SCOPE_SYSTEM);
        todo = false;
        STAMP(4);
      }
      if (m != 0xFFFFFFFFu) {
        const bool need = lane < 32 ? !((m >> lane) & 1) : lane == 32;
        u32 f = 0;
        if (need) f = __hip_atomic_load((const u32*)fl, __ATOMIC_RELAXED, __HIP_MEMORY_SCOPE_SYSTEM);
        const u64 ok = __ballot(need && (lane < 32 ? f >= call : f == call));
        u32 nm = m | (u32)ok;
        if ((ok >> 32) != 0 || __builtin_amdgcn_s_memrealtime() - t0 > DEADLINE) {
          if (lane == 0) __hip_atomic_store(tmo, call, __ATOMIC_RELAXED, __HIP_MEMORY_SCOPE_AGENT);
          bad = true; nm = 0xFFFFFFFFu;
        }
        if (nm != m) {
          m = nm;
          if (lane == 0) __hip_atomic_store((LAS u32*)(ctl + C_MASK), m, __ATOMIC_RELEASE, __HIP_MEMORY_SCOPE_WORKGROUP);
          if (m == 0xFFFFFFFFu) {
            STAMP(5);
            bad = bad || __hip_atomic_load(tmo, __ATOMIC_RELAXED, __HIP_MEMORY_SCOPE_AGENT) == call;
            if (lane == 0) __hip_atomic_store((LAS u32*)(ctl + C_STAT), bad ? 2u : 1u, __ATOMIC_RELEASE, __HIP_MEMORY_SCOPE_WORKGROUP);
          }
        }
      } else if (!todo) break;
      __builtin_amdgcn_s_sleep(2);
    }
    if (lane == 0) __hip_atomic_store(cntp, call, __ATOMIC_RELAXED, __HIP_MEMORY_SCOPE_SYSTEM);
  }
}

extern "C" void kernel_launch(int rank, int nranks, ncclComm_t comm, hipStream_t stream,
                              void* const* d_in, const int* in_sizes, int n_in,
                              void* d_out, int out_size, const int* in_dtypes, int out_dtype,
                              void* const* d_out_all, void* const* d_ws_all, size_t ws_bytes) {
  if (nranks != NRANK || (unsigned)rank >= (unsigned)NRANK || ws_bytes < WS_NEED || n_in < 4) {
    fprintf(stderr, "unexpected configuration\n");
    return;
  }
  Args a;
  a.x = (const unsigned char*)d_in[0]; a.w = (const unsigned char*)d_in[1];
  a.sx = (const float*)d_in[2]; a.sw = (const float*)d_in[3]; a.out = (float*)d_out;
  for (int q = 0; q < NRANK; ++q) a.ws[q] = (char*)d_ws_all[q];
  a.rank = rank; a.pad = 0;
  HIP_CHECK(hipFuncSetAttribute((const void*)a2a_gemm_kernel, hipFuncAttributeMaxDynamicSharedMemorySize, LDS_BYTES));
  a2a_gemm_kernel<<<256, 512, LDS_BYTES, stream>>>(a);
  HIP_CHECK(hipGetLastError());
}
```

```cpp
#include <hip/hip_runtime.h>
#include <rccl/rccl.h>
#include <cstdio>
#include <type_traits>

#define HIP_CHECK(expr) do { hipError_t _e = (expr); if (_e != hipSuccess) { \
    fprintf(stderr, "HIP error %s at %s:%d\n", hipGetErrorString(_e), __FILE__, __LINE__); \
    return; } } while (0)

typedef unsigned u32;
typedef unsigned long long u64;
typedef unsigned u32x4 __attribute__((ext_vector_type(4)));
typedef int i32x2 __attribute__((ext_vector_type(2)));
typedef int i32x4 __attribute__((ext_vector_type(4)));
typedef int i32x8 __attribute__((ext_vector_type(8)));
typedef float f32x4 __attribute__((ext_vector_type(4)));
#define LAS __attribute__((address_space(3)))

#define DELTA 150
#define LEAD 150
#define TSTART 800
#define NWS 7
#define NAS 4
#define ROTM 1
#define ROTN 0
#define ABL_NOCONS 0
#define ABL_NOLOAD 0
#define CMAP 0
#define LDSLEEP 4
#define V_PUT 1
#define GDELAY 0
#define PROT 1
#define RPG 2
#define ABL_NOCOMM 0

namespace {
constexpr int NRANK = 8, GN = 4096, KS = 512;
constexpr size_t OFF_CNT = 0;
constexpr size_t OFF_FLAG = 64 << 10;
constexpr size_t OFF_TMO = 112 << 10;
constexpr size_t OFF_DTAB = 32 << 10;
constexpr size_t OFF_DBG = 128 << 10;
constexpr size_t OFF_SLOT = 1 << 20;
constexpr u32 PIECE = 10240;
constexpr size_t SLOT_BYTES = 4u << 20;
constexpr size_t WS_NEED = OFF_SLOT + 2 * SLOT_BYTES;
constexpr u64 DEADLINE = 40000000ull;

constexpr u32 LDS_W = 0, LDS_A = NWS * 16384, LDS_CTL = LDS_A + NAS * PIECE, LDS_BYTES = LDS_CTL + 256;
constexpr u32 C_FLAGS = 0  , C_STAT = 128, C_DESC = 192  ;
constexpr u32 D_NULL = 0x2007;
static_assert(LDS_BYTES <= 163840 && NAS == 4 && NWS == 7, "LDS budget");

struct Args {
  const unsigned char* x; const unsigned char* w; const float* sx; const float* sw; float* out;
  char* myws; int rank; int pad;
  struct { const unsigned char* xs; char* ws; } peer[NRANK];
};

template <int N> __device__ __forceinline__ void waitvm() { asm volatile("s_waitcnt vmcnt(%0)" :: "i"(N) : "memory"); }
__device__ __forceinline__ void bar() { asm volatile("s_barrier" ::: "memory"); }
__device__ __forceinline__ u32 rfl(u32 v) { return __builtin_amdgcn_readfirstlane(v); }

#define DMA4(A, B, C, D, BASE, SC) \
    "global_load_lds_dwordx4 " A ", " BASE SC "\n\t" \
    "global_load_lds_dwordx4 " B ", " BASE " offset:1024" SC "\n\t" \
    "global_load_lds_dwordx4 " C ", " BASE " offset:2048" SC "\n\t" \
    "global_load_lds_dwordx4 " D ", " BASE " offset:3072" SC "\n\t"
#define M0NEXT "s_add_u32 m0, m0, 0x1000\n\ts_nop 0\n\t"
#define DMA8(SC) \
  asm volatile("s_mov_b32 m0, %9\n\ts_nop 0\n\t" DMA4("%0", "%1", "%2", "%3", "%8", SC) M0NEXT DMA4("%4", "%5", "%6", "%7", "%8", SC) \
    :: "v"(off[0]), "v"(off[1]), "v"(off[2]), "v"(off[3]), "v"(off[4]), "v"(off[5]), "v"(off[6]), "v"(off[7]), \
       "s"(base), "s"(dst) : "memory", "scc")
#define DMA10(SC) \
  asm volatile("s_mov_b32 m0, %11\n\ts_nop 0\n\t" DMA4("%0", "%1", "%2", "%3", "%10", SC) M0NEXT DMA4("%4", "%5", "%6", "%7", "%10", SC) M0NEXT \
    "global_load_lds_dwordx4 %8, %10" SC "\n\tglobal_load_lds_dwordx4 %9, %10 offset:1024" SC \
    :: "v"(off[0]), "v"(off[1]), "v"(off[2]), "v"(off[3]), "v"(off[4]), "v"(off[5]), "v"(off[6]), "v"(off[7]), "v"(off[8]), "v"(off[9]), \
       "s"(base), "s"(dst) : "memory", "scc")
template <int N, bool SYS> __device__ __forceinline__ void dma(const u32 (&off)[10], const void* base, u32 dst) {
  if constexpr (N == 8) { if constexpr (SYS) DMA8(" sc0 sc1"); else DMA8(""); }
  else { if constexpr (SYS) DMA10(" sc0 sc1"); else DMA10(""); }
}

__device__ __forceinline__ int src_of(int q, int rank, int rot) { int i = q + rot; if (i >= 7) i -= 7; return (rank + 1 + i) & 7; }
__device__ __forceinline__ int row0_of(int mt) { return mt < 4 ? 80 * mt : 64 * mt + 64; }

template <int N, int PER> __device__ __forceinline__ void tail_bars() {
  if constexpr (N > 0) { waitvm<PER * (N - 1)>(); bar(); tail_bars<N - 1, PER>(); }
}

#define STAMP(i)
#define DBGARGS
#define DBGPASS

#define CONS_ASM4 \
    "\ts_waitcnt lgkmcnt(0)\n" \
    "\ts_barrier\n" \
    "\tds_read_b32 %[vd], %[dp]\n" \
    "\tds_read_b64_tr_b8 v[64:65], %[wa0] offset:0\n" \
    "\tds_read_b64_tr_b8 v[66:67], %[wa0] offset:1024\n" \
    "\tds_read_b64_tr_b8 v[68:69], %[wa0] offset:8192\n" \
    "\tds_read_b64_tr_b8 v[70:71], %[wa0] offset:9216\n" \
    "\tds_read_b64_tr_b8 v[72:73], %[wa1] offset:0\n" \
    "\tds_read_b64_tr_b8 v[74:75], %[wa1] offset:1024\n" \
    "\tds_read_b64_tr_b8 v[76:77], %[wa1] offset:8192\n" \
    "\tds_read_b64_tr_b8 v[78:79], %[wa1] offset:9216\n" \
    "\ts_waitcnt lgkmcnt(8)\n" \
    "\tds_read_b128 v[80:83], %[ao0] offset:0\n" \
    "\tds_read_b128 v[84:87], %[ao1] offset:0\n" \
    "\tds_read_b128 v[88:91], %[ao0] offset:2048\n" \
    "\tds_read_b128 v[92:95], %[ao1] offset:2048\n" \
    "\tds_read_b128 v[96:99], %[ao0] offset:4096\n" \
    "\tds_read_b128 v[100:103], %[ao1] offset:4096\n" \
    "\tds_read_b128 v[104:107], %[ao0] offset:6144\n" \
    "\tds_read_b128 v[108:111], %[ao1] offset:6144\n" \
    "\tv_cmp_eq_u32 vcc, 0x2007, %[vd]\n" \
    "\tv_and_b32 %[t], 7, %[vd]\n" \
    "\tv_lshl_add_u32 %[wa0], %[t], 14, %[bo0]\n" \
    "\tv_lshl_add_u32 %[wa1], %[t], 14, %[bo1]\n" \
    "\tv_add_u32 %[dp], 4, %[dp]\n" \
    "\tv_bfi_b32 %[dp], 63, %[dp], %[dbase]\n" \
    "3:\n" \
    "\ts_cbranch_vccnz 10f\n" \
    "20:\n" \
    "\ts_waitcnt lgkmcnt(0)\n" \
    "\ts_barrier\n" \
    "\tv_mfma_f32_16x16x128_f8f6f4 %[c0], v[64:71], v[80:87], %[c0] cbsz:1 blgp:1\n" \
    "\tds_read_b64_tr_b8 v[120:121], %[wa0] offset:0\n" \
    "\tds_read_b64_tr_b8 v[122:123], %[wa0] offset:1024\n" \
    "\tds_read_b64_tr_b8 v[124:125], %[wa0] offset:8192\n" \
    "\tds_read_b64_tr_b8 v[126:127], %[wa0] offset:9216\n" \
    "\tv_mfma_f32_16x16x128_f8f6f4 %[c1], v[72:79], v[80:87], %[c1] cbsz:1 blgp:1\n" \
    "\tds_read_b64_tr_b8 v[128:129], %[wa1] offset:0\n" \
    "\tds_read_b64_tr_b8 v[130:131], %[wa1] offset:1024\n" \
    "\tds_read_b64_tr_b8 v[132:133], %[wa1] offset:8192\n" \
    "\tds_read_b64_tr_b8 v[134:135], %[wa1] offset:9216\n" \
    "\tv_mfma_f32_16x16x128_f8f6f4 %[c2], v[64:71], v[88:95], %[c2] cbsz:1 blgp:1\n" \
    "\tds_read_b32 %[vd], %[dp]\n" \
    "\tds_read_b128 v[136:139], %[ao0] offset:10240\n" \
    "\tds_read_b128 v[140:143], %[ao1] offset:10240\n" \
    "\tv_mfma_f32_16x16x128_f8f6f4 %[c3], v[72:79], v[88:95], %[c3] cbsz:1 blgp:1\n" \
    "\tds_read_b128 v[144:147], %[ao0] offset:12288\n" \
    "\tds_read_b128 v[148:151], %[ao1] offset:12288\n" \
    "\tv_mfma_f32_16x16x128_f8f6f4 %[c4], v[64:71], v[96:103], %[c4] cbsz:1 blgp:1\n" \
    "\tds_read_b128 v[152:155], %[ao0] offset:14336\n" \
    "\tds_read_b128 v[156:159], %[ao1] offset:14336\n" \
    "\tv_mfma_f32_16x16x128_f8f6f4 %[c5], v[72:79], v[96:103], %[c5] cbsz:1 blgp:1\n" \
    "\tds_read_b128 v[152:155], %[ao0] offset:14336\n" \
    "\tds_read_b128 v[156:159], %[ao1] offset:14336\n" \
    "\ts_waitcnt lgkmcnt(8)\n" \
    "\tv_and_b32 %[t], 7, %[vd]\n" \
    "\tv_cmp_eq_u32 vcc, 0x2007, %[vd]\n" \
    "\tv_mfma_f32_16x16x128_f8f6f4 %[c6], v[64:71], v[104:111], %[c6] cbsz:1 blgp:1\n" \
    "\tv_lshl_add_u32 %[wa0], %[t], 14, %[bo0]\n" \
    "\tv_lshl_add_u32 %[wa1], %[t], 14, %[bo1]\n" \
    "\tv_add_u32 %[dp], 4, %[dp]\n" \
    "\tv_bfi_b32 %[dp], 63, %[dp], %[dbase]\n" \
    "\tv_mfma_f32_16x16x128_f8f6f4 %[c7], v[72:79], v[104:111], %[c7] cbsz:1 blgp:1\n" \
    "\ts_cbranch_vccnz 11f\n" \
    "21:\n" \
    "\ts_waitcnt lgkmcnt(0)\n" \
    "\ts_barrier\n" \
    "\tv_mfma_f32_16x16x128_f8f6f4 %[c0], v[120:127], v[136:143], %[c0] cbsz:1 blgp:1\n" \
    "\tds_read_b64_tr_b8 v[64:65], %[wa0] offset:0\n" \
    "\tds_read_b64_tr_b8 v[66:67], %[wa0] offset:1024\n" \
    "\tds_read_b64_tr_b8 v[68:69], %[wa0] offset:8192\n" \
    "\tds_read_b64_tr_b8 v[70:71], %[wa0] offset:9216\n" \
    "\tv_mfma_f32_16x16x128_f8f6f4 %[c1], v[128:135], v[136:143], %[c1] cbsz:1 blgp:1\n" \
    "\tds_read_b64_tr_b8 v[72:73], %[wa1] offset:0\n" \
    "\tds_read_b64_tr_b8 v[74:75], %[wa1] offset:1024\n" \
    "\tds_read_b64_tr_b8 v[76:77], %[wa1] offset:8192\n" \
    "\tds_read_b64_tr_b8 v[78:79], %[wa1] offset:9216\n" \
    "\tv_mfma_f32_16x16x128_f8f6f4 %[c2], v[120:127], v[144:151], %[c2] cbsz:1 blgp:1\n" \
    "\tds_read_b32 %[vd], %[dp]\n" \
    "\tds_read_b128 v[80:83], %[ao0] offset:20480\n" \
    "\tds_read_b128 v[84:87], %[ao1] offset:20480\n" \
    "\tv_mfma_f32_16x16x128_f8f6f4 %[c3], v[128:135], v[144:151], %[c3] cbsz:1 blgp:1\n" \
    "\tds_read_b128 v[88:91], %[ao0] offset:22528\n" \
    "\tds_read_b128 v[92:95], %[ao1] offset:22528\n" \
    "\tv_mfma_f32_16x16x128_f8f6f4 %[c4], v[120:127], v[152:159], %[c4] cbsz:1 blgp:1\n" \
    "\tds_read_b128 v[96:99], %[ao0] offset:24576\n" \
    "\tds_read_b128 v[100:103], %[ao1] offset:24576\n" \
    "\tv_mfma_f32_16x16x128_f8f6f4 %[c5], v[128:135], v[152:159], %[c5] cbsz:1 blgp:1\n" \
    "\tds_read_b128 v[96:99], %[ao0] offset:24576\n" \
    "\tds_read_b128 v[100:103], %[ao1] offset:24576\n" \
    "\ts_waitcnt lgkmcnt(8)\n" \
    "\tv_and_b32 %[t], 7, %[vd]\n" \
    "\tv_cmp_eq_u32 vcc, 0x2007, %[vd]\n" \
    "\tv_mfma_f32_16x16x128_f8f6f4 %[c6], v[120:127], v[160:167], %[c6] cbsz:1 blgp:1\n" \
    "\tv_lshl_add_u32 %[wa0], %[t], 14, %[bo0]\n" \
    "\tv_lshl_add_u32 %[wa1], %[t], 14, %[bo1]\n" \
    "\tv_add_u32 %[dp], 4, %[dp]\n" \
    "\tv_bfi_b32 %[dp], 63, %[dp], %[dbase]\n" \
    "\tv_mfma_f32_16x16x128_f8f6f4 %[c7], v[128:135], v[160:167], %[c7] cbsz:1 blgp:1\n" \
    "\ts_cbranch_vccnz 12f\n" \
    "22:\n" \
    "\ts_waitcnt lgkmcnt(0)\n" \
    "\ts_barrier\n" \
    "\tv_mfma_f32_16x16x128_f8f6f4 %[c0], v[64:71], v[80:87], %[c0] cbsz:1 blgp:1\n" \
    "\tds_read_b64_tr_b8 v[120:121], %[wa0] offset:0\n" \
    "\tds_read_b64_tr_b8 v[122:123], %[wa0] offset:1024\n" \
    "\tds_read_b64_tr_b8 v[124:125], %[wa0] offset:8192\n" \
    "\tds_read_b64_tr_b8 v[126:127], %[wa0] offset:9216\n" \
    "\tv_mfma_f32_16x16x128_f8f6f4 %[c1], v[72:79], v[80:87], %[c1] cbsz:1 blgp:1\n" \
    "\tds_read_b64_tr_b8 v[128:129], %[wa1] offset:0\n" \
    "\tds_read_b64_tr_b8 v[130:131], %[wa1] offset:1024\n" \
    "\tds_read_b64_tr_b8 v[132:133], %[wa1] offset:8192\n" \
    "\tds_read_b64_tr_b8 v[134:135], %[wa1] offset:9216\n" \
    "\tv_mfma_f32_16x16x128_f8f6f4 %[c2], v[64:71], v[88:95], %[c2] cbsz:1 blgp:1\n" \
    "\tds_read_b32 %[vd], %[dp]\n" \
    "\tds_read_b128 v[136:139], %[ao0] offset:30720\n" \
    "\tds_read_b128 v[140:143], %[ao1] offset:30720\n" \
    "\tv_mfma_f32_16x16x128_f8f6f4 %[c3], v[72:79], v[88:95], %[c3] cbsz:1 blgp:1\n" \
    "\tds_read_b128 v[144:147], %[ao0] offset:32768\n" \
    "\tds_read_b128 v[148:151], %[ao1] offset:32768\n" \
    "\tv_mfma_f32_16x16x128_f8f6f4 %[c4], v[64:71], v[96:103], %[c4] cbsz:1 blgp:1\n" \
    "\tds_read_b128 v[152:155], %[ao0] offset:34816\n" \
    "\tds_read_b128 v[156:159], %[ao1] offset:34816\n" \
    "\tv_mfma_f32_16x16x128_f8f6f4 %[c5], v[72:79], v[96:103], %[c5] cbsz:1 blgp:1\n" \
    "\tds_read_b128 v[152:155], %[ao0] offset:34816\n" \
    "\tds_read_b128 v[156:159], %[ao1] offset:34816\n" \
    "\ts_waitcnt lgkmcnt(8)\n" \
    "\tv_and_b32 %[t], 7, %[vd]\n" \
    "\tv_cmp_eq_u32 vcc, 0x2007, %[vd]\n" \
    "\tv_mfma_f32_16x16x128_f8f6f4 %[c6], v[64:71], v[104:111], %[c6] cbsz:1 blgp:1\n" \
    "\tv_lshl_add_u32 %[wa0], %[t], 14, %[bo0]\n" \
    "\tv_lshl_add_u32 %[wa1], %[t], 14, %[bo1]\n" \
    "\tv_add_u32 %[dp], 4, %[dp]\n" \
    "\tv_bfi_b32 %[dp], 63, %[dp], %[dbase]\n" \
    "\ts_sub_u32 %[n], %[n], 1\n" \
    "\ts_cmp_eq_u32 %[n], 0\n" \
    "\tv_mfma_f32_16x16x128_f8f6f4 %[c7], v[72:79], v[104:111], %[c7] cbsz:1 blgp:1\n" \
    "\ts_cbranch_scc1 4f\n" \
    "\ts_cbranch_vccnz 13f\n" \
    "23:\n" \
    "\ts_waitcnt lgkmcnt(0)\n" \
    "\ts_barrier\n" \
    "\tv_mfma_f32_16x16x128_f8f6f4 %[c0], v[120:127], v[136:143], %[c0] cbsz:1 blgp:1\n" \
    "\tds_read_b64_tr_b8 v[64:65], %[wa0] offset:0\n" \
    "\tds_read_b64_tr_b8 v[66:67], %[wa0] offset:1024\n" \
    "\tds_read_b64_tr_b8 v[68:69], %[wa0] offset:8192\n" \
    "\tds_read_b64_tr_b8 v[70:71], %[wa0] offset:9216\n" \
    "\tv_mfma_f32_16x16x128_f8f6f4 %[c1], v[128:135], v[136:143], %[c1] cbsz:1 blgp:1\n" \
    "\tds_read_b64_tr_b8 v[72:73], %[wa1] offset:0\n" \
    "\tds_read_b64_tr_b8 v[74:75], %[wa1] offset:1024\n" \
    "\tds_read_b64_tr_b8 v[76:77], %[wa1] offset:8192\n" \
    "\tds_read_b64_tr_b8 v[78:79], %[wa1] offset:9216\n" \
    "\tv_mfma_f32_16x16x128_f8f6f4 %[c2], v[120:127], v[144:151], %[c2] cbsz:1 blgp:1\n" \
    "\tds_read_b32 %[vd], %[dp]\n" \
    "\tds_read_b128 v[80:83], %[ao0] offset:0\n" \
    "\tds_read_b128 v[84:87], %[ao1] offset:0\n" \
    "\tv_mfma_f32_16x16x128_f8f6f4 %[c3], v[128:135], v[144:151], %[c3] cbsz:1 blgp:1\n" \
    "\tds_read_b128 v[88:91], %[ao0] offset:2048\n" \
    "\tds_read_b128 v[92:95], %[ao1] offset:2048\n" \
    "\tv_mfma_f32_16x16x128_f8f6f4 %[c4], v[120:127], v[152:159], %[c4] cbsz:1 blgp:1\n" \
    "\tds_read_b128 v[96:99], %[ao0] offset:4096\n" \
    "\tds_read_b128 v[100:103], %[ao1] offset:4096\n" \
    "\tv_mfma_f32_16x16x128_f8f6f4 %[c5], v[128:135], v[152:159], %[c5] cbsz:1 blgp:1\n" \
    "\tds_read_b128 v[96:99], %[ao0] offset:4096\n" \
    "\tds_read_b128 v[100:103], %[ao1] offset:4096\n" \
    "\ts_waitcnt lgkmcnt(8)\n" \
    "\tv_and_b32 %[t], 7, %[vd]\n" \
    "\tv_cmp_eq_u32 vcc, 0x2007, %[vd]\n" \
    "\tv_mfma_f32_16x16x128_f8f6f4 %[c6], v[120:127], v[160:167], %[c6] cbsz:1 blgp:1\n" \
    "\tv_lshl_add_u32 %[wa0], %[t], 14, %[bo0]\n" \
    "\tv_lshl_add_u32 %[wa1], %[t], 14, %[bo1]\n" \
    "\tv_add_u32 %[dp], 4, %[dp]\n" \
    "\tv_bfi_b32 %[dp], 63, %[dp], %[dbase]\n" \
    "\tv_mfma_f32_16x16x128_f8f6f4 %[c7], v[128:135], v[160:167], %[c7] cbsz:1 blgp:1\n" \
    "\ts_branch 3b\n" \
    "10:\n" \
    "\ts_waitcnt lgkmcnt(0)\n" \
    "\ts_barrier\n" \
    "\tds_read_b32 %[vd], %[dp]\n" \
    "\tv_add_u32 %[dp], 4, %[dp]\n" \
    "\tv_bfi_b32 %[dp], 63, %[dp], %[dbase]\n" \
    "\ts_waitcnt lgkmcnt(0)\n" \
    "\tv_cmp_eq_u32 vcc, 0x2007, %[vd]\n" \
    "\tv_and_b32 %[t], 7, %[vd]\n" \
    "\tv_lshl_add_u32 %[wa0], %[t], 14, %[bo0]\n" \
    "\tv_lshl_add_u32 %[wa1], %[t], 14, %[bo1]\n" \
    "\ts_cbranch_vccnz 10b\n" \
    "\ts_branch 20b\n" \
    "11:\n" \
    "\ts_waitcnt lgkmcnt(0)\n" \
    "\ts_barrier\n" \
    "\tds_read_b32 %[vd], %[dp]\n" \
    "\tv_add_u32 %[dp], 4, %[dp]\n" \
    "\tv_bfi_b32 %[dp], 63, %[dp], %[dbase]\n" \
    "\ts_waitcnt lgkmcnt(0)\n" \
    "\tv_cmp_eq_u32 vcc, 0x2007, %[vd]\n" \
    "\tv_and_b32 %[t], 7, %[vd]\n" \
    "\tv_lshl_add_u32 %[wa0], %[t], 14, %[bo0]\n" \
    "\tv_lshl_add_u32 %[wa1], %[t], 14, %[bo1]\n" \
    "\ts_cbranch_vccnz 11b\n" \
    "\ts_branch 21b\n" \
    "12:\n" \
    "\ts_waitcnt lgkmcnt(0)\n" \
    "\ts_barrier\n" \
    "\tds_read_b32 %[vd], %[dp]\n" \
    "\tv_add_u32 %[dp], 4, %[dp]\n" \
    "\tv_bfi_b32 %[dp], 63, %[dp], %[dbase]\n" \
    "\ts_waitcnt lgkmcnt(0)\n" \
    "\tv_cmp_eq_u32 vcc, 0x2007, %[vd]\n" \
    "\tv_and_b32 %[t], 7, %[vd]\n" \
    "\tv_lshl_add_u32 %[wa0], %[t], 14, %[bo0]\n" \
    "\tv_lshl_add_u32 %[wa1], %[t], 14, %[bo1]\n" \
    "\ts_cbranch_vccnz 12b\n" \
    "\ts_branch 22b\n" \
    "13:\n" \
    "\ts_waitcnt lgkmcnt(0)\n" \
    "\ts_barrier\n" \
    "\tds_read_b32 %[vd], %[dp]\n" \
    "\tv_add_u32 %[dp], 4, %[dp]\n" \
    "\tv_bfi_b32 %[dp], 63, %[dp], %[dbase]\n" \
    "\ts_waitcnt lgkmcnt(0)\n" \
    "\tv_cmp_eq_u32 vcc, 0x2007, %[vd]\n" \
    "\tv_and_b32 %[t], 7, %[vd]\n" \
    "\tv_lshl_add_u32 %[wa0], %[t], 14, %[bo0]\n" \
    "\tv_lshl_add_u32 %[wa1], %[t], 14, %[bo1]\n" \
    "\ts_cbranch_vccnz 13b\n" \
    "\ts_branch 23b\n" \
    "4:\n" \
    "\tds_read_b32 %[vd], %[stat]\n" \
    "\ts_waitcnt lgkmcnt(1)\n" \
    "\tv_mfma_f32_16x16x128_f8f6f4 %[c0], v[120:127], v[136:143], %[c0] cbsz:1 blgp:1\n" \
    "\tv_mfma_f32_16x16x128_f8f6f4 %[c1], v[128:135], v[136:143], %[c1] cbsz:1 blgp:1\n" \
    "\tv_mfma_f32_16x16x128_f8f6f4 %[c2], v[120:127], v[144:151], %[c2] cbsz:1 blgp:1\n" \
    "\tv_mfma_f32_16x16x128_f8f6f4 %[c3], v[128:135], v[144:151], %[c3] cbsz:1 blgp:1\n" \
    "\tv_mfma_f32_16x16x128_f8f6f4 %[c4], v[120:127], v[152:159], %[c4] cbsz:1 blgp:1\n" \
    "\tv_mfma_f32_16x16x128_f8f6f4 %[c5], v[128:135], v[152:159], %[c5] cbsz:1 blgp:1\n" \
    "\tv_mfma_f32_16x16x128_f8f6f4 %[c6], v[120:127], v[160:167], %[c6] cbsz:1 blgp:1\n" \
    "\tv_mfma_f32_16x16x128_f8f6f4 %[c7], v[128:135], v[160:167], %[c7] cbsz:1 blgp:1\n" \
    "\ts_waitcnt lgkmcnt(0)\n" \
    "\ts_nop 7\n" \
    "\ts_nop 3\n"
#define CONS_ASM5 \
    "\ts_waitcnt lgkmcnt(0)\n" \
    "\ts_barrier\n" \
    "\tds_read_b32 %[vd], %[dp]\n" \
    "\tds_read_b64_tr_b8 v[64:65], %[wa0] offset:0\n" \
    "\tds_read_b64_tr_b8 v[66:67], %[wa0] offset:1024\n" \
    "\tds_read_b64_tr_b8 v[68:69], %[wa0] offset:8192\n" \
    "\tds_read_b64_tr_b8 v[70:71], %[wa0] offset:9216\n" \
    "\tds_read_b64_tr_b8 v[72:73], %[wa1] offset:0\n" \
    "\tds_read_b64_tr_b8 v[74:75], %[wa1] offset:1024\n" \
    "\tds_read_b64_tr_b8 v[76:77], %[wa1] offset:8192\n" \
    "\tds_read_b64_tr_b8 v[78:79], %[wa1] offset:9216\n" \
    "\ts_waitcnt lgkmcnt(8)\n" \
    "\tds_read_b128 v[80:83], %[ao0] offset:0\n" \
    "\tds_read_b128 v[84:87], %[ao1] offset:0\n" \
    "\tds_read_b128 v[88:91], %[ao0] offset:2048\n" \
    "\tds_read_b128 v[92:95], %[ao1] offset:2048\n" \
    "\tds_read_b128 v[96:99], %[ao0] offset:4096\n" \
    "\tds_read_b128 v[100:103], %[ao1] offset:4096\n" \
    "\tds_read_b128 v[104:107], %[ao0] offset:6144\n" \
    "\tds_read_b128 v[108:111], %[ao1] offset:6144\n" \
    "\tds_read_b128 v[112:115], %[ao0] offset:8192\n" \
    "\tds_read_b128 v[116:119], %[ao1] offset:8192\n" \
    "\tv_cmp_eq_u32 vcc, 0x2007, %[vd]\n" \
    "\tv_and_b32 %[t], 7, %[vd]\n" \
    "\tv_lshl_add_u32 %[wa0], %[t], 14, %[bo0]\n" \
    "\tv_lshl_add_u32 %[wa1], %[t], 14, %[bo1]\n" \
    "\tv_add_u32 %[dp], 4, %[dp]\n" \
    "\tv_bfi_b32 %[dp], 63, %[dp], %[dbase]\n" \
    "3:\n" \
    "\ts_cbranch_vccnz 10f\n" \
    "20:\n" \
    "\ts_waitcnt lgkmcnt(0)\n" \
    "\ts_barrier\n" \
    "\tv_mfma_f32_16x16x128_f8f6f4 %[c0], v[64:71], v[80:87], %[c0] cbsz:1 blgp:1\n" \
    "\tds_read_b64_tr_b8 v[120:121], %[wa0] offset:0\n" \
    "\tds_read_b64_tr_b8 v[122:123], %[wa0] offset:1024\n" \
    "\tds_read_b64_tr_b8 v[124:125], %[wa0] offset:8192\n" \
    "\tds_read_b64_tr_b8 v[126:127], %[wa0] offset:9216\n" \
    "\tv_mfma_f32_16x16x128_f8f6f4 %[c1], v[72:79], v[80:87], %[c1] cbsz:1 blgp:1\n" \
    "\tds_read_b64_tr_b8 v[128:129], %[wa1] offset:0\n" \
    "\tds_read_b64_tr_b8 v[130:131], %[wa1] offset:1024\n" \
    "\tds_read_b64_tr_b8 v[132:133], %[wa1] offset:8192\n" \
    "\tds_read_b64_tr_b8 v[134:135], %[wa1] offset:9216\n" \
    "\tv_mfma_f32_16x16x128_f8f6f4 %[c2], v[64:71], v[88:95], %[c2] cbsz:1 blgp:1\n" \
    "\tds_read_b32 %[vd], %[dp]\n" \
    "\tds_read_b128 v[136:139], %[ao0] offset:10240\n" \
    "\tds_read_b128 v[140:143], %[ao1] offset:10240\n" \
    "\tv_mfma_f32_16x16x128_f8f6f4 %[c3], v[72:79], v[88:95], %[c3] cbsz:1 blgp:1\n" \
    "\tds_read_b128 v[144:147], %[ao0] offset:12288\n" \
    "\tds_read_b128 v[148:151], %[ao1] offset:12288\n" \
    "\tv_mfma_f32_16x16x128_f8f6f4 %[c4], v[64:71], v[96:103], %[c4] cbsz:1 blgp:1\n" \
    "\tds_read_b128 v[152:155], %[ao0] offset:14336\n" \
    "\tds_read_b128 v[156:159], %[ao1] offset:14336\n" \
    "\tv_mfma_f32_16x16x128_f8f6f4 %[c5], v[72:79], v[96:103], %[c5] cbsz:1 blgp:1\n" \
    "\tds_read_b128 v[152:155], %[ao0] offset:14336\n" \
    "\tds_read_b128 v[156:159], %[ao1] offset:14336\n" \
    "\tv_mfma_f32_16x16x128_f8f6f4 %[c6], v[64:71], v[104:111], %[c6] cbsz:1 blgp:1\n" \
    "\tds_read_b128 v[160:163], %[ao0] offset:16384\n" \
    "\tds_read_b128 v[164:167], %[ao1] offset:16384\n" \
    "\tv_mfma_f32_16x16x128_f8f6f4 %[c7], v[72:79], v[104:111], %[c7] cbsz:1 blgp:1\n" \
    "\ts_waitcnt lgkmcnt(10)\n" \
    "\tv_and_b32 %[t], 7, %[vd]\n" \
    "\tv_lshl_add_u32 %[wa0], %[t], 14, %[bo0]\n" \
    "\tv_lshl_add_u32 %[wa1], %[t], 14, %[bo1]\n" \
    "\tv_mfma_f32_16x16x128_f8f6f4 %[c8], v[64:71], v[112:119], %[c8] cbsz:1 blgp:1\n" \
    "\tv_cmp_eq_u32 vcc, 0x2007, %[vd]\n" \
    "\tv_add_u32 %[dp], 4, %[dp]\n" \
    "\tv_bfi_b32 %[dp], 63, %[dp], %[dbase]\n" \
    "\tv_mfma_f32_16x16x128_f8f6f4 %[c9], v[72:79], v[112:119], %[c9] cbsz:1 blgp:1\n" \
    "\ts_cbranch_vccnz 11f\n" \
    "21:\n" \
    "\ts_waitcnt lgkmcnt(0)\n" \
    "\ts_barrier\n" \
    "\tv_mfma_f32_16x16x128_f8f6f4 %[c0], v[120:127], v[136:143], %[c0] cbsz:1 blgp:1\n" \
    "\tds_read_b64_tr_b8 v[64:65], %[wa0] offset:0\n" \
    "\tds_read_b64_tr_b8 v[66:67], %[wa0] offset:1024\n" \
    "\tds_read_b64_tr_b8 v[68:69], %[wa0] offset:8192\n" \
    "\tds_read_b64_tr_b8 v[70:71], %[wa0] offset:9216\n" \
    "\tv_mfma_f32_16x16x128_f8f6f4 %[c1], v[128:135], v[136:143], %[c1] cbsz:1 blgp:1\n" \
    "\tds_read_b64_tr_b8 v[72:73], %[wa1] offset:0\n" \
    "\tds_read_b64_tr_b8 v[74:75], %[wa1] offset:1024\n" \
    "\tds_read_b64_tr_b8 v[76:77], %[wa1] offset:8192\n" \
    "\tds_read_b64_tr_b8 v[78:79], %[wa1] offset:9216\n" \
    "\tv_mfma_f32_16x16x128_f8f6f4 %[c2], v[120:127], v[144:151], %[c2] cbsz:1 blgp:1\n" \
    "\tds_read_b32 %[vd], %[dp]\n" \
    "\tds_read_b128 v[80:83], %[ao0] offset:20480\n" \
    "\tds_read_b128 v[84:87], %[ao1] offset:20480\n" \
    "\tv_mfma_f32_16x16x128_f8f6f4 %[c3], v[128:135], v[144:151], %[c3] cbsz:1 blgp:1\n" \
    "\tds_read_b128 v[88:91], %[ao0] offset:22528\n" \
    "\tds_read_b128 v[92:95], %[ao1] offset:22528\n" \
    "\tv_mfma_f32_16x16x128_f8f6f4 %[c4], v[120:127], v[152:159], %[c4] cbsz:1 blgp:1\n" \
    "\tds_read_b128 v[96:99], %[ao0] offset:24576\n" \
    "\tds_read_b128 v[100:103], %[ao1] offset:24576\n" \
    "\tv_mfma_f32_16x16x128_f8f6f4 %[c5], v[128:135], v[152:159], %[c5] cbsz:1 blgp:1\n" \
    "\tds_read_b128 v[96:99], %[ao0] offset:24576\n" \
    "\tds_read_b128 v[100:103], %[ao1] offset:24576\n" \
    "\tv_mfma_f32_16x16x128_f8f6f4 %[c6], v[120:127], v[160:167], %[c6] cbsz:1 blgp:1\n" \
    "\tds_read_b128 v[104:107], %[ao0] offset:26624\n" \
    "\tds_read_b128 v[108:111], %[ao1] offset:26624\n" \
    "\tv_mfma_f32_16x16x128_f8f6f4 %[c7], v[128:135], v[160:167], %[c7] cbsz:1 blgp:1\n" \
    "\ts_waitcnt lgkmcnt(10)\n" \
    "\tv_and_b32 %[t], 7, %[vd]\n" \
    "\tv_lshl_add_u32 %[wa0], %[t], 14, %[bo0]\n" \
    "\tv_lshl_add_u32 %[wa1], %[t], 14, %[bo1]\n" \
    "\tv_mfma_f32_16x16x128_f8f6f4 %[c8], v[120:127], v[168:175], %[c8] cbsz:1 blgp:1\n" \
    "\tv_cmp_eq_u32 vcc, 0x2007, %[vd]\n" \
    "\tv_add_u32 %[dp], 4, %[dp]\n" \
    "\tv_bfi_b32 %[dp], 63, %[dp], %[dbase]\n" \
    "\tv_mfma_f32_16x16x128_f8f6f4 %[c9], v[128:135], v[168:175], %[c9] cbsz:1 blgp:1\n" \
    "\ts_cbranch_vccnz 12f\n" \
    "22:\n" \
    "\ts_waitcnt lgkmcnt(0)\n" \
    "\ts_barrier\n" \
    "\tv_mfma_f32_16x16x128_f8f6f4 %[c0], v[64:71], v[80:87], %[c0] cbsz:1 blgp:1\n" \
    "\tds_read_b64_tr_b8 v[120:121], %[wa0] offset:0\n" \
    "\tds_read_b64_tr_b8 v[122:123], %[wa0] offset:1024\n" \
    "\tds_read_b64_tr_b8 v[124:125], %[wa0] offset:8192\n" \
    "\tds_read_b64_tr_b8 v[126:127], %[wa0] offset:9216\n" \
    "\tv_mfma_f32_16x16x128_f8f6f4 %[c1], v[72:79], v[80:87], %[c1] cbsz:1 blgp:1\n" \
    "\tds_read_b64_tr_b8 v[128:129], %[wa1] offset:0\n" \
    "\tds_read_b64_tr_b8 v[130:131], %[wa1] offset:1024\n" \
    "\tds_read_b64_tr_b8 v[132:133], %[wa1] offset:8192\n" \
    "\tds_read_b64_tr_b8 v[134:135], %[wa1] offset:9216\n" \
    "\tv_mfma_f32_16x16x128_f8f6f4 %[c2], v[64:71], v[88:95], %[c2] cbsz:1 blgp:1\n" \
    "\tds_read_b32 %[vd], %[dp]\n" \
    "\tds_read_b128 v[136:139], %[ao0] offset:30720\n" \
    "\tds_read_b128 v[140:143], %[ao1] offset:30720\n" \
    "\tv_mfma_f32_16x16x128_f8f6f4 %[c3], v[72:79], v[88:95], %[c3] cbsz:1 blgp:1\n" \
    "\tds_read_b128 v[144:147], %[ao0] offset:32768\n" \
    "\tds_read_b128 v[148:151], %[ao1] offset:32768\n" \
    "\tv_mfma_f32_16x16x128_f8f6f4 %[c4], v[64:71], v[96:103], %[c4] cbsz:1 blgp:1\n" \
    "\tds_read_b128 v[152:155], %[ao0] offset:34816\n" \
    "\tds_read_b128 v[156:159], %[ao1] offset:34816\n" \
    "\tv_mfma_f32_16x16x128_f8f6f4 %[c5], v[72:79], v[96:103], %[c5] cbsz:1 blgp:1\n" \
    "\tds_read_b128 v[152:155], %[ao0] offset:34816\n" \
    "\tds_read_b128 v[156:159], %[ao1] offset:34816\n" \
    "\tv_mfma_f32_16x16x128_f8f6f4 %[c6], v[64:71], v[104:111], %[c6] cbsz:1 blgp:1\n" \
    "\tds_read_b128 v[160:163], %[ao0] offset:36864\n" \
    "\tds_read_b128 v[164:167], %[ao1] offset:36864\n" \
    "\tv_mfma_f32_16x16x128_f8f6f4 %[c7], v[72:79], v[104:111], %[c7] cbsz:1 blgp:1\n" \
    "\ts_waitcnt lgkmcnt(10)\n" \
    "\tv_and_b32 %[t], 7, %[vd]\n" \
    "\tv_lshl_add_u32 %[wa0], %[t], 14, %[bo0]\n" \
    "\tv_lshl_add_u32 %[wa1], %[t], 14, %[bo1]\n" \
    "\tv_mfma_f32_16x16x128_f8f6f4 %[c8], v[64:71], v[112:119], %[c8] cbsz:1 blgp:1\n" \
    "\tv_cmp_eq_u32 vcc, 0x2007, %[vd]\n" \
    "\tv_add_u32 %[dp], 4, %[dp]\n" \
    "\tv_bfi_b32 %[dp], 63, %[dp], %[dbase]\n" \
    "\ts_sub_u32 %[n], %[n], 1\n" \
    "\ts_cmp_eq_u32 %[n], 0\n" \
    "\tv_mfma_f32_16x16x128_f8f6f4 %[c9], v[72:79], v[112:119], %[c9] cbsz:1 blgp:1\n" \
    "\ts_cbranch_scc1 4f\n" \
    "\ts_cbranch_vccnz 13f\n" \
    "23:\n" \
    "\ts_waitcnt lgkmcnt(0)\n" \
    "\ts_barrier\n" \
    "\tv_mfma_f32_16x16x128_f8f6f4 %[c0], v[120:127], v[136:143], %[c0] cbsz:1 blgp:1\n" \
    "\tds_read_b64_tr_b8 v[64:65], %[wa0] offset:0\n" \
    "\tds_read_b64_tr_b8 v[66:67], %[wa0] offset:1024\n" \
    "\tds_read_b64_tr_b8 v[68:69], %[wa0] offset:8192\n" \
    "\tds_read_b64_tr_b8 v[70:71], %[wa0] offset:9216\n" \
    "\tv_mfma_f32_16x16x128_f8f6f4 %[c1], v[128:135], v[136:143], %[c1] cbsz:1 blgp:1\n" \
    "\tds_read_b64_tr_b8 v[72:73], %[wa1] offset:0\n" \
    "\tds_read_b64_tr_b8 v[74:75], %[wa1] offset:1024\n" \
    "\tds_read_b64_tr_b8 v[76:77], %[wa1] offset:8192\n" \
    "\tds_read_b64_tr_b8 v[78:79], %[wa1] offset:9216\n" \
    "\tv_mfma_f32_16x16x128_f8f6f4 %[c2], v[120:127], v[144:151], %[c2] cbsz:1 blgp:1\n" \
    "\tds_read_b32 %[vd], %[dp]\n" \
    "\tds_read_b128 v[80:83], %[ao0] offset:0\n" \
    "\tds_read_b128 v[84:87], %[ao1] offset:0\n" \
    "\tv_mfma_f32_16x16x128_f8f6f4 %[c3], v[128:135], v[144:151], %[c3] cbsz:1 blgp:1\n" \
    "\tds_read_b128 v[88:91], %[ao0] offset:2048\n" \
    "\tds_read_b128 v[92:95], %[ao1] offset:2048\n" \
    "\tv_mfma_f32_16x16x128_f8f6f4 %[c4], v[120:127], v[152:159], %[c4] cbsz:1 blgp:1\n" \
    "\tds_read_b128 v[96:99], %[ao0] offset:4096\n" \
    "\tds_read_b128 v[100:103], %[ao1] offset:4096\n" \
    "\tv_mfma_f32_16x16x128_f8f6f4 %[c5], v[128:135], v[152:159], %[c5] cbsz:1 blgp:1\n" \
    "\tds_read_b128 v[96:99], %[ao0] offset:4096\n" \
    "\tds_read_b128 v[100:103], %[ao1] offset:4096\n" \
    "\tv_mfma_f32_16x16x128_f8f6f4 %[c6], v[120:127], v[160:167], %[c6] cbsz:1 blgp:1\n" \
    "\tds_read_b128 v[104:107], %[ao0] offset:6144\n" \
    "\tds_read_b128 v[108:111], %[ao1] offset:6144\n" \
    "\tv_mfma_f32_16x16x128_f8f6f4 %[c7], v[128:135], v[160:167], %[c7] cbsz:1 blgp:1\n" \
    "\ts_waitcnt lgkmcnt(10)\n" \
    "\tv_and_b32 %[t], 7, %[vd]\n" \
    "\tv_lshl_add_u32 %[wa0], %[t], 14, %[bo0]\n" \
    "\tv_lshl_add_u32 %[wa1], %[t], 14, %[bo1]\n" \
    "\tv_mfma_f32_16x16x128_f8f6f4 %[c8], v[120:127], v[168:175], %[c8] cbsz:1 blgp:1\n" \
    "\tv_cmp_eq_u32 vcc, 0x2007, %[vd]\n" \
    "\tv_add_u32 %[dp], 4, %[dp]\n" \
    "\tv_bfi_b32 %[dp], 63, %[dp], %[dbase]\n" \
    "\tv_mfma_f32_16x16x128_f8f6f4 %[c9], v[128:135], v[168:175], %[c9] cbsz:1 blgp:1\n" \
    "\ts_branch 3b\n" \
    "10:\n" \
    "\ts_waitcnt lgkmcnt(0)\n" \
    "\ts_barrier\n" \
    "\tds_read_b32 %[vd], %[dp]\n" \
    "\tv_add_u32 %[dp], 4, %[dp]\n" \
    "\tv_bfi_b32 %[dp], 63, %[dp], %[dbase]\n" \
    "\ts_waitcnt lgkmcnt(0)\n" \
    "\tv_cmp_eq_u32 vcc, 0x2007, %[vd]\n" \
    "\tv_and_b32 %[t], 7, %[vd]\n" \
    "\tv_lshl_add_u32 %[wa0], %[t], 14, %[bo0]\n" \
    "\tv_lshl_add_u32 %[wa1], %[t], 14, %[bo1]\n" \
    "\ts_cbranch_vccnz 10b\n" \
    "\ts_branch 20b\n" \
    "11:\n" \
    "\ts_waitcnt lgkmcnt(0)\n" \
    "\ts_barrier\n" \
    "\tds_read_b32 %[vd], %[dp]\n" \
    "\tv_add_u32 %[dp], 4, %[dp]\n" \
    "\tv_bfi_b32 %[dp], 63, %[dp], %[dbase]\n" \
    "\ts_waitcnt lgkmcnt(0)\n" \
    "\tv_cmp_eq_u32 vcc, 0x2007, %[vd]\n" \
    "\tv_and_b32 %[t], 7, %[vd]\n" \
    "\tv_lshl_add_u32 %[wa0], %[t], 14, %[bo0]\n" \
    "\tv_lshl_add_u32 %[wa1], %[t], 14, %[bo1]\n" \
    "\ts_cbranch_vccnz 11b\n" \
    "\ts_branch 21b\n" \
    "12:\n" \
    "\ts_waitcnt lgkmcnt(0)\n" \
    "\ts_barrier\n" \
    "\tds_read_b32 %[vd], %[dp]\n" \
    "\tv_add_u32 %[dp], 4, %[dp]\n" \
    "\tv_bfi_b32 %[dp], 63, %[dp], %[dbase]\n" \
    "\ts_waitcnt lgkmcnt(0)\n" \
    "\tv_cmp_eq_u32 vcc, 0x2007, %[vd]\n" \
    "\tv_and_b32 %[t], 7, %[vd]\n" \
    "\tv_lshl_add_u32 %[wa0], %[t], 14, %[bo0]\n" \
    "\tv_lshl_add_u32 %[wa1], %[t], 14, %[bo1]\n" \
    "\ts_cbranch_vccnz 12b\n" \
    "\ts_branch 22b\n" \
    "13:\n" \
    "\ts_waitcnt lgkmcnt(0)\n" \
    "\ts_barrier\n" \
    "\tds_read_b32 %[vd], %[dp]\n" \
    "\tv_add_u32 %[dp], 4, %[dp]\n" \
    "\tv_bfi_b32 %[dp], 63, %[dp], %[dbase]\n" \
    "\ts_waitcnt lgkmcnt(0)\n" \
    "\tv_cmp_eq_u32 vcc, 0x2007, %[vd]\n" \
    "\tv_and_b32 %[t], 7, %[vd]\n" \
    "\tv_lshl_add_u32 %[wa0], %[t], 14, %[bo0]\n" \
    "\tv_lshl_add_u32 %[wa1], %[t], 14, %[bo1]\n" \
    "\ts_cbranch_vccnz 13b\n" \
    "\ts_branch 23b\n" \
    "4:\n" \
    "\tds_read_b32 %[vd], %[stat]\n" \
    "\ts_waitcnt lgkmcnt(1)\n" \
    "\tv_mfma_f32_16x16x128_f8f6f4 %[c0], v[120:127], v[136:143], %[c0] cbsz:1 blgp:1\n" \
    "\tv_mfma_f32_16x16x128_f8f6f4 %[c1], v[128:135], v[136:143], %[c1] cbsz:1 blgp:1\n" \
    "\tv_mfma_f32_16x16x128_f8f6f4 %[c2], v[120:127], v[144:151], %[c2] cbsz:1 blgp:1\n" \
    "\tv_mfma_f32_16x16x128_f8f6f4 %[c3], v[128:135], v[144:151], %[c3] cbsz:1 blgp:1\n" \
    "\tv_mfma_f32_16x16x128_f8f6f4 %[c4], v[120:127], v[152:159], %[c4] cbsz:1 blgp:1\n" \
    "\tv_mfma_f32_16x16x128_f8f6f4 %[c5], v[128:135], v[152:159], %[c5] cbsz:1 blgp:1\n" \
    "\tv_mfma_f32_16x16x128_f8f6f4 %[c6], v[120:127], v[160:167], %[c6] cbsz:1 blgp:1\n" \
    "\tv_mfma_f32_16x16x128_f8f6f4 %[c7], v[128:135], v[160:167], %[c7] cbsz:1 blgp:1\n" \
    "\tv_mfma_f32_16x16x128_f8f6f4 %[c8], v[120:127], v[168:175], %[c8] cbsz:1 blgp:1\n" \
    "\tv_mfma_f32_16x16x128_f8f6f4 %[c9], v[128:135], v[168:175], %[c9] cbsz:1 blgp:1\n" \
    "\ts_waitcnt lgkmcnt(0)\n" \
    "\ts_nop 7\n" \
    "\ts_nop 3\n"

#define CONS_CLOB "memory", "scc", "vcc", "s84", "s85", "v64", "v65", "v66", "v67", "v68", "v69", "v70", "v71", "v72", "v73", "v74", "v75", "v76", "v77", "v78", "v79", "v80", "v81", "v82", "v83", "v84", "v85", "v86", "v87", "v88", "v89", "v90", "v91", "v92", "v93", "v94", "v95", "v96", "v97", "v98", "v99", "v100", "v101", "v102", "v103", "v104", "v105", "v106", "v107", "v108", "v109", "v110", "v111", "v112", "v113", "v114", "v115", "v116", "v117", "v118", "v119", "v120", "v121", "v122", "v123", "v124", "v125", "v126", "v127", "v128", "v129", "v130", "v131", "v132", "v133", "v134", "v135", "v136", "v137", "v138", "v139", "v140", "v141", "v142", "v143", "v144", "v145", "v146", "v147", "v148", "v149", "v150", "v151", "v152", "v153", "v154", "v155", "v156", "v157", "v158", "v159", "v160", "v161", "v162", "v163", "v164", "v165", "v166", "v167", "v168", "v169", "v170", "v171", "v172", "v173", "v174", "v175"
template <int RT>
__device__ __forceinline__ void consumer(const Args& p, u32 lds0, int wave, int lane, int mt, int n0 DBGARGS) {
  const u32 ctl = lds0 + LDS_CTL;
  if (wave == 0) *(volatile LAS u32*)(ctl + lane * 4) = (lane >= 48 && lane < 52) ? (u32)(lane - 45) * 0x101u : 0u;
  const float scale = *p.sx * *p.sw;
  const int g = lane >> 4, t = lane & 15;
  u32 aoff[2], boff[2];
  #pragma unroll
  for (int i = 0; i < 2; ++i) {
    aoff[i] = lds0 + LDS_A + 128 * t + 16 * ((g + 4 * i) ^ (t >> 1));
    boff[i] = lds0 + LDS_W + 128 * (16 * g + (t >> 1)) + 16 * ((2 * wave + i) ^ ((t >> 2) | ((g & 1) << 2))) + 8 * (t & 1);
  }
  f32x4 acc[10] = {};
  u32 dp = ctl + C_DESC + 4, wa0 = boff[0] + 3 * 16384u, wa1 = boff[1] + 3 * 16384u, n = 8, vd, tmp;
  u32 stv = 0;
  float* o = p.out + (size_t)(row0_of(mt) + t) * GN + n0 + wave * 32 + 4 * g;
#define CONS_IN [ao0] "v"(aoff[0]), [ao1] "v"(aoff[1]), [bo0] "v"(boff[0]), [bo1] "v"(boff[1]), [stat] "v"(ctl + C_STAT), [dbase] "s"(ctl + C_DESC)
#define CONS_OUT [c0] "+&v"(acc[0]), [c1] "+&v"(acc[1]), [c2] "+&v"(acc[2]), [c3] "+&v"(acc[3]), [c4] "+&v"(acc[4]), [c5] "+&v"(acc[5]), \
                 [c6] "+&v"(acc[6]), [c7] "+&v"(acc[7]), [dp] "+&v"(dp), [wa0] "+&v"(wa0), [wa1] "+&v"(wa1), [n] "+&s"(n), [vd] "=&v"(vd), [t] "=&v"(tmp), [stv] "+&v"(stv)
  if constexpr (RT == 5) asm volatile(CONS_ASM5 : CONS_OUT, [c8] "+&v"(acc[8]), [c9] "+&v"(acc[9]) : CONS_IN : CONS_CLOB);
  else asm volatile(CONS_ASM4 : CONS_OUT : CONS_IN : CONS_CLOB);
  const float sc = (vd == 0) ? scale : __builtin_nanf("");
  #pragma unroll
  for (int k = 0; k < 2 * RT; ++k)
    *(f32x4*)(o + (size_t)(k >> 1) * 16 * GN + (k & 1) * 16) = acc[k] * sc;
}

template <int RT>
__device__ __forceinline__ void aloader(const Args& p, u32 lds0, int lane, int b, int mt, int rot DBGARGS) {
  constexpr int NI = 2 * RT;
  const int rank = p.rank;
  char* const myws = p.myws;
  const u32 ctl = lds0 + LDS_CTL;
  u32* cntp = (u32*)(myws + OFF_CNT + (size_t)b * 128) + 6;
  u32 c;
  asm volatile("global_load_dword %0, %1, off sc0 sc1" : "=v"(c) : "v"(cntp) : "memory");
  const char* dtab = myws + OFF_DTAB + (size_t)b * 128;
  asm volatile("global_store_dword %0, %1, %2" :: "v"((u32)(lane & 31) * 4), "v"((lane & 31) == 7 ? D_NULL : (u32)(lane & 7) | (u32)((lane & 31) + 8) << 8), "s"(dtab) : "memory");
  u32 off[10];
  #pragma unroll
  for (int q = 0; q < 10; ++q) off[q] = (u32)(8 * q + (lane >> 3)) * KS + 16 * ((lane & 7) ^ ((4 * q + (lane >> 4)) & 7)) - (q & 3) * 1024;
  const unsigned char* xl = p.x + (size_t)(rank * 512 + row0_of(mt)) * KS;
  const u32 d0 = lds0 + LDS_A;
  #pragma unroll
  for (int s = 0; s < 4; ++s) dma<NI, false>(off, xl + s * 128, d0 + s * PIECE);
  const int lq = lane & 7, lj = (lane >> 3) & 3, lsrc = src_of(lq < 7 ? lq : 0, rank, rot);
  const u32 tab = (ABL_NOCOMM ? (u32)((lsrc * 4 + lj) * 7 + mt) * 8192u : (u32)(lsrc * 4 + lj) * PIECE);
  #pragma unroll
  for (int q = 0; q < 10; ++q) off[q] = (u32)((q >> 2) * 4096 + lane * 16);
  asm volatile("s_waitcnt vmcnt(%1)" : "+v"(c) : "i"(3 * NI) : "memory");
  const u32 call = rfl(c) + 1;
  const char* slot = ABL_NOCOMM ? (const char*)p.x : myws + OFF_SLOT + (size_t)(call & 1) * SLOT_BYTES + (size_t)mt * 32 * PIECE;
  u32* tmo = (u32*)(myws + OFF_TMO);
  const u32 snap = ctl + C_FLAGS + lq * 16 + lj * 4;
  const u32 t0 = (u32)__builtin_amdgcn_s_memrealtime();
  u32 mask = ABL_NOCOMM ? 0x7F7F7F7Fu : 0u;
  u32 next = 0x7Fu;
  u32 l1 = 0x40u;
  u32 dst = d0, dpa = ctl + C_DESC + 16, bad = 0;
  u32 slo = (u32)(u64)slot, shi = (u32)((u64)slot >> 32);
  u32 c_, x_, o_, nl_, nn_, vD, vF = 0;
  bar();
  waitvm<2 * NI>();
#define GL(V, OFF) "global_load_lds_dwordx4 " V ", s[80:81]" OFF " sc0 sc1\n\t"
#define PICK \
    "s_andn2_b32 %[c], %[mask], %[l1]\n\t" \
    "s_and_b32 %[c], %[c], %[next]\n\t" \
    "s_ff1_i32_b32 %[x], %[c]\n\t" \
    "v_readlane_b32 %[o], %[tab], %[x]\n\t"
#define PICK2 \
    "s_lshl_b32 %[nl], 0x100, %[x]\n\t" \
    "s_lshl_b32 %[nn], 0x101, %[x]\n\t" \
    "v_lshlrev_b32 %[vD], 2, %[x]\n\t" \
    "s_add_u32 s82, %[slo], %[o]\n\t" \
    "s_addc_u32 s83, %[shi], 0\n\t"
#define NEXTDPA \
    "s_add_u32 %[dpa], %[dpa], 4\n\t" \
    "s_or_b32 %[dpa], %[dpa], 0xc0\n\t" \
    "s_bitset0_b32 %[dpa], 8\n\t"
#define NEXTDST \
    "s_add_u32 %[dst], %[dst], %[piece]\n\t" \
    "s_cmp_eq_u32 %[dst], %[dend]\n\t" \
    "s_cselect_b32 %[dst], %[d0], %[dst]\n\t" \
    "s_cmp_lg_u32 %[c], 0\n\t"
#define ARM \
    "s_mov_b64 s[80:81], s[82:83]\n\t" \
    "s_mov_b32 m0, %[dpa]\n\t" \
    "s_mov_b64 exec, 1\n\t"
#define ALOOP(TAILDMA) asm volatile( \
    "s_mov_b64 vcc, 0\n\t" \
    "s_branch 3f\n" \
    "1:\n\t" \
    "s_waitcnt vmcnt(%[n2])\n\t" \
    "s_barrier\n" \
    "2:\n\t" \
    "global_load_lds_dword %[vD], %[dtab] sc1\n\t" \
    "s_mov_b32 m0, %[dst]\n\t" \
    "s_mov_b64 exec, -1\n\t" \
    GL("%[o0]", "") \
    "s_waitcnt lgkmcnt(0)\n\t" \
    "v_cmp_le_u32 s[84:85], %[call], %[vF]\n\t" \
    "s_mov_b32 %[l1], %[nl]\n\t" \
    "s_xor_b32 %[next], %[next], %[nn]\n\t" \
    "s_or_b32 %[mask], %[mask], s84\n\t" \
    GL("%[o0]", " offset:1024") \
    "ds_read_b32 %[vF], %[snap]\n\t" \
    NEXTDPA \
    GL("%[o0]", " offset:2048") \
    PICK \
    GL("%[o0]", " offset:3072") \
    "s_add_u32 m0, m0, 0x1000\n\t" \
    PICK2 \
    GL("%[o1]", "") \
    GL("%[o1]", " offset:1024") \
    GL("%[o1]", " offset:2048") \
    TAILDMA \
    ARM \
    "s_cbranch_vccnz 5f\n" \
    "6:\n\t" \
    "s_cbranch_scc1 1b\n\t" \
    "s_mov_b64 exec, -1\n\t" \
    "s_waitcnt vmcnt(%[n2])\n" \
    "3:\n\t"                        \
    "s_barrier\n\t" \
    "s_cmp_eq_u32 %[next], 0\n\t" \
    "s_cbranch_scc1 9f\n\t" \
    "ds_read_b32 %[vF], %[snap]\n\t" \
    "s_memrealtime s[86:87]\n\t" \
    "s_waitcnt lgkmcnt(0)\n\t" \
    "v_cmp_le_u32 s[84:85], %[call], %[vF]\n\t" \
    "s_or_b32 %[mask], %[mask], s84\n" \
    "4:\n\t" \
    PICK PICK2 ARM \
    "s_cmp_lg_u32 %[c], 0\n\t" \
    "s_cbranch_scc1 2b\n\t" \
    "s_sub_u32 s86, s86, %[t0]\n\t" \
    "s_cmp_gt_u32 s86, %[dl]\n\t" \
    "s_cbranch_scc1 8f\n\t" \
    "s_bitcmp1_b32 %[mask], 7\n\t" \
    "s_cbranch_scc1 8f\n" \
    "7:\n\t" \
    "v_mov_b32 %[vD], 28\n\t" \
    "s_mov_b32 %[l1], 0\n\t" \
    "global_load_lds_dword %[vD], %[dtab] sc1\n\t" \
    "s_mov_b64 exec, -1\n\t" \
    NEXTDPA \
    "s_waitcnt vmcnt(%[n1])\n\t" \
    "s_cbranch_vccz 10f\n\t" \
    "s_waitcnt vmcnt(1)\n" \
    "10:\n\t" \
    "s_mov_b64 vcc, -1\n\t" \
    "s_branch 3b\n" \
    "5:\n\t"                        \
    "s_waitcnt vmcnt(%[n1])\n\t" \
    "s_mov_b64 vcc, 0\n\t" \
    "s_branch 6b\n" \
    "8:\n\t"                        \
    "s_cmp_lg_u32 %[bad], 0\n\t" \
    "s_cbranch_scc1 7b\n\t" \
    "s_mov_b32 %[bad], 1\n\t" \
    "v_mov_b32 %[vD], 0\n\t" \
    "v_mov_b32 %[vF], %[call]\n\t" \
    "global_store_dword %[vD], %[vF], %[tmo] sc1\n\t" \
    "s_mov_b64 exec, -1\n\t" \
    "s_mov_b32 %[mask], -1\n\t" \
    "s_mov_b32 %[slo], %[xlo]\n\t" \
    "s_mov_b32 %[shi], %[xhi]\n\t" \
    "s_waitcnt vmcnt(0)\n\t" \
    "s_branch 4b\n" \
    "9:\n\t" \
    "s_waitcnt lgkmcnt(0)" \
    : [mask] "+&s"(mask), [next] "+&s"(next), [l1] "+&s"(l1), [dst] "+&s"(dst), [dpa] "+&s"(dpa), [bad] "+&s"(bad), [slo] "+&s"(slo), [shi] "+&s"(shi), \
      [c] "=&s"(c_), [x] "=&s"(x_), [o] "=&s"(o_), [nl] "=&s"(nl_), [nn] "=&s"(nn_), [vD] "=&v"(vD), [vF] "+&v"(vF) \
    : [o0] "v"(off[0]), [o1] "v"(off[4]), [o2] "v"(off[8]), [tab] "v"(tab), [snap] "v"(snap), [call] "s"(call), [t0] "s"(t0), \
      [d0] "s"(d0), [dend] "s"(d0 + NAS * PIECE), [dtab] "s"(dtab), [tmo] "s"(tmo), [xlo] "s"((u32)(u64)p.x), [xhi] "s"((u32)((u64)p.x >> 32)), \
      [piece] "i"(PIECE), [n2] "i"(2 * NI + 1), [n1] "i"(NI + 1), [dl] "i"((u32)DEADLINE) \
    : "memory", "scc", "vcc", "s80", "s81", "s82", "s83", "s84", "s85", "s86", "s87")
  if constexpr (NI == 10)
    ALOOP(GL("%[o1]", " offset:3072") "s_add_u32 m0, m0, 0x1000\n\t" NEXTDST GL("%[o2]", "") GL("%[o2]", " offset:1024"));
  else
    ALOOP(NEXTDST GL("%[o1]", " offset:3072"));
#undef GL
  u32 tv;
  asm volatile("global_load_dword %0, %1, %2 sc1\n\ts_waitcnt vmcnt(%3)\n\ts_barrier\n\ts_waitcnt vmcnt(0)"
               : "=&v"(tv) : "v"(0u), "s"(tmo), "i"(NI + 1) : "memory");
  if (bad || rfl(tv) == call) { *(volatile LAS u32*)(ctl + C_STAT) = 1u; asm volatile("s_waitcnt lgkmcnt(0)" ::: "memory"); }
  bar();
  if (lane == 0) __hip_atomic_store(cntp, call, __ATOMIC_RELAXED, __HIP_MEMORY_SCOPE_SYSTEM);
  STAMP(8);
}
}

__global__ __launch_bounds__(512, 2) void a2a_gemm_kernel(Args p) {
  extern __shared__ __attribute__((aligned(16))) char smem[];
  const u32 lds0 = (u32)(size_t)(LAS char*)smem;
  const int b = blockIdx.x, lane = threadIdx.x & 63;
  const int wave = rfl(threadIdx.x >> 6);
  const int rank = p.rank;
  char* const myws = p.myws;

  if (__builtin_expect(b < 32, 1)) {
    const u32 sub = (u32)b >> 3, link = ((u32)b + 2 * sub) & 7u;
    const unsigned char* xs = p.peer[link].xs;
    char* dws = p.peer[link].ws;
    const u64 t0 = __builtin_amdgcn_s_memrealtime();
    const u32 ps = (u32)wave * 4 + sub, pj = (ps * 37) >> 8, pk = ps - pj * 7 + (PROT ? link : 0u), pmt = pk >= 7 ? pk - 7 : pk;
    const u32 r0 = 64 * pmt + 16 * min(pmt, 4u);
    const bool big = pmt < 4;
    const u32 o0 = (r0 + (u32)(lane >> 3)) * KS + pj * 128 + 16 * ((lane & 7) ^ (lane >> 4));
    const u32 o1 = (o0 ^ 64u) + 8 * KS;
    const u32 ol = big ? 64 * KS : 48 * KS;
    asm volatile("" :: "s"(xs), "s"(dws), "s"(myws), "s"(rank));
    if (link >= 7u || wave >= 7 || ABL_NOCOMM) return;
    const u32 cnto = (u32)OFF_CNT + (u32)b * 128 + (u32)wave * 4;
    u32* cntp = (u32*)(myws + cnto);
    u32x4 v[10];
    u32 c;
    if (wave >= 2) { for (int z = 1; z < wave; ++z) __builtin_amdgcn_s_sleep(LDSLEEP); }
    asm volatile(
      "global_load_dwordx4 %0, %13, %23\n\t"
      "global_load_dwordx4 %1, %14, %23\n\t"
      "global_load_dwordx4 %2, %15, %23\n\t"
      "global_load_dwordx4 %3, %16, %23\n\t"
      "global_load_dwordx4 %4, %17, %23\n\t"
      "global_load_dwordx4 %5, %18, %23\n\t"
      "global_load_dwordx4 %6, %19, %23\n\t"
      "global_load_dwordx4 %7, %20, %23\n\t"
      "global_load_dwordx4 %8, %21, %23\n\t"
      "global_load_dwordx4 %9, %22, %23\n\t"
      "global_load_dword %10, %11, %12 sc0 sc1"
      : "=&v"(v[0]), "=&v"(v[1]), "=&v"(v[2]), "=&v"(v[3]), "=&v"(v[4]), "=&v"(v[5]), "=&v"(v[6]), "=&v"(v[7]), "=&v"(v[8]), "=&v"(v[9]), "=&v"(c)
      : "v"(cnto), "s"(myws), "v"(o0), "v"(o1), "v"(o0 + 16 * KS), "v"(o1 + 16 * KS), "v"(o0 + 32 * KS), "v"(o1 + 32 * KS),
        "v"(o0 + 48 * KS), "v"(o1 + 48 * KS), "v"(o0 + ol), "v"(o1 + ol), "s"(xs) : "memory");
    const int tl = (int)(PROT ? ps * (DELTA * 8 / 7) : (pj * 512 + r0) * DELTA / 64) - LEAD;
    u64 tgo = t0 + (u64)((u32)(TSTART + (tl > 0 ? tl : 0)) / 10);
    char* d = dws + OFF_SLOT + (size_t)((pmt * 8 + rank) * 4 + pj) * PIECE + lane * 16;
    asm volatile("s_waitcnt vmcnt(0)"
      : "+v"(v[0]), "+v"(v[1]), "+v"(v[2]), "+v"(v[3]), "+v"(v[4]), "+v"(v[5]), "+v"(v[6]), "+v"(v[7]), "+v"(v[8]), "+v"(v[9]), "+v"(c), "+s"(tgo), "+v"(d) :: "memory");
    const u32 call = rfl(c) + 1;
    d += (size_t)(call & 1) * SLOT_BYTES;
    while (__builtin_amdgcn_s_memrealtime() < tgo) __builtin_amdgcn_s_sleep(1);
    #pragma unroll
    for (int i = 0; i < 8; ++i)
      asm volatile("global_store_dwordx4 %0, %1, off sc0 sc1\n\ts_nop 1" :: "v"(d + i * 1024), "v"(v[i]) : "memory");
    if (big) {
      #pragma unroll
      for (int i = 8; i < 10; ++i)
        asm volatile("global_store_dwordx4 %0, %1, off sc0 sc1\n\ts_nop 1" :: "v"(d + i * 1024), "v"(v[i]) : "memory");
    }
    waitvm<0>();
    if (lane == 0) {
      __hip_atomic_store((u32*)(dws + OFF_FLAG + (size_t)(pmt * 8 + rank) * 128) + pj, call, __ATOMIC_RELAXED, __HIP_MEMORY_SCOPE_SYSTEM);
      __hip_atomic_store(cntp, call, __ATOMIC_RELAXED, __HIP_MEMORY_SCOPE_SYSTEM);
    }
    return;
  }

  const int gi = (b >> 3) - 4, ntl = gi / 7, mt = gi - ntl * 7;
  const int n0 = (CMAP ? (b & 7) + 8 * ntl : (b & 7) * 4 + ntl) * 128;
  const int rot = PROT ? (6 - mt + ROTN * ntl) % 7 : (ROTM * mt + ROTN * ntl) % 7;
  if (GDELAY && wave >= 4) __builtin_amdgcn_s_sleep(GDELAY);
  if (wave < 4) {
    if (mt < 4) consumer<5>(p, lds0, wave, lane, mt, n0 DBGPASS); else consumer<4>(p, lds0, wave, lane, mt, n0 DBGPASS);
  } else if (wave < 6) {
    const int h = wave - 4;
    u32 off[10];
    #pragma unroll
    for (int q = 0; q < 8; ++q) off[q] = (u32)(8 * q + (lane >> 3)) * GN + 16 * ((lane & 7) ^ ((lane >> 4) | ((q & 2) << 1))) - (q & 3) * 1024;
    off[8] = off[9] = 0;
    const unsigned char* wb = p.w + (size_t)(64 * h) * GN + n0;
    const u32 d0 = lds0 + LDS_W + h * 8192u;
    #pragma unroll
    for (int s = 0; s < 4; ++s) dma<8, false>(off, wb + (size_t)(rank * KS + s * 128) * GN, d0 + (3 + s) * 16384u);
    const u32 tab = (u32)(src_of((lane & 7) < 7 ? (lane & 7) : 0, rank, rot) * KS + ((lane >> 3) & 3) * 128) * GN;
    #pragma unroll
    for (int s = 0; s < 3; ++s) dma<8, false>(off, wb + (u32)__builtin_amdgcn_readlane(tab, s), d0 + s * 16384u);
    waitvm<48>(); bar();
    waitvm<40>(); bar();
    dma<8, false>(off, wb + (u32)__builtin_amdgcn_readlane(tab, 3), d0 + 3 * 16384u);
    waitvm<40>(); bar();
    dma<8, false>(off, wb + (u32)__builtin_amdgcn_readlane(tab, 4), d0 + 4 * 16384u);
    waitvm<40>(); bar();
    u32 vdp = lds0 + LDS_CTL + C_DESC + 16, vd;
    u32 d1 = 0x606u;
    u32 left = 23, q_, m_, nm_, un_ = 0;
    const unsigned char* b0 = wb + (u32)__builtin_amdgcn_readlane(tab, 5);
#define GL(V, OFF) "global_load_lds_dwordx4 " V ", s[80:81]" OFF "\n\t"
#define RDDESC \
      "ds_read_b32 %[vd], %[vdp]\n\t" \
      "v_add_u32 %[vdp], 4, %[vdp]\n\t" \
      "v_bfi_b32 %[vdp], 63, %[vdp], %[dbase]\n\t"
#define UNSAFE   \
      "s_lshr_b32 %[un], s84, 16\n\t" \
      "s_or_b32 %[un], %[un], s84\n\t" \
      "s_lshr_b32 %[q], %[un], 8\n\t" \
      "s_or_b32 %[un], %[un], %[q]\n\t" \
      "s_or_b32 %[un], %[un], s85\n\t"
#define NEXTREF   \
      "s_lshr_b32 %[m], %[d1], 8\n\t" \
      "v_readlane_b32 %[q], %[tab], %[m]\n\t" \
      "s_and_b32 %[nm], %[d1], 7\n\t" \
      "s_lshl_b64 s[86:87], s[84:85], 8\n\t" \
      "s_bitset1_b32 s86, %[nm]\n\t" \
      "s_lshl_b32 %[nm], %[nm], 14\n\t"
#define NEXTREF2 \
      "s_add_u32 %[nm], %[nm], %[d0]\n\t" \
      "s_add_u32 s82, %[wlo], %[q]\n\t" \
      "s_addc_u32 s83, %[whi], 0\n\t"
    asm volatile(
      "s_mov_b64 s[80:81], %[b0]\n\t"
      "s_mov_b32 s86, 0x04081020\n\t"
      "s_mov_b32 s87, 0x02\n\t"
      "s_add_u32 m0, %[d0], 0x14000\n\t"
      "s_branch 2f\n"
      "1:\n\t"
      "s_waitcnt vmcnt(40)\n"
      "6:\n\t"
      "s_cmp_lt_u32 %[m], 32\n\t"
      "s_barrier\n\t"
      "s_cbranch_scc0 3f\n"
      "2:\n\t"
      GL("%[o0]", "")
      "s_mov_b64 s[84:85], s[86:87]\n\t"
      RDDESC
      GL("%[o1]", " offset:1024")
      "s_sub_u32 %[left], %[left], 1\n\t"
      UNSAFE
      GL("%[o2]", " offset:2048")
      NEXTREF
      GL("%[o3]", " offset:3072")
      "s_add_u32 m0, m0, 0x1000\n\t"
      NEXTREF2
      GL("%[o4]", "")
      GL("%[o5]", " offset:1024")
      GL("%[o6]", " offset:2048")
      GL("%[o7]", " offset:3072")
      "4:\n\t"
      "s_waitcnt lgkmcnt(0)\n\t"
      "v_readfirstlane_b32 %[d1], %[vd]\n\t"
      "s_mov_b32 m0, %[nm]\n\t"
      "s_mov_b64 s[80:81], s[82:83]\n\t"
      "s_bitcmp1_b32 %[un], %[d1]\n\t"
      "s_cbranch_scc0 1b\n\t"
      "s_and_b32 %[q], %[d1], 7\n\t"
      "s_lshl_b32 %[q], 0x01010101, %[q]\n\t"
      "s_waitcnt vmcnt(32)\n\t"
      "s_and_b32 %[q], %[q], s84\n\t"
      "s_cbranch_scc0 6b\n\t"
      "s_waitcnt vmcnt(24)\n\t"
      "s_and_b32 %[q], %[q], 0xffffff\n\t"
      "s_cbranch_scc0 6b\n\t"
      "s_waitcnt vmcnt(16)\n\t"
      "s_and_b32 %[q], %[q], 0xffff\n\t"
      "s_cbranch_scc0 6b\n\t"
      "s_waitcnt vmcnt(8)\n\t"
      "s_and_b32 %[q], %[q], 0xff\n\t"
      "s_cbranch_scc0 6b\n\t"
      "s_waitcnt vmcnt(0)\n\t"
      "s_cmp_eq_u32 %[left], 0\n\t"
      "s_cbranch_scc0 6b\n\t"
      "s_branch 7f\n"
      "3:\n\t"
      RDDESC NEXTREF NEXTREF2
      "s_branch 4b\n"
      "7:"
      : [vdp] "+&v"(vdp), [vd] "=&v"(vd), [left] "+&s"(left), [d1] "+&s"(d1), [q] "=&s"(q_), [m] "=&s"(m_), [nm] "=&s"(nm_), [un] "+&s"(un_)
      : [o0] "v"(off[0]), [o1] "v"(off[1]), [o2] "v"(off[2]), [o3] "v"(off[3]), [o4] "v"(off[4]), [o5] "v"(off[5]), [o6] "v"(off[6]), [o7] "v"(off[7]),
        [tab] "v"(tab), [wlo] "s"((u32)(u64)wb), [whi] "s"((u32)((u64)wb >> 32)), [b0] "s"(b0), [d0] "s"(d0), [dbase] "s"(lds0 + LDS_CTL + C_DESC)
      : "memory", "scc", "s80", "s81", "s82", "s83", "s84", "s85", "s86", "s87");
#undef GL
    STAMP(7);
  } else if (wave == 6) {
    if (mt < 4) aloader<5>(p, lds0, lane, b, mt, rot DBGPASS); else aloader<4>(p, lds0, lane, b, mt, rot DBGPASS);
  } else {
    if (ABL_NOCOMM) return;
    u32* cntp = (u32*)(myws + OFF_CNT + (size_t)b * 128) + 7;
    const u32 call = rfl(__hip_atomic_load(cntp, __ATOMIC_RELAXED, __HIP_MEMORY_SCOPE_SYSTEM)) + 1;
    if (lane == 0) __hip_atomic_store(cntp, call, __ATOMIC_RELAXED, __HIP_MEMORY_SCOPE_SYSTEM);
    const u32 voff = (lane & 7) < 7 ? (u32)OFF_FLAG + (u32)(mt * 8 + src_of(lane & 7, rank, rot)) * 128 : (u32)OFF_TMO;
    const u32 dst = lds0 + LDS_CTL + C_FLAGS;
    u32 need = 0xFFu, t0_;
    u32x4 f = {0u, 0u, 0u, 0u};
    asm volatile(
      "s_mov_b32 m0, %[dst]\n\t"
      "s_mov_b32 exec_hi, 0\n"
      "1:\n\t"
      "s_mov_b32 exec_lo, %[need]\n\t"
      "s_barrier\n\t"
      "global_load_lds_dwordx4 %[voff], %[fb] sc0 sc1\n\t"
      "s_waitcnt lgkmcnt(0)\n\t"
      "v_min_u32 %[t0], v200, v201\n\t"
      "v_min3_u32 %[t0], v202, v203, %[t0]\n\t"
      "v_cmp_le_u32 vcc, %[call], %[t0]\n\t"
      "v_cmp_le_u32 s[84:85], %[call], v200\n\t"
      "ds_read_b128 v[200:203], %[sn]\n\t"
      "s_andn2_b32 %[need], %[need], vcc_lo\n\t"
      "s_bitcmp1_b32 s84, 7\n\t"
      "s_cselect_b32 %[need], 0x80, %[need]\n\t"
      "s_waitcnt vmcnt(24)\n\t"
      "s_cmp_gt_u32 %[need], 0x80\n\t"
      "s_cbranch_scc1 1b\n\t"
      "s_mov_b64 exec, -1\n\t"
      "s_barrier\n\ts_barrier\n\ts_barrier\n\ts_barrier\n\t"
      "s_waitcnt vmcnt(0) lgkmcnt(0)"
      : [need] "+&s"(need), [t0] "=&v"(t0_), "+{v[200:203]}"(f)
      : [voff] "v"(voff), [fb] "s"(myws), [dst] "s"(dst), [call] "s"(call), [sn] "v"(dst + (lane & 7) * 16)
      : "memory", "scc", "vcc", "s84", "s85");
    STAMP(5);
  }
}

extern "C" void kernel_launch(int rank, int nranks, ncclComm_t comm, hipStream_t stream,
                              void* const* d_in, const int* in_sizes, int n_in,
                              void* d_out, int out_size, const int* in_dtypes, int out_dtype,
                              void* const* d_out_all, void* const* d_ws_all, size_t ws_bytes) {
  if (nranks != NRANK || (unsigned)rank >= (unsigned)NRANK || ws_bytes < WS_NEED || n_in < 4) {
    fprintf(stderr, "unexpected configuration\n");
    return;
  }
  Args a;
  a.x = (const unsigned char*)d_in[0]; a.w = (const unsigned char*)d_in[1];
  a.sx = (const float*)d_in[2]; a.sw = (const float*)d_in[3]; a.out = (float*)d_out;
  for (int q = 0; q < NRANK; ++q) {
    const int d = (rank + 1 + q) % NRANK;
    a.peer[q].xs = a.x + (size_t)d * 512 * KS; a.peer[q].ws = (char*)d_ws_all[d];
  }
  a.myws = (char*)d_ws_all[rank]; a.rank = rank; a.pad = 0;
  HIP_CHECK(hipFuncSetAttribute((const void*)a2a_gemm_kernel, hipFuncAttributeMaxDynamicSharedMemorySize, LDS_BYTES));
  a2a_gemm_kernel<<<256, 512, LDS_BYTES, stream>>>(a);
  HIP_CHECK(hipGetLastError());
}
```

```cpp
#include <hip/hip_runtime.h>
#include <rccl/rccl.h>
#include <cstdio>
#include <type_traits>

#define HIP_CHECK(expr) do { hipError_t _e = (expr); if (_e != hipSuccess) { \
    fprintf(stderr, "HIP error %s at %s:%d\n", hipGetErrorString(_e), __FILE__, __LINE__); \
    return; } } while (0)

typedef unsigned u32;
typedef unsigned long long u64;
typedef unsigned u32x4 __attribute__((ext_vector_type(4)));
typedef int i32x2 __attribute__((ext_vector_type(2)));
typedef int i32x4 __attribute__((ext_vector_type(4)));
typedef int i32x8 __attribute__((ext_vector_type(8)));
typedef float f32x4 __attribute__((ext_vector_type(4)));
#define LAS __attribute__((address_space(3)))

#define DELTA 150
#define LEAD 150
#define TSTART 800
#define NWS 7
#define NAS 4
#define ROTM 1
#define ROTN 0
#define ABL_NOCONS 0
#define ABL_NOLOAD 0
#define CMAP 0
#define LDSLEEP 4
#define V_PUT 1
#define GDELAY 6
#define PROT 1
#define PFMODE 3
#define PFSLEEP 6
#define LROT 1
#define R0POS 1
#define ADAPT 1
#define PC1 99
#define PG1 2
#define PC2 6
#define STPOL " sc1"
#define XPF 1
#define LEADP 0
#define PFN 2
#define CPF 1
#define CPFT 680
#define SLOWH 1
#define PREPOLL 1
#define CODEPF 0
#define RPG 2
#define ABL_NOCOMM 0

namespace {
constexpr int NRANK = 8, GN = 4096, KS = 512;
constexpr size_t OFF_CNT = 0;
constexpr size_t OFF_FLAG = 64 << 10;
constexpr size_t OFF_TMO = 112 << 10;
constexpr size_t OFF_DTAB = 32 << 10;
constexpr size_t OFF_DBG = 128 << 10;
constexpr size_t OFF_SLOT = 1 << 20;
constexpr u32 PIECE = 10240;
constexpr size_t SLOT_BYTES = 4u << 20;
constexpr size_t WS_NEED = OFF_SLOT + 2 * SLOT_BYTES;
constexpr u64 DEADLINE = 40000000ull;

constexpr u32 LDS_W = 0, LDS_A = NWS * 16384, LDS_CTL = LDS_A + NAS * PIECE, LDS_BYTES = LDS_CTL + 256;
constexpr u32 C_FLAGS = 0  , C_STAT = 128, C_DESC = 192  ;
constexpr u32 D_NULL = 0x2007;
static_assert(LDS_BYTES <= 163840 && NAS == 4 && NWS == 7, "LDS budget");

struct Args {
  const unsigned char* x; const unsigned char* w; const float* sx; const float* sw; float* out;
  char* myws; int rank; int pad;
  struct { const unsigned char* xs; char* ws; } peer[NRANK];
};

template <int N> __device__ __forceinline__ void waitvm() { asm volatile("s_waitcnt vmcnt(%0)" :: "i"(N) : "memory"); }
__device__ __forceinline__ void bar() { asm volatile("s_barrier" ::: "memory"); }
__device__ __forceinline__ u32 rfl(u32 v) { return __builtin_amdgcn_readfirstlane(v); }

#define DMA4(A, B, C, D, BASE, SC) \
    "global_load_lds_dwordx4 " A ", " BASE SC "\n\t" \
    "global_load_lds_dwordx4 " B ", " BASE " offset:1024" SC "\n\t" \
    "global_load_lds_dwordx4 " C ", " BASE " offset:2048" SC "\n\t" \
    "global_load_lds_dwordx4 " D ", " BASE " offset:3072" SC "\n\t"
#define M0NEXT "s_add_u32 m0, m0, 0x1000\n\ts_nop 0\n\t"
#define DMA8(SC) \
  asm volatile("s_mov_b32 m0, %9\n\ts_nop 0\n\t" DMA4("%0", "%1", "%2", "%3", "%8", SC) M0NEXT DMA4("%4", "%5", "%6", "%7", "%8", SC) \
    :: "v"(off[0]), "v"(off[1]), "v"(off[2]), "v"(off[3]), "v"(off[4]), "v"(off[5]), "v"(off[6]), "v"(off[7]), \
       "s"(base), "s"(dst) : "memory", "scc")
#define DMA10(SC) \
  asm volatile("s_mov_b32 m0, %11\n\ts_nop 0\n\t" DMA4("%0", "%1", "%2", "%3", "%10", SC) M0NEXT DMA4("%4", "%5", "%6", "%7", "%10", SC) M0NEXT \
    "global_load_lds_dwordx4 %8, %10" SC "\n\tglobal_load_lds_dwordx4 %9, %10 offset:1024" SC \
    :: "v"(off[0]), "v"(off[1]), "v"(off[2]), "v"(off[3]), "v"(off[4]), "v"(off[5]), "v"(off[6]), "v"(off[7]), "v"(off[8]), "v"(off[9]), \
       "s"(base), "s"(dst) : "memory", "scc")
template <int N, bool SYS> __device__ __forceinline__ void dma(const u32 (&off)[10], const void* base, u32 dst) {
  if constexpr (N == 8) { if constexpr (SYS) DMA8(" sc0 sc1"); else DMA8(""); }
  else { if constexpr (SYS) DMA10(" sc0 sc1"); else DMA10(""); }
}

__device__ __forceinline__ int src_of(int q, int rank, int rot) { int i = q + rot; if (i >= 7) i -= 7; return (rank + 1 + i) & 7; }
__device__ __forceinline__ int row0_of(int mt) { return mt < 4 ? 80 * mt : 64 * mt + 64; }

template <int N, int PER> __device__ __forceinline__ void tail_bars() {
  if constexpr (N > 0) { waitvm<PER * (N - 1)>(); bar(); tail_bars<N - 1, PER>(); }
}

#define STAMP(i)
#define DBGARGS
#define DBGPASS

#define CONS_ASM4 \
    "\ts_waitcnt lgkmcnt(0)\n" \
    "\ts_barrier\n" \
    "\tds_read_b32 %[vd], %[dp]\n" \
    "\tds_read_b64_tr_b8 v[64:65], %[wa0] offset:0\n" \
    "\tds_read_b64_tr_b8 v[66:67], %[wa0] offset:1024\n" \
    "\tds_read_b64_tr_b8 v[68:69], %[wa0] offset:8192\n" \
    "\tds_read_b64_tr_b8 v[70:71], %[wa0] offset:9216\n" \
    "\tds_read_b64_tr_b8 v[72:73], %[wa1] offset:0\n" \
    "\tds_read_b64_tr_b8 v[74:75], %[wa1] offset:1024\n" \
    "\tds_read_b64_tr_b8 v[76:77], %[wa1] offset:8192\n" \
    "\tds_read_b64_tr_b8 v[78:79], %[wa1] offset:9216\n" \
    "\ts_waitcnt lgkmcnt(8)\n" \
    "\tds_read_b128 v[80:83], %[ao0] offset:0\n" \
    "\tds_read_b128 v[84:87], %[ao1] offset:0\n" \
    "\tds_read_b128 v[88:91], %[ao0] offset:2048\n" \
    "\tds_read_b128 v[92:95], %[ao1] offset:2048\n" \
    "\tds_read_b128 v[96:99], %[ao0] offset:4096\n" \
    "\tds_read_b128 v[100:103], %[ao1] offset:4096\n" \
    "\tds_read_b128 v[104:107], %[ao0] offset:6144\n" \
    "\tds_read_b128 v[108:111], %[ao1] offset:6144\n" \
    "\tv_cmp_eq_u32 vcc, 0x2007, %[vd]\n" \
    "\tv_and_b32 %[t], 7, %[vd]\n" \
    "\tv_lshl_add_u32 %[wa0], %[t], 14, %[bo0]\n" \
    "\tv_lshl_add_u32 %[wa1], %[t], 14, %[bo1]\n" \
    "\tv_add_u32 %[dp], 4, %[dp]\n" \
    "\tv_bfi_b32 %[dp], 63, %[dp], %[dbase]\n" \
    "3:\n" \
    "\ts_cbranch_vccnz 10f\n" \
    "20:\n" \
    "\ts_waitcnt lgkmcnt(0)\n" \
    "\ts_barrier\n" \
    "\tv_mfma_f32_16x16x128_f8f6f4 %[c0], v[64:71], v[80:87], %[c0] cbsz:1 blgp:1\n" \
    "\tds_read_b64_tr_b8 v[120:121], %[wa0] offset:0\n" \
    "\tds_read_b64_tr_b8 v[122:123], %[wa0] offset:1024\n" \
    "\tds_read_b64_tr_b8 v[124:125], %[wa0] offset:8192\n" \
    "\tds_read_b64_tr_b8 v[126:127], %[wa0] offset:9216\n" \
    "\tv_mfma_f32_16x16x128_f8f6f4 %[c1], v[72:79], v[80:87], %[c1] cbsz:1 blgp:1\n" \
    "\tds_read_b64_tr_b8 v[128:129], %[wa1] offset:0\n" \
    "\tds_read_b64_tr_b8 v[130:131], %[wa1] offset:1024\n" \
    "\tds_read_b64_tr_b8 v[132:133], %[wa1] offset:8192\n" \
    "\tds_read_b64_tr_b8 v[134:135], %[wa1] offset:9216\n" \
    "\tv_mfma_f32_16x16x128_f8f6f4 %[c2], v[64:71], v[88:95], %[c2] cbsz:1 blgp:1\n" \
    "\tds_read_b32 %[vd], %[dp]\n" \
    "\tds_read_b128 v[136:139], %[ao0] offset:10240\n" \
    "\tds_read_b128 v[140:143], %[ao1] offset:10240\n" \
    "\tv_mfma_f32_16x16x128_f8f6f4 %[c3], v[72:79], v[88:95], %[c3] cbsz:1 blgp:1\n" \
    "\tds_read_b128 v[144:147], %[ao0] offset:12288\n" \
    "\tds_read_b128 v[148:151], %[ao1] offset:12288\n" \
    "\tv_mfma_f32_16x16x128_f8f6f4 %[c4], v[64:71], v[96:103], %[c4] cbsz:1 blgp:1\n" \
    "\tds_read_b128 v[152:155], %[ao0] offset:14336\n" \
    "\tds_read_b128 v[156:159], %[ao1] offset:14336\n" \
    "\tv_mfma_f32_16x16x128_f8f6f4 %[c5], v[72:79], v[96:103], %[c5] cbsz:1 blgp:1\n" \
    "\tds_read_b128 v[152:155], %[ao0] offset:14336\n" \
    "\tds_read_b128 v[156:159], %[ao1] offset:14336\n" \
    "\ts_waitcnt lgkmcnt(8)\n" \
    "\tv_and_b32 %[t], 7, %[vd]\n" \
    "\tv_cmp_eq_u32 vcc, 0x2007, %[vd]\n" \
    "\tv_mfma_f32_16x16x128_f8f6f4 %[c6], v[64:71], v[104:111], %[c6] cbsz:1 blgp:1\n" \
    "\tv_lshl_add_u32 %[wa0], %[t], 14, %[bo0]\n" \
    "\tv_lshl_add_u32 %[wa1], %[t], 14, %[bo1]\n" \
    "\tv_add_u32 %[dp], 4, %[dp]\n" \
    "\tv_bfi_b32 %[dp], 63, %[dp], %[dbase]\n" \
    "\tv_mfma_f32_16x16x128_f8f6f4 %[c7], v[72:79], v[104:111], %[c7] cbsz:1 blgp:1\n" \
    "\ts_cbranch_vccnz 11f\n" \
    "21:\n" \
    "\ts_waitcnt lgkmcnt(0)\n" \
    "\ts_barrier\n" \
    "\tv_mfma_f32_16x16x128_f8f6f4 %[c0], v[120:127], v[136:143], %[c0] cbsz:1 blgp:1\n" \
    "\tds_read_b64_tr_b8 v[64:65], %[wa0] offset:0\n" \
    "\tds_read_b64_tr_b8 v[66:67], %[wa0] offset:1024\n" \
    "\tds_read_b64_tr_b8 v[68:69], %[wa0] offset:8192\n" \
    "\tds_read_b64_tr_b8 v[70:71], %[wa0] offset:9216\n" \
    "\tv_mfma_f32_16x16x128_f8f6f4 %[c1], v[128:135], v[136:143], %[c1] cbsz:1 blgp:1\n" \
    "\tds_read_b64_tr_b8 v[72:73], %[wa1] offset:0\n" \
    "\tds_read_b64_tr_b8 v[74:75], %[wa1] offset:1024\n" \
    "\tds_read_b64_tr_b8 v[76:77], %[wa1] offset:8192\n" \
    "\tds_read_b64_tr_b8 v[78:79], %[wa1] offset:9216\n" \
    "\tv_mfma_f32_16x16x128_f8f6f4 %[c2], v[120:127], v[144:151], %[c2] cbsz:1 blgp:1\n" \
    "\tds_read_b32 %[vd], %[dp]\n" \
    "\tds_read_b128 v[80:83], %[ao0] offset:20480\n" \
    "\tds_read_b128 v[84:87], %[ao1] offset:20480\n" \
    "\tv_mfma_f32_16x16x128_f8f6f4 %[c3], v[128:135], v[144:151], %[c3] cbsz:1 blgp:1\n" \
    "\tds_read_b128 v[88:91], %[ao0] offset:22528\n" \
    "\tds_read_b128 v[92:95], %[ao1] offset:22528\n" \
    "\tv_mfma_f32_16x16x128_f8f6f4 %[c4], v[120:127], v[152:159], %[c4] cbsz:1 blgp:1\n" \
    "\tds_read_b128 v[96:99], %[ao0] offset:24576\n" \
    "\tds_read_b128 v[100:103], %[ao1] offset:24576\n" \
    "\tv_mfma_f32_16x16x128_f8f6f4 %[c5], v[128:135], v[152:159], %[c5] cbsz:1 blgp:1\n" \
    "\tds_read_b128 v[96:99], %[ao0] offset:24576\n" \
    "\tds_read_b128 v[100:103], %[ao1] offset:24576\n" \
    "\ts_waitcnt lgkmcnt(8)\n" \
    "\tv_and_b32 %[t], 7, %[vd]\n" \
    "\tv_cmp_eq_u32 vcc, 0x2007, %[vd]\n" \
    "\tv_mfma_f32_16x16x128_f8f6f4 %[c6], v[120:127], v[160:167], %[c6] cbsz:1 blgp:1\n" \
    "\tv_lshl_add_u32 %[wa0], %[t], 14, %[bo0]\n" \
    "\tv_lshl_add_u32 %[wa1], %[t], 14, %[bo1]\n" \
    "\tv_add_u32 %[dp], 4, %[dp]\n" \
    "\tv_bfi_b32 %[dp], 63, %[dp], %[dbase]\n" \
    "\tv_mfma_f32_16x16x128_f8f6f4 %[c7], v[128:135], v[160:167], %[c7] cbsz:1 blgp:1\n" \
    "\ts_cbranch_vccnz 12f\n" \
    "22:\n" \
    "\ts_waitcnt lgkmcnt(0)\n" \
    "\ts_barrier\n" \
    "\tv_mfma_f32_16x16x128_f8f6f4 %[c0], v[64:71], v[80:87], %[c0] cbsz:1 blgp:1\n" \
    "\tds_read_b64_tr_b8 v[120:121], %[wa0] offset:0\n" \
    "\tds_read_b64_tr_b8 v[122:123], %[wa0] offset:1024\n" \
    "\tds_read_b64_tr_b8 v[124:125], %[wa0] offset:8192\n" \
    "\tds_read_b64_tr_b8 v[126:127], %[wa0] offset:9216\n" \
    "\tv_mfma_f32_16x16x128_f8f6f4 %[c1], v[72:79], v[80:87], %[c1] cbsz:1 blgp:1\n" \
    "\tds_read_b64_tr_b8 v[128:129], %[wa1] offset:0\n" \
    "\tds_read_b64_tr_b8 v[130:131], %[wa1] offset:1024\n" \
    "\tds_read_b64_tr_b8 v[132:133], %[wa1] offset:8192\n" \
    "\tds_read_b64_tr_b8 v[134:135], %[wa1] offset:9216\n" \
    "\tv_mfma_f32_16x16x128_f8f6f4 %[c2], v[64:71], v[88:95], %[c2] cbsz:1 blgp:1\n" \
    "\tds_read_b32 %[vd], %[dp]\n" \
    "\tds_read_b128 v[136:139], %[ao0] offset:30720\n" \
    "\tds_read_b128 v[140:143], %[ao1] offset:30720\n" \
    "\tv_mfma_f32_16x16x128_f8f6f4 %[c3], v[72:79], v[88:95], %[c3] cbsz:1 blgp:1\n" \
    "\tds_read_b128 v[144:147], %[ao0] offset:32768\n" \
    "\tds_read_b128 v[148:151], %[ao1] offset:32768\n" \
    "\tv_mfma_f32_16x16x128_f8f6f4 %[c4], v[64:71], v[96:103], %[c4] cbsz:1 blgp:1\n" \
    "\tds_read_b128 v[152:155], %[ao0] offset:34816\n" \
    "\tds_read_b128 v[156:159], %[ao1] offset:34816\n" \
    "\tv_mfma_f32_16x16x128_f8f6f4 %[c5], v[72:79], v[96:103], %[c5] cbsz:1 blgp:1\n" \
    "\tds_read_b128 v[152:155], %[ao0] offset:34816\n" \
    "\tds_read_b128 v[156:159], %[ao1] offset:34816\n" \
    "\ts_waitcnt lgkmcnt(8)\n" \
    "\tv_and_b32 %[t], 7, %[vd]\n" \
    "\tv_cmp_eq_u32 vcc, 0x2007, %[vd]\n" \
    "\tv_mfma_f32_16x16x128_f8f6f4 %[c6], v[64:71], v[104:111], %[c6] cbsz:1 blgp:1\n" \
    "\tv_lshl_add_u32 %[wa0], %[t], 14, %[bo0]\n" \
    "\tv_lshl_add_u32 %[wa1], %[t], 14, %[bo1]\n" \
    "\tv_add_u32 %[dp], 4, %[dp]\n" \
    "\tv_bfi_b32 %[dp], 63, %[dp], %[dbase]\n" \
    "\ts_sub_u32 %[n], %[n], 1\n" \
    "\ts_cmp_eq_u32 %[n], 0\n" \
    "\tv_mfma_f32_16x16x128_f8f6f4 %[c7], v[72:79], v[104:111], %[c7] cbsz:1 blgp:1\n" \
    "\ts_cbranch_scc1 4f\n" \
    "\ts_cbranch_vccnz 13f\n" \
    "23:\n" \
    "\ts_waitcnt lgkmcnt(0)\n" \
    "\ts_barrier\n" \
    "\tv_mfma_f32_16x16x128_f8f6f4 %[c0], v[120:127], v[136:143], %[c0] cbsz:1 blgp:1\n" \
    "\tds_read_b64_tr_b8 v[64:65], %[wa0] offset:0\n" \
    "\tds_read_b64_tr_b8 v[66:67], %[wa0] offset:1024\n" \
    "\tds_read_b64_tr_b8 v[68:69], %[wa0] offset:8192\n" \
    "\tds_read_b64_tr_b8 v[70:71], %[wa0] offset:9216\n" \
    "\tv_mfma_f32_16x16x128_f8f6f4 %[c1], v[128:135], v[136:143], %[c1] cbsz:1 blgp:1\n" \
    "\tds_read_b64_tr_b8 v[72:73], %[wa1] offset:0\n" \
    "\tds_read_b64_tr_b8 v[74:75], %[wa1] offset:1024\n" \
    "\tds_read_b64_tr_b8 v[76:77], %[wa1] offset:8192\n" \
    "\tds_read_b64_tr_b8 v[78:79], %[wa1] offset:9216\n" \
    "\tv_mfma_f32_16x16x128_f8f6f4 %[c2], v[120:127], v[144:151], %[c2] cbsz:1 blgp:1\n" \
    "\tds_read_b32 %[vd], %[dp]\n" \
    "\tds_read_b128 v[80:83], %[ao0] offset:0\n" \
    "\tds_read_b128 v[84:87], %[ao1] offset:0\n" \
    "\tv_mfma_f32_16x16x128_f8f6f4 %[c3], v[128:135], v[144:151], %[c3] cbsz:1 blgp:1\n" \
    "\tds_read_b128 v[88:91], %[ao0] offset:2048\n" \
    "\tds_read_b128 v[92:95], %[ao1] offset:2048\n" \
    "\tv_mfma_f32_16x16x128_f8f6f4 %[c4], v[120:127], v[152:159], %[c4] cbsz:1 blgp:1\n" \
    "\tds_read_b128 v[96:99], %[ao0] offset:4096\n" \
    "\tds_read_b128 v[100:103], %[ao1] offset:4096\n" \
    "\tv_mfma_f32_16x16x128_f8f6f4 %[c5], v[128:135], v[152:159], %[c5] cbsz:1 blgp:1\n" \
    "\tds_read_b128 v[96:99], %[ao0] offset:4096\n" \
    "\tds_read_b128 v[100:103], %[ao1] offset:4096\n" \
    "\ts_waitcnt lgkmcnt(8)\n" \
    "\tv_and_b32 %[t], 7, %[vd]\n" \
    "\tv_cmp_eq_u32 vcc, 0x2007, %[vd]\n" \
    "\tv_mfma_f32_16x16x128_f8f6f4 %[c6], v[120:127], v[160:167], %[c6] cbsz:1 blgp:1\n" \
    "\tv_lshl_add_u32 %[wa0], %[t], 14, %[bo0]\n" \
    "\tv_lshl_add_u32 %[wa1], %[t], 14, %[bo1]\n" \
    "\tv_add_u32 %[dp], 4, %[dp]\n" \
    "\tv_bfi_b32 %[dp], 63, %[dp], %[dbase]\n" \
    "\tv_mfma_f32_16x16x128_f8f6f4 %[c7], v[128:135], v[160:167], %[c7] cbsz:1 blgp:1\n" \
    "\ts_branch 3b\n" \
    "10:\n" \
    "\ts_waitcnt lgkmcnt(0)\n" \
    "\ts_barrier\n" \
    "\tds_read_b32 %[vd], %[dp]\n" \
    "\tv_add_u32 %[dp], 4, %[dp]\n" \
    "\tv_bfi_b32 %[dp], 63, %[dp], %[dbase]\n" \
    "\ts_waitcnt lgkmcnt(0)\n" \
    "\tv_cmp_eq_u32 vcc, 0x2007, %[vd]\n" \
    "\tv_and_b32 %[t], 7, %[vd]\n" \
    "\tv_lshl_add_u32 %[wa0], %[t], 14, %[bo0]\n" \
    "\tv_lshl_add_u32 %[wa1], %[t], 14, %[bo1]\n" \
    "\ts_cbranch_vccnz 10b\n" \
    "\ts_branch 20b\n" \
    "11:\n" \
    "\ts_waitcnt lgkmcnt(0)\n" \
    "\ts_barrier\n" \
    "\tds_read_b32 %[vd], %[dp]\n" \
    "\tv_add_u32 %[dp], 4, %[dp]\n" \
    "\tv_bfi_b32 %[dp], 63, %[dp], %[dbase]\n" \
    "\ts_waitcnt lgkmcnt(0)\n" \
    "\tv_cmp_eq_u32 vcc, 0x2007, %[vd]\n" \
    "\tv_and_b32 %[t], 7, %[vd]\n" \
    "\tv_lshl_add_u32 %[wa0], %[t], 14, %[bo0]\n" \
    "\tv_lshl_add_u32 %[wa1], %[t], 14, %[bo1]\n" \
    "\ts_cbranch_vccnz 11b\n" \
    "\ts_branch 21b\n" \
    "12:\n" \
    "\ts_waitcnt lgkmcnt(0)\n" \
    "\ts_barrier\n" \
    "\tds_read_b32 %[vd], %[dp]\n" \
    "\tv_add_u32 %[dp], 4, %[dp]\n" \
    "\tv_bfi_b32 %[dp], 63, %[dp], %[dbase]\n" \
    "\ts_waitcnt lgkmcnt(0)\n" \
    "\tv_cmp_eq_u32 vcc, 0x2007, %[vd]\n" \
    "\tv_and_b32 %[t], 7, %[vd]\n" \
    "\tv_lshl_add_u32 %[wa0], %[t], 14, %[bo0]\n" \
    "\tv_lshl_add_u32 %[wa1], %[t], 14, %[bo1]\n" \
    "\ts_cbranch_vccnz 12b\n" \
    "\ts_branch 22b\n" \
    "13:\n" \
    "\ts_waitcnt lgkmcnt(0)\n" \
    "\ts_barrier\n" \
    "\tds_read_b32 %[vd], %[dp]\n" \
    "\tv_add_u32 %[dp], 4, %[dp]\n" \
    "\tv_bfi_b32 %[dp], 63, %[dp], %[dbase]\n" \
    "\ts_waitcnt lgkmcnt(0)\n" \
    "\tv_cmp_eq_u32 vcc, 0x2007, %[vd]\n" \
    "\tv_and_b32 %[t], 7, %[vd]\n" \
    "\tv_lshl_add_u32 %[wa0], %[t], 14, %[bo0]\n" \
    "\tv_lshl_add_u32 %[wa1], %[t], 14, %[bo1]\n" \
    "\ts_cbranch_vccnz 13b\n" \
    "\ts_branch 23b\n" \
    "4:\n" \
    "\tds_read_b32 %[vd], %[stat]\n" \
    "\ts_waitcnt lgkmcnt(1)\n" \
    "\tv_mfma_f32_16x16x128_f8f6f4 %[c0], v[120:127], v[136:143], %[c0] cbsz:1 blgp:1\n" \
    "\tv_mfma_f32_16x16x128_f8f6f4 %[c1], v[128:135], v[136:143], %[c1] cbsz:1 blgp:1\n" \
    "\tv_mfma_f32_16x16x128_f8f6f4 %[c2], v[120:127], v[144:151], %[c2] cbsz:1 blgp:1\n" \
    "\tv_mfma_f32_16x16x128_f8f6f4 %[c3], v[128:135], v[144:151], %[c3] cbsz:1 blgp:1\n" \
    "\tv_mfma_f32_16x16x128_f8f6f4 %[c4], v[120:127], v[152:159], %[c4] cbsz:1 blgp:1\n" \
    "\tv_mfma_f32_16x16x128_f8f6f4 %[c5], v[128:135], v[152:159], %[c5] cbsz:1 blgp:1\n" \
    "\tv_mfma_f32_16x16x128_f8f6f4 %[c6], v[120:127], v[160:167], %[c6] cbsz:1 blgp:1\n" \
    "\tv_mfma_f32_16x16x128_f8f6f4 %[c7], v[128:135], v[160:167], %[c7] cbsz:1 blgp:1\n" \
    "\ts_waitcnt lgkmcnt(0)\n" \
    "\ts_nop 7\n" \
    "\ts_nop 3\n"
#define CONS_ASM5 \
    "\ts_waitcnt lgkmcnt(0)\n" \
    "\ts_barrier\n" \
    "\tds_read_b32 %[vd], %[dp]\n" \
    "\tds_read_b64_tr_b8 v[64:65], %[wa0] offset:0\n" \
    "\tds_read_b64_tr_b8 v[66:67], %[wa0] offset:1024\n" \
    "\tds_read_b64_tr_b8 v[68:69], %[wa0] offset:8192\n" \
    "\tds_read_b64_tr_b8 v[70:71], %[wa0] offset:9216\n" \
    "\tds_read_b64_tr_b8 v[72:73], %[wa1] offset:0\n" \
    "\tds_read_b64_tr_b8 v[74:75], %[wa1] offset:1024\n" \
    "\tds_read_b64_tr_b8 v[76:77], %[wa1] offset:8192\n" \
    "\tds_read_b64_tr_b8 v[78:79], %[wa1] offset:9216\n" \
    "\ts_waitcnt lgkmcnt(8)\n" \
    "\tds_read_b128 v[80:83], %[ao0] offset:0\n" \
    "\tds_read_b128 v[84:87], %[ao1] offset:0\n" \
    "\tds_read_b128 v[88:91], %[ao0] offset:2048\n" \
    "\tds_read_b128 v[92:95], %[ao1] offset:2048\n" \
    "\tds_read_b128 v[96:99], %[ao0] offset:4096\n" \
    "\tds_read_b128 v[100:103], %[ao1] offset:4096\n" \
    "\tds_read_b128 v[104:107], %[ao0] offset:6144\n" \
    "\tds_read_b128 v[108:111], %[ao1] offset:6144\n" \
    "\tds_read_b128 v[112:115], %[ao0] offset:8192\n" \
    "\tds_read_b128 v[116:119], %[ao1] offset:8192\n" \
    "\tv_cmp_eq_u32 vcc, 0x2007, %[vd]\n" \
    "\tv_and_b32 %[t], 7, %[vd]\n" \
    "\tv_lshl_add_u32 %[wa0], %[t], 14, %[bo0]\n" \
    "\tv_lshl_add_u32 %[wa1], %[t], 14, %[bo1]\n" \
    "\tv_add_u32 %[dp], 4, %[dp]\n" \
    "\tv_bfi_b32 %[dp], 63, %[dp], %[dbase]\n" \
    "3:\n" \
    "\ts_cbranch_vccnz 10f\n" \
    "20:\n" \
    "\ts_waitcnt lgkmcnt(0)\n" \
    "\ts_barrier\n" \
    "\tv_mfma_f32_16x16x128_f8f6f4 %[c0], v[64:71], v[80:87], %[c0] cbsz:1 blgp:1\n" \
    "\tds_read_b64_tr_b8 v[120:121], %[wa0] offset:0\n" \
    "\tds_read_b64_tr_b8 v[122:123], %[wa0] offset:1024\n" \
    "\tds_read_b64_tr_b8 v[124:125], %[wa0] offset:8192\n" \
    "\tds_read_b64_tr_b8 v[126:127], %[wa0] offset:9216\n" \
    "\tv_mfma_f32_16x16x128_f8f6f4 %[c1], v[72:79], v[80:87], %[c1] cbsz:1 blgp:1\n" \
    "\tds_read_b64_tr_b8 v[128:129], %[wa1] offset:0\n" \
    "\tds_read_b64_tr_b8 v[130:131], %[wa1] offset:1024\n" \
    "\tds_read_b64_tr_b8 v[132:133], %[wa1] offset:8192\n" \
    "\tds_read_b64_tr_b8 v[134:135], %[wa1] offset:9216\n" \
    "\tv_mfma_f32_16x16x128_f8f6f4 %[c2], v[64:71], v[88:95], %[c2] cbsz:1 blgp:1\n" \
    "\tds_read_b32 %[vd], %[dp]\n" \
    "\tds_read_b128 v[136:139], %[ao0] offset:10240\n" \
    "\tds_read_b128 v[140:143], %[ao1] offset:10240\n" \
    "\tv_mfma_f32_16x16x128_f8f6f4 %[c3], v[72:79], v[88:95], %[c3] cbsz:1 blgp:1\n" \
    "\tds_read_b128 v[144:147], %[ao0] offset:12288\n" \
    "\tds_read_b128 v[148:151], %[ao1] offset:12288\n" \
    "\tv_mfma_f32_16x16x128_f8f6f4 %[c4], v[64:71], v[96:103], %[c4] cbsz:1 blgp:1\n" \
    "\tds_read_b128 v[152:155], %[ao0] offset:14336\n" \
    "\tds_read_b128 v[156:159], %[ao1] offset:14336\n" \
    "\tv_mfma_f32_16x16x128_f8f6f4 %[c5], v[72:79], v[96:103], %[c5] cbsz:1 blgp:1\n" \
    "\tds_read_b128 v[152:155], %[ao0] offset:14336\n" \
    "\tds_read_b128 v[156:159], %[ao1] offset:14336\n" \
    "\tv_mfma_f32_16x16x128_f8f6f4 %[c6], v[64:71], v[104:111], %[c6] cbsz:1 blgp:1\n" \
    "\tds_read_b128 v[160:163], %[ao0] offset:16384\n" \
    "\tds_read_b128 v[164:167], %[ao1] offset:16384\n" \
    "\tv_mfma_f32_16x16x128_f8f6f4 %[c7], v[72:79], v[104:111], %[c7] cbsz:1 blgp:1\n" \
    "\ts_waitcnt lgkmcnt(10)\n" \
    "\tv_and_b32 %[t], 7, %[vd]\n" \
    "\tv_lshl_add_u32 %[wa0], %[t], 14, %[bo0]\n" \
    "\tv_lshl_add_u32 %[wa1], %[t], 14, %[bo1]\n" \
    "\tv_mfma_f32_16x16x128_f8f6f4 %[c8], v[64:71], v[112:119], %[c8] cbsz:1 blgp:1\n" \
    "\tv_cmp_eq_u32 vcc, 0x2007, %[vd]\n" \
    "\tv_add_u32 %[dp], 4, %[dp]\n" \
    "\tv_bfi_b32 %[dp], 63, %[dp], %[dbase]\n" \
    "\tv_mfma_f32_16x16x128_f8f6f4 %[c9], v[72:79], v[112:119], %[c9] cbsz:1 blgp:1\n" \
    "\ts_cbranch_vccnz 11f\n" \
    "21:\n" \
    "\ts_waitcnt lgkmcnt(0)\n" \
    "\ts_barrier\n" \
    "\tv_mfma_f32_16x16x128_f8f6f4 %[c0], v[120:127], v[136:143], %[c0] cbsz:1 blgp:1\n" \
    "\tds_read_b64_tr_b8 v[64:65], %[wa0] offset:0\n" \
    "\tds_read_b64_tr_b8 v[66:67], %[wa0] offset:1024\n" \
    "\tds_read_b64_tr_b8 v[68:69], %[wa0] offset:8192\n" \
    "\tds_read_b64_tr_b8 v[70:71], %[wa0] offset:9216\n" \
    "\tv_mfma_f32_16x16x128_f8f6f4 %[c1], v[128:135], v[136:143], %[c1] cbsz:1 blgp:1\n" \
    "\tds_read_b64_tr_b8 v[72:73], %[wa1] offset:0\n" \
    "\tds_read_b64_tr_b8 v[74:75], %[wa1] offset:1024\n" \
    "\tds_read_b64_tr_b8 v[76:77], %[wa1] offset:8192\n" \
    "\tds_read_b64_tr_b8 v[78:79], %[wa1] offset:9216\n" \
    "\tv_mfma_f32_16x16x128_f8f6f4 %[c2], v[120:127], v[144:151], %[c2] cbsz:1 blgp:1\n" \
    "\tds_read_b32 %[vd], %[dp]\n" \
    "\tds_read_b128 v[80:83], %[ao0] offset:20480\n" \
    "\tds_read_b128 v[84:87], %[ao1] offset:20480\n" \
    "\tv_mfma_f32_16x16x128_f8f6f4 %[c3], v[128:135], v[144:151], %[c3] cbsz:1 blgp:1\n" \
    "\tds_read_b128 v[88:91], %[ao0] offset:22528\n" \
    "\tds_read_b128 v[92:95], %[ao1] offset:22528\n" \
    "\tv_mfma_f32_16x16x128_f8f6f4 %[c4], v[120:127], v[152:159], %[c4] cbsz:1 blgp:1\n" \
    "\tds_read_b128 v[96:99], %[ao0] offset:24576\n" \
    "\tds_read_b128 v[100:103], %[ao1] offset:24576\n" \
    "\tv_mfma_f32_16x16x128_f8f6f4 %[c5], v[128:135], v[152:159], %[c5] cbsz:1 blgp:1\n" \
    "\tds_read_b128 v[96:99], %[ao0] offset:24576\n" \
    "\tds_read_b128 v[100:103], %[ao1] offset:24576\n" \
    "\tv_mfma_f32_16x16x128_f8f6f4 %[c6], v[120:127], v[160:167], %[c6] cbsz:1 blgp:1\n" \
    "\tds_read_b128 v[104:107], %[ao0] offset:26624\n" \
    "\tds_read_b128 v[108:111], %[ao1] offset:26624\n" \
    "\tv_mfma_f32_16x16x128_f8f6f4 %[c7], v[128:135], v[160:167], %[c7] cbsz:1 blgp:1\n" \
    "\ts_waitcnt lgkmcnt(10)\n" \
    "\tv_and_b32 %[t], 7, %[vd]\n" \
    "\tv_lshl_add_u32 %[wa0], %[t], 14, %[bo0]\n" \
    "\tv_lshl_add_u32 %[wa1], %[t], 14, %[bo1]\n" \
    "\tv_mfma_f32_16x16x128_f8f6f4 %[c8], v[120:127], v[168:175], %[c8] cbsz:1 blgp:1\n" \
    "\tv_cmp_eq_u32 vcc, 0x2007, %[vd]\n" \
    "\tv_add_u32 %[dp], 4, %[dp]\n" \
    "\tv_bfi_b32 %[dp], 63, %[dp], %[dbase]\n" \
    "\tv_mfma_f32_16x16x128_f8f6f4 %[c9], v[128:135], v[168:175], %[c9] cbsz:1 blgp:1\n" \
    "\ts_cbranch_vccnz 12f\n" \
    "22:\n" \
    "\ts_waitcnt lgkmcnt(0)\n" \
    "\ts_barrier\n" \
    "\tv_mfma_f32_16x16x128_f8f6f4 %[c0], v[64:71], v[80:87], %[c0] cbsz:1 blgp:1\n" \
    "\tds_read_b64_tr_b8 v[120:121], %[wa0] offset:0\n" \
    "\tds_read_b64_tr_b8 v[122:123], %[wa0] offset:1024\n" \
    "\tds_read_b64_tr_b8 v[124:125], %[wa0] offset:8192\n" \
    "\tds_read_b64_tr_b8 v[126:127], %[wa0] offset:9216\n" \
    "\tv_mfma_f32_16x16x128_f8f6f4 %[c1], v[72:79], v[80:87], %[c1] cbsz:1 blgp:1\n" \
    "\tds_read_b64_tr_b8 v[128:129], %[wa1] offset:0\n" \
    "\tds_read_b64_tr_b8 v[130:131], %[wa1] offset:1024\n" \
    "\tds_read_b64_tr_b8 v[132:133], %[wa1] offset:8192\n" \
    "\tds_read_b64_tr_b8 v[134:135], %[wa1] offset:9216\n" \
    "\tv_mfma_f32_16x16x128_f8f6f4 %[c2], v[64:71], v[88:95], %[c2] cbsz:1 blgp:1\n" \
    "\tds_read_b32 %[vd], %[dp]\n" \
    "\tds_read_b128 v[136:139], %[ao0] offset:30720\n" \
    "\tds_read_b128 v[140:143], %[ao1] offset:30720\n" \
    "\tv_mfma_f32_16x16x128_f8f6f4 %[c3], v[72:79], v[88:95], %[c3] cbsz:1 blgp:1\n" \
    "\tds_read_b128 v[144:147], %[ao0] offset:32768\n" \
    "\tds_read_b128 v[148:151], %[ao1] offset:32768\n" \
    "\tv_mfma_f32_16x16x128_f8f6f4 %[c4], v[64:71], v[96:103], %[c4] cbsz:1 blgp:1\n" \
    "\tds_read_b128 v[152:155], %[ao0] offset:34816\n" \
    "\tds_read_b128 v[156:159], %[ao1] offset:34816\n" \
    "\tv_mfma_f32_16x16x128_f8f6f4 %[c5], v[72:79], v[96:103], %[c5] cbsz:1 blgp:1\n" \
    "\tds_read_b128 v[152:155], %[ao0] offset:34816\n" \
    "\tds_read_b128 v[156:159], %[ao1] offset:34816\n" \
    "\tv_mfma_f32_16x16x128_f8f6f4 %[c6], v[64:71], v[104:111], %[c6] cbsz:1 blgp:1\n" \
    "\tds_read_b128 v[160:163], %[ao0] offset:36864\n" \
    "\tds_read_b128 v[164:167], %[ao1] offset:36864\n" \
    "\tv_mfma_f32_16x16x128_f8f6f4 %[c7], v[72:79], v[104:111], %[c7] cbsz:1 blgp:1\n" \
    "\ts_waitcnt lgkmcnt(10)\n" \
    "\tv_and_b32 %[t], 7, %[vd]\n" \
    "\tv_lshl_add_u32 %[wa0], %[t], 14, %[bo0]\n" \
    "\tv_lshl_add_u32 %[wa1], %[t], 14, %[bo1]\n" \
    "\tv_mfma_f32_16x16x128_f8f6f4 %[c8], v[64:71], v[112:119], %[c8] cbsz:1 blgp:1\n" \
    "\tv_cmp_eq_u32 vcc, 0x2007, %[vd]\n" \
    "\tv_add_u32 %[dp], 4, %[dp]\n" \
    "\tv_bfi_b32 %[dp], 63, %[dp], %[dbase]\n" \
    "\ts_sub_u32 %[n], %[n], 1\n" \
    "\ts_cmp_eq_u32 %[n], 0\n" \
    "\tv_mfma_f32_16x16x128_f8f6f4 %[c9], v[72:79], v[112:119], %[c9] cbsz:1 blgp:1\n" \
    "\ts_cbranch_scc1 4f\n" \
    "\ts_cbranch_vccnz 13f\n" \
    "23:\n" \
    "\ts_waitcnt lgkmcnt(0)\n" \
    "\ts_barrier\n" \
    "\tv_mfma_f32_16x16x128_f8f6f4 %[c0], v[120:127], v[136:143], %[c0] cbsz:1 blgp:1\n" \
    "\tds_read_b64_tr_b8 v[64:65], %[wa0] offset:0\n" \
    "\tds_read_b64_tr_b8 v[66:67], %[wa0] offset:1024\n" \
    "\tds_read_b64_tr_b8 v[68:69], %[wa0] offset:8192\n" \
    "\tds_read_b64_tr_b8 v[70:71], %[wa0] offset:9216\n" \
    "\tv_mfma_f32_16x16x128_f8f6f4 %[c1], v[128:135], v[136:143], %[c1] cbsz:1 blgp:1\n" \
    "\tds_read_b64_tr_b8 v[72:73], %[wa1] offset:0\n" \
    "\tds_read_b64_tr_b8 v[74:75], %[wa1] offset:1024\n" \
    "\tds_read_b64_tr_b8 v[76:77], %[wa1] offset:8192\n" \
    "\tds_read_b64_tr_b8 v[78:79], %[wa1] offset:9216\n" \
    "\tv_mfma_f32_16x16x128_f8f6f4 %[c2], v[120:127], v[144:151], %[c2] cbsz:1 blgp:1\n" \
    "\tds_read_b32 %[vd], %[dp]\n" \
    "\tds_read_b128 v[80:83], %[ao0] offset:0\n" \
    "\tds_read_b128 v[84:87], %[ao1] offset:0\n" \
    "\tv_mfma_f32_16x16x128_f8f6f4 %[c3], v[128:135], v[144:151], %[c3] cbsz:1 blgp:1\n" \
    "\tds_read_b128 v[88:91], %[ao0] offset:2048\n" \
    "\tds_read_b128 v[92:95], %[ao1] offset:2048\n" \
    "\tv_mfma_f32_16x16x128_f8f6f4 %[c4], v[120:127], v[152:159], %[c4] cbsz:1 blgp:1\n" \
    "\tds_read_b128 v[96:99], %[ao0] offset:4096\n" \
    "\tds_read_b128 v[100:103], %[ao1] offset:4096\n" \
    "\tv_mfma_f32_16x16x128_f8f6f4 %[c5], v[128:135], v[152:159], %[c5] cbsz:1 blgp:1\n" \
    "\tds_read_b128 v[96:99], %[ao0] offset:4096\n" \
    "\tds_read_b128 v[100:103], %[ao1] offset:4096\n" \
    "\tv_mfma_f32_16x16x128_f8f6f4 %[c6], v[120:127], v[160:167], %[c6] cbsz:1 blgp:1\n" \
    "\tds_read_b128 v[104:107], %[ao0] offset:6144\n" \
    "\tds_read_b128 v[108:111], %[ao1] offset:6144\n" \
    "\tv_mfma_f32_16x16x128_f8f6f4 %[c7], v[128:135], v[160:167], %[c7] cbsz:1 blgp:1\n" \
    "\ts_waitcnt lgkmcnt(10)\n" \
    "\tv_and_b32 %[t], 7, %[vd]\n" \
    "\tv_lshl_add_u32 %[wa0], %[t], 14, %[bo0]\n" \
    "\tv_lshl_add_u32 %[wa1], %[t], 14, %[bo1]\n" \
    "\tv_mfma_f32_16x16x128_f8f6f4 %[c8], v[120:127], v[168:175], %[c8] cbsz:1 blgp:1\n" \
    "\tv_cmp_eq_u32 vcc, 0x2007, %[vd]\n" \
    "\tv_add_u32 %[dp], 4, %[dp]\n" \
    "\tv_bfi_b32 %[dp], 63, %[dp], %[dbase]\n" \
    "\tv_mfma_f32_16x16x128_f8f6f4 %[c9], v[128:135], v[168:175], %[c9] cbsz:1 blgp:1\n" \
    "\ts_branch 3b\n" \
    "10:\n" \
    "\ts_waitcnt lgkmcnt(0)\n" \
    "\ts_barrier\n" \
    "\tds_read_b32 %[vd], %[dp]\n" \
    "\tv_add_u32 %[dp], 4, %[dp]\n" \
    "\tv_bfi_b32 %[dp], 63, %[dp], %[dbase]\n" \
    "\ts_waitcnt lgkmcnt(0)\n" \
    "\tv_cmp_eq_u32 vcc, 0x2007, %[vd]\n" \
    "\tv_and_b32 %[t], 7, %[vd]\n" \
    "\tv_lshl_add_u32 %[wa0], %[t], 14, %[bo0]\n" \
    "\tv_lshl_add_u32 %[wa1], %[t], 14, %[bo1]\n" \
    "\ts_cbranch_vccnz 10b\n" \
    "\ts_branch 20b\n" \
    "11:\n" \
    "\ts_waitcnt lgkmcnt(0)\n" \
    "\ts_barrier\n" \
    "\tds_read_b32 %[vd], %[dp]\n" \
    "\tv_add_u32 %[dp], 4, %[dp]\n" \
    "\tv_bfi_b32 %[dp], 63, %[dp], %[dbase]\n" \
    "\ts_waitcnt lgkmcnt(0)\n" \
    "\tv_cmp_eq_u32 vcc, 0x2007, %[vd]\n" \
    "\tv_and_b32 %[t], 7, %[vd]\n" \
    "\tv_lshl_add_u32 %[wa0], %[t], 14, %[bo0]\n" \
    "\tv_lshl_add_u32 %[wa1], %[t], 14, %[bo1]\n" \
    "\ts_cbranch_vccnz 11b\n" \
    "\ts_branch 21b\n" \
    "12:\n" \
    "\ts_waitcnt lgkmcnt(0)\n" \
    "\ts_barrier\n" \
    "\tds_read_b32 %[vd], %[dp]\n" \
    "\tv_add_u32 %[dp], 4, %[dp]\n" \
    "\tv_bfi_b32 %[dp], 63, %[dp], %[dbase]\n" \
    "\ts_waitcnt lgkmcnt(0)\n" \
    "\tv_cmp_eq_u32 vcc, 0x2007, %[vd]\n" \
    "\tv_and_b32 %[t], 7, %[vd]\n" \
    "\tv_lshl_add_u32 %[wa0], %[t], 14, %[bo0]\n" \
    "\tv_lshl_add_u32 %[wa1], %[t], 14, %[bo1]\n" \
    "\ts_cbranch_vccnz 12b\n" \
    "\ts_branch 22b\n" \
    "13:\n" \
    "\ts_waitcnt lgkmcnt(0)\n" \
    "\ts_barrier\n" \
    "\tds_read_b32 %[vd], %[dp]\n" \
    "\tv_add_u32 %[dp], 4, %[dp]\n" \
    "\tv_bfi_b32 %[dp], 63, %[dp], %[dbase]\n" \
    "\ts_waitcnt lgkmcnt(0)\n" \
    "\tv_cmp_eq_u32 vcc, 0x2007, %[vd]\n" \
    "\tv_and_b32 %[t], 7, %[vd]\n" \
    "\tv_lshl_add_u32 %[wa0], %[t], 14, %[bo0]\n" \
    "\tv_lshl_add_u32 %[wa1], %[t], 14, %[bo1]\n" \
    "\ts_cbranch_vccnz 13b\n" \
    "\ts_branch 23b\n" \
    "4:\n" \
    "\tds_read_b32 %[vd], %[stat]\n" \
    "\ts_waitcnt lgkmcnt(1)\n" \
    "\tv_mfma_f32_16x16x128_f8f6f4 %[c0], v[120:127], v[136:143], %[c0] cbsz:1 blgp:1\n" \
    "\tv_mfma_f32_16x16x128_f8f6f4 %[c1], v[128:135], v[136:143], %[c1] cbsz:1 blgp:1\n" \
    "\tv_mfma_f32_16x16x128_f8f6f4 %[c2], v[120:127], v[144:151], %[c2] cbsz:1 blgp:1\n" \
    "\tv_mfma_f32_16x16x128_f8f6f4 %[c3], v[128:135], v[144:151], %[c3] cbsz:1 blgp:1\n" \
    "\tv_mfma_f32_16x16x128_f8f6f4 %[c4], v[120:127], v[152:159], %[c4] cbsz:1 blgp:1\n" \
    "\tv_mfma_f32_16x16x128_f8f6f4 %[c5], v[128:135], v[152:159], %[c5] cbsz:1 blgp:1\n" \
    "\tv_mfma_f32_16x16x128_f8f6f4 %[c6], v[120:127], v[160:167], %[c6] cbsz:1 blgp:1\n" \
    "\tv_mfma_f32_16x16x128_f8f6f4 %[c7], v[128:135], v[160:167], %[c7] cbsz:1 blgp:1\n" \
    "\tv_mfma_f32_16x16x128_f8f6f4 %[c8], v[120:127], v[168:175], %[c8] cbsz:1 blgp:1\n" \
    "\tv_mfma_f32_16x16x128_f8f6f4 %[c9], v[128:135], v[168:175], %[c9] cbsz:1 blgp:1\n" \
    "\ts_waitcnt lgkmcnt(0)\n" \
    "\ts_nop 7\n" \
    "\ts_nop 3\n"

#define CONS_CLOB "memory", "scc", "vcc", "s84", "s85", "v64", "v65", "v66", "v67", "v68", "v69", "v70", "v71", "v72", "v73", "v74", "v75", "v76", "v77", "v78", "v79", "v80", "v81", "v82", "v83", "v84", "v85", "v86", "v87", "v88", "v89", "v90", "v91", "v92", "v93", "v94", "v95", "v96", "v97", "v98", "v99", "v100", "v101", "v102", "v103", "v104", "v105", "v106", "v107", "v108", "v109", "v110", "v111", "v112", "v113", "v114", "v115", "v116", "v117", "v118", "v119", "v120", "v121", "v122", "v123", "v124", "v125", "v126", "v127", "v128", "v129", "v130", "v131", "v132", "v133", "v134", "v135", "v136", "v137", "v138", "v139", "v140", "v141", "v142", "v143", "v144", "v145", "v146", "v147", "v148", "v149", "v150", "v151", "v152", "v153", "v154", "v155", "v156", "v157", "v158", "v159", "v160", "v161", "v162", "v163", "v164", "v165", "v166", "v167", "v168", "v169", "v170", "v171", "v172", "v173", "v174", "v175"
template <int RT>
__device__ __forceinline__ void consumer(const Args& p, u32 lds0, int wave, int lane, int mt, int n0 DBGARGS) {
  const u32 ctl = lds0 + LDS_CTL;
  if (wave == 1 && lane >= 32) *(volatile LAS u32*)(ctl + lane * 4) = (lane >= 48 && lane < 52) ? (u32)(lane - 45) * 0x101u : 0u;
  const float scale = *p.sx * *p.sw;
  const int g = lane >> 4, t = lane & 15;
  u32 aoff[2], boff[2];
  #pragma unroll
  for (int i = 0; i < 2; ++i) {
    aoff[i] = lds0 + LDS_A + 128 * t + 16 * ((g + 4 * i) ^ (t >> 1));
    boff[i] = lds0 + LDS_W + 128 * (16 * g + (t >> 1)) + 16 * ((2 * wave + i) ^ ((t >> 2) | ((g & 1) << 2))) + 8 * (t & 1);
  }
  f32x4 acc[10] = {};
  u32 dp = ctl + C_DESC + 4, wa0 = boff[0] + 3 * 16384u, wa1 = boff[1] + 3 * 16384u, n = 8, vd, tmp;
  u32 stv = 0;
  float* o = p.out + (size_t)(row0_of(mt) + t) * GN + n0 + wave * 32 + 4 * g;
#define CONS_IN [ao0] "v"(aoff[0]), [ao1] "v"(aoff[1]), [bo0] "v"(boff[0]), [bo1] "v"(boff[1]), [stat] "v"(ctl + C_STAT), [dbase] "s"(ctl + C_DESC)
#define CONS_OUT [c0] "+&v"(acc[0]), [c1] "+&v"(acc[1]), [c2] "+&v"(acc[2]), [c3] "+&v"(acc[3]), [c4] "+&v"(acc[4]), [c5] "+&v"(acc[5]), \
                 [c6] "+&v"(acc[6]), [c7] "+&v"(acc[7]), [dp] "+&v"(dp), [wa0] "+&v"(wa0), [wa1] "+&v"(wa1), [n] "+&s"(n), [vd] "=&v"(vd), [t] "=&v"(tmp), [stv] "+&v"(stv)
  if constexpr (RT == 5) asm volatile(CONS_ASM5 : CONS_OUT, [c8] "+&v"(acc[8]), [c9] "+&v"(acc[9]) : CONS_IN : CONS_CLOB);
  else asm volatile(CONS_ASM4 : CONS_OUT : CONS_IN : CONS_CLOB);
  const float sc = (vd == 0) ? scale : __builtin_nanf("");
  #pragma unroll
  for (int k = 0; k < 2 * RT; ++k)
  {
    const f32x4 r = acc[k] * sc;
    const float* ob = o + (size_t)(k >> 1) * 16 * GN;
    if (k & 1) asm volatile("global_store_dwordx4 %0, %1, off offset:64" STPOL "\n\ts_nop 1" :: "v"(ob), "v"(r) : "memory");
    else asm volatile("global_store_dwordx4 %0, %1, off" STPOL "\n\ts_nop 1" :: "v"(ob), "v"(r) : "memory");
  }
}

template <int RT>
__device__ __forceinline__ void aloader(const Args& p, u32 lds0, int lane, int b, int mt, int rot, int lr, int ntl DBGARGS) {
  constexpr int NI = 2 * RT;
  const int rank = p.rank;
  char* const myws = p.myws;
  const u32 ctl = lds0 + LDS_CTL;
  u32* cntp = (u32*)(myws + OFF_CNT + (size_t)b * 128) + 6;
  u32 c;
  asm volatile("global_load_dword %0, %1, off sc0 sc1" : "=v"(c) : "v"(cntp) : "memory");
  const char* dtab = myws + OFF_DTAB + (size_t)b * 128;
  asm volatile("global_store_dword %0, %1, %2" :: "v"((u32)(lane & 31) * 4), "v"((lane & 31) == 7 ? D_NULL : (u32)(lane & 7) | (u32)((lane & 31) + 8) << 8), "s"(dtab) : "memory");
  u32 off[10];
  #pragma unroll
  for (int q = 0; q < 10; ++q) off[q] = (u32)(8 * q + (lane >> 3)) * KS + 16 * ((lane & 7) ^ ((4 * q + (lane >> 4)) & 7)) - (q & 3) * 1024;
  const unsigned char* xl = p.x + (size_t)(rank * 512 + row0_of(mt)) * KS;
  const u32 d0 = lds0 + LDS_A;
  #pragma unroll
  for (int s = 0; s < 2; ++s) dma<NI, false>(off, xl + ((s + lr) & 3) * 128, d0 + s * PIECE);
  const int lq = lane & 7, lj = (lane >> 3) & 3, lsrc = src_of(lq < 7 ? lq : 0, rank, rot);
  const u32 tab = (ABL_NOCOMM ? (u32)((lsrc * 4 + lj) * 7 + mt) * 8192u : (u32)(lsrc * 4 + lj) * PIECE);
  const u32 or0 = (u32)lane * 16, or1 = or0 + 4096, or2 = or0 + 8192;
  asm volatile("s_waitcnt vmcnt(%1)" : "+v"(c) : "i"(NI) : "memory");
  const u32 call = rfl(c) + 1;
  const char* slot = ABL_NOCOMM ? (const char*)p.x : myws + OFF_SLOT + (size_t)(call & 1) * SLOT_BYTES + (size_t)mt * 32 * PIECE;
  u32* tmo = (u32*)(myws + OFF_TMO);
  const u32 snap = ctl + C_FLAGS + lq * 16 + lj * 4;
  const u32 t0 = (u32)__builtin_amdgcn_s_memrealtime();
  u32 mask = ABL_NOCOMM ? 0x7F7F7F7Fu : 0u;
  u32 next = 0x7Fu;
  u32 l1 = 0x40u;
  u32 dst = d0, dpa = ctl + C_DESC + 16, bad = 0;
  u32 slo = (u32)(u64)slot, shi = (u32)((u64)slot >> 32);
  u32 c_, x_, o_, nl_, nn_, vD, vF = 0;
  bar();
  #pragma unroll
  for (int s = 2; s < 4; ++s) dma<NI, false>(off, xl + ((s + lr) & 3) * 128, d0 + s * PIECE);
  waitvm<2 * NI>();
  u32 l2 = 0x20u, l3 = 0x10u;
  const u32 lead = (u32)__builtin_amdgcn_ballot_w64(((lq + 3 * lj) & 3) == ntl);
#define LEADPICK
#define SHIFTL
#define GL(V, OFF) "global_load_lds_dwordx4 " V ", s[80:81]" OFF " sc0 sc1\n\t"
#define PICK \
    "s_andn2_b32 %[c], %[mask], %[l1]\n\t" \
    "s_and_b32 %[c], %[c], %[next]\n\t" \
    LEADPICK \
    "s_ff1_i32_b32 %[x], %[c]\n\t" \
    "v_readlane_b32 %[o], %[tab], %[x]\n\t"
#define PICK2 \
    "s_lshl_b32 %[nl], 0x100, %[x]\n\t" \
    "s_lshl_b32 %[nn], 0x101, %[x]\n\t" \
    "v_lshlrev_b32 %[vD], 2, %[x]\n\t" \
    "s_add_u32 s82, %[slo], %[o]\n\t" \
    "s_addc_u32 s83, %[shi], 0\n\t"
#define NEXTDPA \
    "s_add_u32 %[dpa], %[dpa], 4\n\t" \
    "s_or_b32 %[dpa], %[dpa], 0xc0\n\t" \
    "s_bitset0_b32 %[dpa], 8\n\t"
#define NEXTDST \
    "s_add_u32 %[dst], %[dst], %[piece]\n\t" \
    "s_cmp_eq_u32 %[dst], %[dend]\n\t" \
    "s_cselect_b32 %[dst], %[d0], %[dst]\n\t" \
    "s_cmp_lg_u32 %[c], 0\n\t"
#define ARM \
    "s_mov_b64 s[80:81], s[82:83]\n\t" \
    "s_mov_b32 m0, %[dpa]\n\t" \
    "s_mov_b64 exec, 1\n\t"
#define ALOOP(TAILDMA) asm volatile( \
    "s_mov_b64 vcc, 0\n\t" \
    "s_branch 3f\n" \
    "1:\n\t" \
    "s_waitcnt vmcnt(%[n2])\n\t" \
    "s_barrier\n" \
    "2:\n\t" \
    "global_load_lds_dword %[vD], %[dtab] sc1\n\t" \
    "s_mov_b32 m0, %[dst]\n\t" \
    "s_mov_b64 exec, -1\n\t" \
    GL("%[o0]", "") \
    "s_waitcnt lgkmcnt(0)\n\t" \
    "v_cmp_le_u32 s[84:85], %[call], %[vF]\n\t" \
    SHIFTL \
    "s_mov_b32 %[l1], %[nl]\n\t" \
    "s_xor_b32 %[next], %[next], %[nn]\n\t" \
    "s_or_b32 %[mask], %[mask], s84\n\t" \
    GL("%[o0]", " offset:1024") \
    "ds_read_b32 %[vF], %[snap]\n\t" \
    NEXTDPA \
    GL("%[o0]", " offset:2048") \
    PICK \
    GL("%[o0]", " offset:3072") \
    "s_add_u32 m0, m0, 0x1000\n\t" \
    PICK2 \
    GL("%[o1]", "") \
    GL("%[o1]", " offset:1024") \
    GL("%[o1]", " offset:2048") \
    TAILDMA \
    ARM \
    "s_cbranch_vccnz 5f\n" \
    "6:\n\t" \
    "s_cbranch_scc1 1b\n\t" \
    "s_mov_b64 exec, -1\n\t" \
    "s_waitcnt vmcnt(%[n2])\n" \
    "3:\n\t"                        \
    "s_barrier\n\t" \
    "s_cmp_eq_u32 %[next], 0\n\t" \
    "s_cbranch_scc1 9f\n\t" \
    "ds_read_b32 %[vF], %[snap]\n\t" \
    "s_memrealtime s[86:87]\n\t" \
    "s_waitcnt lgkmcnt(0)\n\t" \
    "v_cmp_le_u32 s[84:85], %[call], %[vF]\n\t" \
    "s_or_b32 %[mask], %[mask], s84\n" \
    "4:\n\t" \
    PICK PICK2 ARM \
    "s_cmp_lg_u32 %[c], 0\n\t" \
    "s_cbranch_scc1 2b\n\t" \
    "s_sub_u32 s86, s86, %[t0]\n\t" \
    "s_cmp_gt_u32 s86, %[dl]\n\t" \
    "s_cbranch_scc1 8f\n\t" \
    "s_bitcmp1_b32 %[mask], 7\n\t" \
    "s_cbranch_scc1 8f\n" \
    "7:\n\t" \
    "v_mov_b32 %[vD], 28\n\t" \
    SHIFTL \
    "s_mov_b32 %[l1], 0\n\t" \
    "global_load_lds_dword %[vD], %[dtab] sc1\n\t" \
    "s_mov_b64 exec, -1\n\t" \
    NEXTDPA \
    "s_waitcnt vmcnt(%[n1])\n\t" \
    "s_cbranch_vccz 10f\n\t" \
    "s_waitcnt vmcnt(1)\n" \
    "10:\n\t" \
    "s_mov_b64 vcc, -1\n\t" \
    "s_branch 3b\n" \
    "5:\n\t"                        \
    "s_waitcnt vmcnt(%[n1])\n\t" \
    "s_mov_b64 vcc, 0\n\t" \
    "s_branch 6b\n" \
    "8:\n\t"                        \
    "s_cmp_lg_u32 %[bad], 0\n\t" \
    "s_cbranch_scc1 7b\n\t" \
    "s_mov_b32 %[bad], 1\n\t" \
    "v_mov_b32 %[vD], 0\n\t" \
    "v_mov_b32 %[vF], %[call]\n\t" \
    "global_store_dword %[vD], %[vF], %[tmo] sc1\n\t" \
    "s_mov_b64 exec, -1\n\t" \
    "s_mov_b32 %[mask], -1\n\t" \
    "s_mov_b32 %[slo], %[xlo]\n\t" \
    "s_mov_b32 %[shi], %[xhi]\n\t" \
    "s_waitcnt vmcnt(0)\n\t" \
    "s_branch 4b\n" \
    "9:\n\t" \
    "s_waitcnt lgkmcnt(0)" \
    : [mask] "+&s"(mask), [next] "+&s"(next), [l1] "+&s"(l1), [l2] "+&s"(l2), [l3] "+&s"(l3), [dst] "+&s"(dst), [dpa] "+&s"(dpa), [bad] "+&s"(bad), [slo] "+&s"(slo), [shi] "+&s"(shi), \
      [c] "=&s"(c_), [x] "=&s"(x_), [o] "=&s"(o_), [nl] "=&s"(nl_), [nn] "=&s"(nn_), [vD] "=&v"(vD), [vF] "+&v"(vF) \
    : [o0] "v"(or0), [o1] "v"(or1), [o2] "v"(or2), [tab] "v"(tab), [snap] "v"(snap), [call] "s"(call), [t0] "s"(t0), [lead] "s"(lead), \
      [d0] "s"(d0), [dend] "s"(d0 + NAS * PIECE), [dtab] "s"(dtab), [tmo] "s"(tmo), [xlo] "s"((u32)(u64)p.x), [xhi] "s"((u32)((u64)p.x >> 32)), \
      [piece] "i"(PIECE), [n2] "i"(2 * NI + 1), [n1] "i"(NI + 1), [dl] "i"((u32)DEADLINE) \
    : "memory", "scc", "vcc", "s80", "s81", "s82", "s83", "s84", "s85", "s86", "s87")
  if constexpr (NI == 10)
    ALOOP(GL("%[o1]", " offset:3072") "s_add_u32 m0, m0, 0x1000\n\t" NEXTDST GL("%[o2]", "") GL("%[o2]", " offset:1024"));
  else
    ALOOP(NEXTDST GL("%[o1]", " offset:3072"));
#undef GL
  u32 tv;
  asm volatile("global_load_dword %0, %1, %2 sc1\n\ts_waitcnt vmcnt(%3)\n\ts_barrier\n\ts_waitcnt vmcnt(0)"
               : "=&v"(tv) : "v"(0u), "s"(tmo), "i"(NI + 1) : "memory");
  if (bad || rfl(tv) == call) { *(volatile LAS u32*)(ctl + C_STAT) = 1u; asm volatile("s_waitcnt lgkmcnt(0)" ::: "memory"); }
  bar();
  if (lane == 0) __hip_atomic_store(cntp, call, __ATOMIC_RELAXED, __HIP_MEMORY_SCOPE_SYSTEM);
  STAMP(8);
}
}

__global__ __launch_bounds__(512, 2) void a2a_gemm_kernel(Args p) {
  extern __shared__ __attribute__((aligned(16))) char smem[];
  u64 pc0;
  asm volatile("s_getpc_b64 %0" : "=s"(pc0));
  const u32 lds0 = (u32)(size_t)(LAS char*)smem;
  const int b = blockIdx.x, lane = threadIdx.x & 63;
  const int wave = rfl(threadIdx.x >> 6);
  const int rank = p.rank;
  char* const myws = p.myws;

  if (__builtin_expect(b < 32, 1)) {
    const u32 sub = (u32)b >> 3, link = ((u32)b + 2 * sub) & 7u;
    const unsigned char* xs = p.peer[link].xs;
    char* dws = p.peer[link].ws;
    const u64 t0 = __builtin_amdgcn_s_memrealtime();
    const u32 ps = (u32)wave * 4 + sub, pj = (ps * 37) >> 8, pk = ps - pj * 7 + (PROT ? link : 0u), pmt = pk >= 7 ? pk - 7 : pk;
    const u32 r0 = 64 * pmt + 16 * min(pmt, 4u);
    const bool big = pmt < 4;
    const u32 o0 = (r0 + (u32)(lane >> 3)) * KS + pj * 128 + 16 * ((lane & 7) ^ (lane >> 4));
    const u32 o1 = (o0 ^ 64u) + 8 * KS;
    const u32 ol = big ? 64 * KS : 48 * KS;
    asm volatile("" :: "s"(xs), "s"(dws), "s"(myws), "s"(rank));
    if (CPF && wave == 7) {
      const u64 tp = t0 + (link >= 7u ? 250 : CPFT);
      while (__builtin_amdgcn_s_memrealtime() < tp) __builtin_amdgcn_s_sleep(8);
      const u32 po = (u32)(lane >> 2) * GN + (lane & 3) * 128;
      #pragma unroll 1
      for (int i = 0; i < 7; ++i) {
        const unsigned char* pf = p.w + (size_t)(((rank + 1 + i) & 7) * KS + 384 + sub * 32) * GN + (b & 7) * 512;
        asm volatile("global_load_dword v204, %0, %1\n\tglobal_load_dword v204, %0, %2" :: "v"(po), "s"(pf), "s"(pf + 16 * GN) : "memory", "v204");
      }
      waitvm<0>();
    }
    if (link >= 7u || wave >= 7 || ABL_NOCOMM) return;
    const u32 cnto = (u32)OFF_CNT + (u32)b * 128 + (u32)wave * 4;
    u32* cntp = (u32*)(myws + cnto);
    u32x4 v[10];
    u32 c;
    if (wave >= 2) { for (int z = 1; z < wave; ++z) __builtin_amdgcn_s_sleep(LDSLEEP); }
    asm volatile(
      "global_load_dwordx4 %0, %13, %23\n\t"
      "global_load_dwordx4 %1, %14, %23\n\t"
      "global_load_dwordx4 %2, %15, %23\n\t"
      "global_load_dwordx4 %3, %16, %23\n\t"
      "global_load_dwordx4 %4, %17, %23\n\t"
      "global_load_dwordx4 %5, %18, %23\n\t"
      "global_load_dwordx4 %6, %19, %23\n\t"
      "global_load_dwordx4 %7, %20, %23\n\t"
      "global_load_dwordx4 %8, %21, %23\n\t"
      "global_load_dwordx4 %9, %22, %23\n\t"
      "global_load_dword %10, %11, %12 sc0 sc1"
      : "=&v"(v[0]), "=&v"(v[1]), "=&v"(v[2]), "=&v"(v[3]), "=&v"(v[4]), "=&v"(v[5]), "=&v"(v[6]), "=&v"(v[7]), "=&v"(v[8]), "=&v"(v[9]), "=&v"(c)
      : "v"(cnto), "s"(myws), "v"(o0), "v"(o1), "v"(o0 + 16 * KS), "v"(o1 + 16 * KS), "v"(o0 + 32 * KS), "v"(o1 + 32 * KS),
        "v"(o0 + 48 * KS), "v"(o1 + 48 * KS), "v"(o0 + ol), "v"(o1 + ol), "s"(xs) : "memory");
    const int tl = (int)(PROT ? ps * (DELTA * 8 / 7) : (pj * 512 + r0) * DELTA / 64) - LEAD;
    u64 tgo = t0 + (u64)((u32)(TSTART + (tl > 0 ? tl : 0)) / 10);
    char* d = dws + OFF_SLOT + (size_t)((pmt * 8 + rank) * 4 + pj) * PIECE + lane * 16;
    asm volatile("s_waitcnt vmcnt(0)"
      : "+v"(v[0]), "+v"(v[1]), "+v"(v[2]), "+v"(v[3]), "+v"(v[4]), "+v"(v[5]), "+v"(v[6]), "+v"(v[7]), "+v"(v[8]), "+v"(v[9]), "+v"(c), "+s"(tgo), "+v"(d) :: "memory");
    const u32 call = rfl(c) + 1;
    d += (size_t)(call & 1) * SLOT_BYTES;
    if (ADAPT) {
      volatile LAS u64* tr = (volatile LAS u64*)(size_t)lds0;
      u64 tref;
      if (wave == 0) { tref = __builtin_amdgcn_s_memrealtime(); *tr = tref; }
      else {
        int tries = 16;
        do { const u64 x = *tr; tref = ((u64)rfl((u32)(x >> 32)) << 32) | rfl((u32)x); } while (tref + 300 - t0 > 2000ull && --tries > 0);
        if (tries == 0) tref = __builtin_amdgcn_s_memrealtime();
      }
      tgo = tref + (u64)((u32)(tl > 0 ? tl : 0) / 10);
    }
    while (__builtin_amdgcn_s_memrealtime() < tgo) __builtin_amdgcn_s_sleep(1);
    #pragma unroll
    for (int i = 0; i < 8; ++i)
      asm volatile("global_store_dwordx4 %0, %1, off sc0 sc1\n\ts_nop 1" :: "v"(d + i * 1024), "v"(v[i]) : "memory");
    if (big) {
      #pragma unroll
      for (int i = 8; i < 10; ++i)
        asm volatile("global_store_dwordx4 %0, %1, off sc0 sc1\n\ts_nop 1" :: "v"(d + i * 1024), "v"(v[i]) : "memory");
    }
    waitvm<0>();
    if (lane == 0) {
      __hip_atomic_store((u32*)(dws + OFF_FLAG + (size_t)(pmt * 8 + rank) * 128) + pj, call, __ATOMIC_RELAXED, __HIP_MEMORY_SCOPE_SYSTEM);
      __hip_atomic_store(cntp, call, __ATOMIC_RELAXED, __HIP_MEMORY_SCOPE_SYSTEM);
    }
    return;
  }

  const int gi = (b >> 3) - 4, ntl = gi / 7, mt = gi - ntl * 7;
  const int n0 = (CMAP ? (b & 7) + 8 * ntl : (b & 7) * 4 + ntl) * 128;
  const int rot = PROT ? (6 - mt + ROTN * ntl) % 7 : (ROTM * mt + ROTN * ntl) % 7;
  const int lr = LROT ? (mt + ntl) & 3 : 0;
  if (CODEPF && wave == 2) {
    const u32 co = ((u32)gi * 6 + (u32)lane) * 128;
    asm volatile("s_mov_b64 exec, 0x3f\n\tglobal_load_dword v204, %0, %1\n\ts_mov_b64 exec, -1" :: "v"(min(co, (u32)CODEPF - 128)), "s"((const char*)pc0) : "memory", "v204");
  }
  if (XPF && wave == 0) {
    const u32 sub = (u32)gi & 3u, w = (u32)gi >> 2, link = ((u32)b + 2 * sub) & 7u;
    const u32 ps = w * 4 + sub, pj = (ps * 37) >> 8, pk = ps - pj * 7 + link, pmt = pk >= 7 ? pk - 7 : pk;
    const u32 r0 = 64 * pmt + 16 * min(pmt, 4u);
    if (link < 7u) {
      asm volatile("global_load_dword v204, %0, %1" :: "v"((r0 + (u32)lane) * KS + pj * 128), "s"(p.peer[link].xs) : "memory", "v204");
      if (pmt < 4) asm volatile("global_load_dword v204, %0, %1" :: "v"((r0 + 64 + (u32)(lane & 15)) * KS + pj * 128), "s"(p.peer[link].xs) : "memory", "v204");
    }
  }
  if (GDELAY && wave >= 4) __builtin_amdgcn_s_sleep(GDELAY);
  if (wave < 4) {
    if (mt < 4) consumer<5>(p, lds0, wave, lane, mt, n0 DBGPASS); else consumer<4>(p, lds0, wave, lane, mt, n0 DBGPASS);
  } else if (wave < 6) {
    const int h = wave - 4;
    u32 off[10];
    #pragma unroll
    for (int q = 0; q < 8; ++q) off[q] = (u32)(8 * q + (lane >> 3)) * GN + 16 * ((lane & 7) ^ ((lane >> 4) | ((q & 2) << 1))) - (q & 3) * 1024;
    off[8] = off[9] = 0;
    const unsigned char* wb = p.w + (size_t)(64 * h) * GN + n0;
    const u32 d0 = lds0 + LDS_W + h * 8192u;
    const unsigned char* wl = wb + (size_t)(rank * KS) * GN;
    auto local = [&](int s) { dma<8, false>(off, wl + (size_t)(((s + lr) & 3) * 128) * GN, d0 + (3 + s) * 16384u); };
    local(0);
    dma<8, false>(off, wb + (size_t)(src_of(0, rank, rot) * KS) * GN, d0);
    local(1);
    const u32 tab = (u32)(src_of((lane & 7) < 7 ? (lane & 7) : 0, rank, rot) * KS + ((lane >> 3) & 3) * 128) * GN;
    const unsigned char* w1 = wb + (u32)__builtin_amdgcn_readlane(tab, 1);
    const unsigned char* w2 = wb + (u32)__builtin_amdgcn_readlane(tab, 2);
    const unsigned char* w3 = wb + (u32)__builtin_amdgcn_readlane(tab, 3);
    const unsigned char* w4 = wb + (u32)__builtin_amdgcn_readlane(tab, 4);
    asm volatile("" :: "s"(w1), "s"(w2), "s"(w3), "s"(w4));
    waitvm<16>(); bar();
    local(2); local(3);
    waitvm<16>(); bar();
    dma<8, false>(off, w1, d0 + 1 * 16384u);
    dma<8, false>(off, w2, d0 + 2 * 16384u);
    waitvm<24>(); bar();
    dma<8, false>(off, w3, d0 + 3 * 16384u);
    dma<8, false>(off, w4, d0 + 4 * 16384u);
    waitvm<32>(); bar();
    u32 vdp = lds0 + LDS_CTL + C_DESC + 16, vd;
    u32 d1 = 0x606u;
    u32 left = 23, q_, m_, nm_, un_ = 0;
    const unsigned char* b0 = wb + (u32)__builtin_amdgcn_readlane(tab, 5);
#define GL(V, OFF) "global_load_lds_dwordx4 " V ", s[80:81]" OFF "\n\t"
#define RDDESC \
      "ds_read_b32 %[vd], %[vdp]\n\t" \
      "v_add_u32 %[vdp], 4, %[vdp]\n\t" \
      "v_bfi_b32 %[vdp], 63, %[vdp], %[dbase]\n\t"
#define UNSAFE   \
      "s_lshr_b32 %[un], s84, 16\n\t" \
      "s_or_b32 %[un], %[un], s84\n\t" \
      "s_lshr_b32 %[q], %[un], 8\n\t" \
      "s_or_b32 %[un], %[un], %[q]\n\t" \
      "s_or_b32 %[un], %[un], s85\n\t"
#define NEXTREF   \
      "s_lshr_b32 %[m], %[d1], 8\n\t" \
      "v_readlane_b32 %[q], %[tab], %[m]\n\t" \
      "s_and_b32 %[nm], %[d1], 7\n\t" \
      "s_lshl_b64 s[86:87], s[84:85], 8\n\t" \
      "s_bitset1_b32 s86, %[nm]\n\t" \
      "s_lshl_b32 %[nm], %[nm], 14\n\t"
#define NEXTREF2 \
      "s_add_u32 %[nm], %[nm], %[d0]\n\t" \
      "s_add_u32 s82, %[wlo], %[q]\n\t" \
      "s_addc_u32 s83, %[whi], 0\n\t"
    asm volatile(
      "s_mov_b64 s[80:81], %[b0]\n\t"
      "s_mov_b32 s86, 0x04081020\n\t"
      "s_mov_b32 s87, 0x02\n\t"
      "s_add_u32 m0, %[d0], 0x14000\n\t"
      "s_branch 2f\n"
      "1:\n\t"
      "s_waitcnt vmcnt(40)\n"
      "6:\n\t"
      "s_cmp_lt_u32 %[m], 32\n\t"
      "s_barrier\n\t"
      "s_cbranch_scc0 3f\n"
      "2:\n\t"
      GL("%[o0]", "")
      "s_mov_b64 s[84:85], s[86:87]\n\t"
      RDDESC
      GL("%[o1]", " offset:1024")
      "s_sub_u32 %[left], %[left], 1\n\t"
      UNSAFE
      GL("%[o2]", " offset:2048")
      NEXTREF
      GL("%[o3]", " offset:3072")
      "s_add_u32 m0, m0, 0x1000\n\t"
      NEXTREF2
      GL("%[o4]", "")
      GL("%[o5]", " offset:1024")
      GL("%[o6]", " offset:2048")
      GL("%[o7]", " offset:3072")
      "4:\n\t"
      "s_waitcnt lgkmcnt(0)\n\t"
      "v_readfirstlane_b32 %[d1], %[vd]\n\t"
      "s_mov_b32 m0, %[nm]\n\t"
      "s_mov_b64 s[80:81], s[82:83]\n\t"
      "s_bitcmp1_b32 %[un], %[d1]\n\t"
      "s_cbranch_scc0 1b\n\t"
      "s_and_b32 %[q], %[d1], 7\n\t"
      "s_lshl_b32 %[q], 0x01010101, %[q]\n\t"
      "s_waitcnt vmcnt(32)\n\t"
      "s_and_b32 %[q], %[q], s84\n\t"
      "s_cbranch_scc0 6b\n\t"
      "s_waitcnt vmcnt(24)\n\t"
      "s_and_b32 %[q], %[q], 0xffffff\n\t"
      "s_cbranch_scc0 6b\n\t"
      "s_waitcnt vmcnt(16)\n\t"
      "s_and_b32 %[q], %[q], 0xffff\n\t"
      "s_cbranch_scc0 6b\n\t"
      "s_waitcnt vmcnt(8)\n\t"
      "s_and_b32 %[q], %[q], 0xff\n\t"
      "s_cbranch_scc0 6b\n\t"
      "s_waitcnt vmcnt(0)\n\t"
      "s_cmp_eq_u32 %[left], 0\n\t"
      "s_cbranch_scc0 6b\n\t"
      "s_branch 7f\n"
      "3:\n\t"
      RDDESC NEXTREF NEXTREF2
      "s_branch 4b\n"
      "7:"
      : [vdp] "+&v"(vdp), [vd] "=&v"(vd), [left] "+&s"(left), [d1] "+&s"(d1), [q] "=&s"(q_), [m] "=&s"(m_), [nm] "=&s"(nm_), [un] "+&s"(un_)
      : [o0] "v"(off[0]), [o1] "v"(off[1]), [o2] "v"(off[2]), [o3] "v"(off[3]), [o4] "v"(off[4]), [o5] "v"(off[5]), [o6] "v"(off[6]), [o7] "v"(off[7]),
        [tab] "v"(tab), [wlo] "s"((u32)(u64)wb), [whi] "s"((u32)((u64)wb >> 32)), [b0] "s"(b0), [d0] "s"(d0), [dbase] "s"(lds0 + LDS_CTL + C_DESC)
      : "memory", "scc", "s80", "s81", "s82", "s83", "s84", "s85", "s86", "s87");
#undef GL
    STAMP(7);
  } else if (wave == 6) {
    if (mt < 4) aloader<5>(p, lds0, lane, b, mt, rot, lr, ntl DBGPASS); else aloader<4>(p, lds0, lane, b, mt, rot, lr, ntl DBGPASS);
  } else {
    u32* cntp = (u32*)(myws + OFF_CNT + (size_t)b * 128) + 6;
    u32 c, j0 = 0;
    *(volatile LAS u32*)(lds0 + LDS_CTL + C_FLAGS + (lane & 31) * 4) = 0u;
    asm volatile("global_load_dword v205, %0, off sc0 sc1" :: "v"(cntp) : "memory", "v205");
    const u32 voff = (lane & 7) < 7 ? (u32)OFF_FLAG + (u32)(mt * 8 + src_of(lane & 7, rank, rot)) * 128 : (u32)OFF_TMO;
    const u32 dst = lds0 + LDS_CTL + C_FLAGS;
    const bool early = PFMODE == 1 || (PFMODE == 3 && (mt & 1));
    const unsigned char* pf = p.w + (size_t)(((rank + 1 + (gi >> 2)) & 7) * KS + (gi & 3) * 32) * GN + (b & 7) * 512;
    const u32 po = (u32)(lane >> 2) * GN + (lane & 3) * 128;
#define TOUCH2(J) asm volatile("global_load_dword %0, %1, %2\n\tglobal_load_dword %0, %1, %3" \
      : "+{v204}"(j0) : "v"(po), "s"(pf + (J) * 128 * GN), "s"(pf + ((J) * 128 + 16) * GN) : "memory")
#define TOUCH6 do { TOUCH2(1); if (PFN > 1) TOUCH2(2); if (PFN > 2) TOUCH2(3); } while (0)
    if (SLOWH) {
      const u32 cb = (3u - (u32)((size_t)p.w >> 7) - (u32)(b & 7) * 4) & 7u;
      if (cb < 4u) asm volatile("s_mov_b64 exec, 0xffffffff\n\tglobal_load_dword %0, %1, %2\n\ts_mov_b64 exec, -1"
                                : "+{v204}"(j0) : "v"((u32)lane * GN + cb * 128), "s"(pf) : "memory");
    }
    if (PFMODE && early) {
      __builtin_amdgcn_s_sleep(PFSLEEP);
      TOUCH6;
      asm volatile("s_waitcnt vmcnt(%1)\n\tv_readfirstlane_b32 %0, v205" : "=s"(c) : "i"(2 * PFN) : "memory", "v205");
    } else {
      asm volatile("s_waitcnt vmcnt(0)\n\tv_readfirstlane_b32 %0, v205" : "=s"(c) :: "memory", "v205");
    }
    const u32 call = c + 1;
    asm volatile("s_waitcnt lgkmcnt(0)" ::: "memory");
    if (PREPOLL && !ABL_NOCOMM)
      asm volatile("s_mov_b32 m0, %2\n\ts_mov_b64 exec, 0xff\n\tglobal_load_lds_dwordx4 %0, %1 sc0 sc1\n\ts_mov_b64 exec, -1" :: "v"(voff), "s"(myws), "s"(dst) : "memory");
    if (PFMODE && !early) { bar(); TOUCH6; }
    if (ABL_NOCOMM) { asm volatile("s_barrier\n\ts_barrier\n\ts_barrier\n\ts_barrier\n\ts_barrier\n\ts_barrier\n\ts_barrier\n\ts_barrier\n\ts_waitcnt vmcnt(0)" :: "{v204}"(j0) : "memory"); return; }
    u32 need = 0xFFu, seen = 0, credit = 0, gap = 0, t0_, t1_, vf = 0;
    asm volatile(
      "s_mov_b32 m0, %[dst]\n\t"
      "s_mov_b32 exec_hi, 0\n\t"
      "s_cmp_eq_u32 0, 0\n"
      "1:\n\t"
      "s_mov_b32 exec_lo, %[need]\n\t"
      "s_barrier\n\t"
      "s_cbranch_scc0 2f\n\t"
      "global_load_lds_dwordx4 %[voff], %[fb] sc0 sc1\n\t"
      "s_mov_b32 %[gap], 0\n"
      "2:\n\t"
      "s_mov_b32 exec_lo, -1\n\t"
      "s_waitcnt lgkmcnt(0)\n\t"
      "v_cmp_le_u32 vcc, %[call], %[vf]\n\t"
      "ds_read_b32 %[vf], %[sn]\n\t"
      "s_andn2_b32 %[t0], vcc_lo, %[seen]\n\t"
      "s_bcnt1_i32_b32 %[t0], %[t0]\n\t"
      "s_or_b32 %[seen], %[seen], vcc_lo\n\t"
      "s_add_u32 %[credit], %[credit], %[t0]\n\t"
      "s_max_u32 %[credit], %[credit], 1\n\t"
      "s_sub_u32 %[credit], %[credit], 1\n\t"
      "s_not_b32 %[t0], %[seen]\n\t"
      "s_quadmask_b32 %[need], %[t0]\n\t"
      "s_bitcmp1_b32 %[seen], 28\n\t"
      "s_cselect_b32 %[need], 0x80, %[need]\n\t"
      "s_cmp_eq_u32 %[need], 0x80\n\t"
      "s_cbranch_scc1 3f\n\t"
      "s_add_u32 %[gap], %[gap], 1\n\t"
      "s_cmp_ge_u32 %[credit], %[c1]\n\t"
      "s_cselect_b32 %[t0], %[g1], 1\n\t"
      "s_cmp_ge_u32 %[credit], %[c2]\n\t"
      "s_cselect_b32 %[t0], 99, %[t0]\n\t"
      "s_waitcnt vmcnt(24)\n\t"
      "s_cmp_ge_u32 %[gap], %[t0]\n\t"
      "s_branch 1b\n"
      "3:\n\t"
      "s_mov_b64 exec, -1\n\t"
      "s_barrier\n\ts_barrier\n\ts_barrier\n\ts_barrier\n\t"
      "s_waitcnt vmcnt(0) lgkmcnt(0)"
      : [need] "+&s"(need), [seen] "+&s"(seen), [credit] "+&s"(credit), [gap] "+&s"(gap), [t0] "=&s"(t0_), [vf] "+&v"(vf)
      : [voff] "v"(voff), [fb] "s"(myws), [dst] "s"(dst), [call] "s"(call), [sn] "v"(dst + (lane & 31) * 4), "{v204}"(j0),
        [c1] "i"(PC1), [g1] "i"(PG1), [c2] "i"(PC2)
      : "memory", "scc", "vcc");
    STAMP(5);
  }
}

extern "C" void kernel_launch(int rank, int nranks, ncclComm_t comm, hipStream_t stream,
                              void* const* d_in, const int* in_sizes, int n_in,
                              void* d_out, int out_size, const int* in_dtypes, int out_dtype,
                              void* const* d_out_all, void* const* d_ws_all, size_t ws_bytes) {
  if (nranks != NRANK || (unsigned)rank >= (unsigned)NRANK || ws_bytes < WS_NEED || n_in < 4) {
    fprintf(stderr, "unexpected configuration\n");
    return;
  }
  Args a;
  a.x = (const unsigned char*)d_in[0]; a.w = (const unsigned char*)d_in[1];
  a.sx = (const float*)d_in[2]; a.sw = (const float*)d_in[3]; a.out = (float*)d_out;
  for (int q = 0; q < NRANK; ++q) {
    const int d = (rank + 1 + q) % NRANK;
    a.peer[q].xs = a.x + (size_t)d * 512 * KS; a.peer[q].ws = (char*)d_ws_all[d];
  }
  a.myws = (char*)d_ws_all[rank]; a.rank = rank; a.pad = 0;
  HIP_CHECK(hipFuncSetAttribute((const void*)a2a_gemm_kernel, hipFuncAttributeMaxDynamicSharedMemorySize, LDS_BYTES));
  a2a_gemm_kernel<<<256, 512, LDS_BYTES, stream>>>(a);
  HIP_CHECK(hipGetLastError());
}
```

```cpp
#include <hip/hip_runtime.h>
#include <rccl/rccl.h>
#include <cstdio>

#define HIP_CHECK(expr) do { hipError_t _e = (expr); if (_e != hipSuccess) { \
    fprintf(stderr, "HIP error %s at %s:%d\n", hipGetErrorString(_e), __FILE__, __LINE__); \
    return; } } while (0)

typedef unsigned u32;
typedef unsigned long long u64;
typedef unsigned u32x4 __attribute__((ext_vector_type(4)));
#define LAS __attribute__((address_space(3)))

#define SLOTNS 163
#define TSTART 750
#define LDSLEEP 7
#define GDELAY 3
#define PC1 1
#define PC2 8
#define PFR 10u
#define PFS 6u
#define CPFT 620

namespace {
constexpr int NRANK = 8, GN = 4096, KS = 512;
constexpr int NWS = 7, NAS = 4;
constexpr size_t OFF_CNT = 0;
constexpr size_t OFF_FLAG = 512 << 10;
constexpr u32 FSTR = 4224;
constexpr size_t OFF_TMO = 112 << 10;
constexpr size_t OFF_SLOT = 1 << 20;
constexpr u32 PIECE = 10240;
constexpr size_t SLOT_BYTES = 4u << 20;
constexpr size_t WS_NEED = OFF_SLOT + 2 * SLOT_BYTES;
constexpr u64 DEADLINE = 40000000ull;

constexpr u32 LDS_W = 0, LDS_PAD = NWS * 16384, LDS_A = LDS_PAD + 4096, LDS_CTL = LDS_A + NAS * PIECE, LDS_BYTES = LDS_CTL + 256;
constexpr u32 C_FLAGS = 0  , C_STAT = 128, C_DESC = 192  ;
static_assert(OFF_FLAG + 56 * FSTR <= OFF_SLOT, "workspace map");
static_assert(LDS_BYTES <= 163840 && LDS_BYTES > 81920 && LDS_CTL % 512 == 0, "LDS map");

struct Args {
  const unsigned char* x; const unsigned char* w; const float* sx; const float* sw; float* out;
  char* myws; int rank; int pad;
  char* peer[NRANK];
};

template <int N> __device__ __forceinline__ void waitvm() { asm volatile("s_waitcnt vmcnt(%0)" :: "i"(N) : "memory"); }
__device__ __forceinline__ void bar() { asm volatile("s_barrier" ::: "memory"); }
__device__ __forceinline__ u32 rfl(u32 v) { return __builtin_amdgcn_readfirstlane(v); }

#define DMA4(A, B, C, D, BASE) \
    "global_load_lds_dwordx4 " A ", " BASE "\n\t" \
    "global_load_lds_dwordx4 " B ", " BASE " offset:1024\n\t" \
    "global_load_lds_dwordx4 " C ", " BASE " offset:2048\n\t" \
    "global_load_lds_dwordx4 " D ", " BASE " offset:3072\n\t"
#define M0NEXT "s_add_u32 m0, m0, 0x1000\n\ts_nop 0\n\t"
template <int N> __device__ __forceinline__ void dma(const u32 (&off)[10], const void* base, u32 dst) {
  if constexpr (N == 8)
    asm volatile("s_mov_b32 m0, %9\n\ts_nop 0\n\t" DMA4("%0", "%1", "%2", "%3", "%8") M0NEXT DMA4("%4", "%5", "%6", "%7", "%8")
      :: "v"(off[0]), "v"(off[1]), "v"(off[2]), "v"(off[3]), "v"(off[4]), "v"(off[5]), "v"(off[6]), "v"(off[7]),
         "s"(base), "s"(dst) : "memory", "scc");
  else
    asm volatile("s_mov_b32 m0, %11\n\ts_nop 0\n\t" DMA4("%0", "%1", "%2", "%3", "%10") M0NEXT DMA4("%4", "%5", "%6", "%7", "%10") M0NEXT
      "global_load_lds_dwordx4 %8, %10\n\tglobal_load_lds_dwordx4 %9, %10 offset:1024"
      :: "v"(off[0]), "v"(off[1]), "v"(off[2]), "v"(off[3]), "v"(off[4]), "v"(off[5]), "v"(off[6]), "v"(off[7]), "v"(off[8]), "v"(off[9]),
         "s"(base), "s"(dst) : "memory", "scc");
}
__device__ __forceinline__ void touch(u32 voff, const void* base) { asm volatile("global_load_dword v204, %0, %1" :: "v"(voff), "s"(base) : "memory", "v204"); }
__device__ __forceinline__ void touch_w(const unsigned char* w, u32 rank, u32 i, int lane) {
  const u32 k = (i * 147) >> 10, src = (rank + 1 + i - 7 * k) & 7u;
  touch((u32)(lane >> 2) * GN + (lane & 3) * 128, w + (size_t)(src * KS + 128 + k * 16) * GN);
}

__device__ __forceinline__ int src_of(int q, int rank, int rot) { int i = q + rot; if (i >= 7) i -= 7; return (rank + 1 + i) & 7; }
__device__ __forceinline__ int row0_of(int mt) { return mt < 4 ? 80 * mt : 64 * mt + 64; }

#define CONS_ASM4 \
    "\tv_mov_b64 v[20:21], 0\n" \
    "\tv_mov_b64 v[22:23], 0\n" \
    "\tv_mov_b64 v[24:25], 0\n" \
    "\tv_mov_b64 v[26:27], 0\n" \
    "\tv_mov_b64 v[28:29], 0\n" \
    "\tv_mov_b64 v[30:31], 0\n" \
    "\tv_mov_b64 v[32:33], 0\n" \
    "\tv_mov_b64 v[34:35], 0\n" \
    "\tv_mov_b64 v[36:37], 0\n" \
    "\tv_mov_b64 v[38:39], 0\n" \
    "\tv_mov_b64 v[40:41], 0\n" \
    "\tv_mov_b64 v[42:43], 0\n" \
    "\tv_mov_b64 v[44:45], 0\n" \
    "\tv_mov_b64 v[46:47], 0\n" \
    "\tv_mov_b64 v[48:49], 0\n" \
    "\tv_mov_b64 v[50:51], 0\n" \
    "\ts_mov_b64 s[90:91], %[ob]\n" \
    "\ts_waitcnt lgkmcnt(0)\n" \
    "\ts_barrier\n" \
    "\tds_read_b32 %[vd], %[dp]\n" \
    "\tds_read_b64_tr_b8 v[64:65], %[wa0] offset:0\n" \
    "\tds_read_b64_tr_b8 v[66:67], %[wa0] offset:1024\n" \
    "\tds_read_b64_tr_b8 v[68:69], %[wa0] offset:8192\n" \
    "\tds_read_b64_tr_b8 v[70:71], %[wa0] offset:9216\n" \
    "\tds_read_b64_tr_b8 v[72:73], %[wa1] offset:0\n" \
    "\tds_read_b64_tr_b8 v[74:75], %[wa1] offset:1024\n" \
    "\tds_read_b64_tr_b8 v[76:77], %[wa1] offset:8192\n" \
    "\tds_read_b64_tr_b8 v[78:79], %[wa1] offset:9216\n" \
    "\ts_waitcnt lgkmcnt(8)\n" \
    "\tds_read_b128 v[80:83], %[ao0] offset:0\n" \
    "\tds_read_b128 v[84:87], %[ao1] offset:0\n" \
    "\tds_read_b128 v[88:91], %[ao0] offset:2048\n" \
    "\tds_read_b128 v[92:95], %[ao1] offset:2048\n" \
    "\tds_read_b128 v[96:99], %[ao0] offset:4096\n" \
    "\tds_read_b128 v[100:103], %[ao1] offset:4096\n" \
    "\tds_read_b128 v[104:107], %[ao0] offset:6144\n" \
    "\tds_read_b128 v[108:111], %[ao1] offset:6144\n" \
    "\tv_cmp_eq_u32 vcc, 0x2007, %[vd]\n" \
    "\tv_and_b32 %[t], 7, %[vd]\n" \
    "\tv_lshl_add_u32 %[wa0], %[t], 14, %[bo0]\n" \
    "\tv_lshl_add_u32 %[wa1], %[t], 14, %[bo1]\n" \
    "\tv_add_u32 %[dp], 4, %[dp]\n" \
    "\tv_bfi_b32 %[dp], 63, %[dp], %[dbase]\n" \
    "3:\n" \
    "\ts_cbranch_vccnz 10f\n" \
    "20:\n" \
    "\ts_waitcnt lgkmcnt(0)\n" \
    "\ts_barrier\n" \
    "\tv_mfma_f32_16x16x128_f8f6f4 v[20:23], v[64:71], v[80:87], v[20:23] cbsz:1 blgp:1\n" \
    "\tds_read_b64_tr_b8 v[120:121], %[wa0] offset:0\n" \
    "\tds_read_b64_tr_b8 v[122:123], %[wa0] offset:1024\n" \
    "\tds_read_b64_tr_b8 v[124:125], %[wa0] offset:8192\n" \
    "\tds_read_b64_tr_b8 v[126:127], %[wa0] offset:9216\n" \
    "\tv_mfma_f32_16x16x128_f8f6f4 v[24:27], v[72:79], v[80:87], v[24:27] cbsz:1 blgp:1\n" \
    "\tds_read_b64_tr_b8 v[128:129], %[wa1] offset:0\n" \
    "\tds_read_b64_tr_b8 v[130:131], %[wa1] offset:1024\n" \
    "\tds_read_b64_tr_b8 v[132:133], %[wa1] offset:8192\n" \
    "\tds_read_b64_tr_b8 v[134:135], %[wa1] offset:9216\n" \
    "\tv_mfma_f32_16x16x128_f8f6f4 v[28:31], v[64:71], v[88:95], v[28:31] cbsz:1 blgp:1\n" \
    "\tds_read_b32 %[vd], %[dp]\n" \
    "\tds_read_b128 v[136:139], %[ao0] offset:10240\n" \
    "\tds_read_b128 v[140:143], %[ao1] offset:10240\n" \
    "\tv_mfma_f32_16x16x128_f8f6f4 v[32:35], v[72:79], v[88:95], v[32:35] cbsz:1 blgp:1\n" \
    "\tds_read_b128 v[144:147], %[ao0] offset:12288\n" \
    "\tds_read_b128 v[148:151], %[ao1] offset:12288\n" \
    "\tv_mfma_f32_16x16x128_f8f6f4 v[36:39], v[64:71], v[96:103], v[36:39] cbsz:1 blgp:1\n" \
    "\tds_read_b128 v[152:155], %[ao0] offset:14336\n" \
    "\tds_read_b128 v[156:159], %[ao1] offset:14336\n" \
    "\tv_mfma_f32_16x16x128_f8f6f4 v[40:43], v[72:79], v[96:103], v[40:43] cbsz:1 blgp:1\n" \
    "\tds_read_b128 v[160:163], %[ao0] offset:16384\n" \
    "\tds_read_b128 v[164:167], %[ao1] offset:16384\n" \
    "\ts_waitcnt lgkmcnt(8)\n" \
    "\tv_and_b32 %[t], 7, %[vd]\n" \
    "\tv_cmp_eq_u32 vcc, 0x2007, %[vd]\n" \
    "\tv_mfma_f32_16x16x128_f8f6f4 v[44:47], v[64:71], v[104:111], v[44:47] cbsz:1 blgp:1\n" \
    "\tv_lshl_add_u32 %[wa0], %[t], 14, %[bo0]\n" \
    "\tv_lshl_add_u32 %[wa1], %[t], 14, %[bo1]\n" \
    "\tv_add_u32 %[dp], 4, %[dp]\n" \
    "\tv_bfi_b32 %[dp], 63, %[dp], %[dbase]\n" \
    "\tv_mfma_f32_16x16x128_f8f6f4 v[48:51], v[72:79], v[104:111], v[48:51] cbsz:1 blgp:1\n" \
    "\ts_cbranch_vccnz 11f\n" \
    "21:\n" \
    "\ts_waitcnt lgkmcnt(0)\n" \
    "\ts_barrier\n" \
    "\tv_mfma_f32_16x16x128_f8f6f4 v[20:23], v[120:127], v[136:143], v[20:23] cbsz:1 blgp:1\n" \
    "\tds_read_b64_tr_b8 v[64:65], %[wa0] offset:0\n" \
    "\tds_read_b64_tr_b8 v[66:67], %[wa0] offset:1024\n" \
    "\tds_read_b64_tr_b8 v[68:69], %[wa0] offset:8192\n" \
    "\tds_read_b64_tr_b8 v[70:71], %[wa0] offset:9216\n" \
    "\tv_mfma_f32_16x16x128_f8f6f4 v[24:27], v[128:135], v[136:143], v[24:27] cbsz:1 blgp:1\n" \
    "\tds_read_b64_tr_b8 v[72:73], %[wa1] offset:0\n" \
    "\tds_read_b64_tr_b8 v[74:75], %[wa1] offset:1024\n" \
    "\tds_read_b64_tr_b8 v[76:77], %[wa1] offset:8192\n" \
    "\tds_read_b64_tr_b8 v[78:79], %[wa1] offset:9216\n" \
    "\tv_mfma_f32_16x16x128_f8f6f4 v[28:31], v[120:127], v[144:151], v[28:31] cbsz:1 blgp:1\n" \
    "\tds_read_b32 %[vd], %[dp]\n" \
    "\tds_read_b128 v[80:83], %[ao0] offset:20480\n" \
    "\tds_read_b128 v[84:87], %[ao1] offset:20480\n" \
    "\tv_mfma_f32_16x16x128_f8f6f4 v[32:35], v[128:135], v[144:151], v[32:35] cbsz:1 blgp:1\n" \
    "\tds_read_b128 v[88:91], %[ao0] offset:22528\n" \
    "\tds_read_b128 v[92:95], %[ao1] offset:22528\n" \
    "\tv_mfma_f32_16x16x128_f8f6f4 v[36:39], v[120:127], v[152:159], v[36:39] cbsz:1 blgp:1\n" \
    "\tds_read_b128 v[96:99], %[ao0] offset:24576\n" \
    "\tds_read_b128 v[100:103], %[ao1] offset:24576\n" \
    "\tv_mfma_f32_16x16x128_f8f6f4 v[40:43], v[128:135], v[152:159], v[40:43] cbsz:1 blgp:1\n" \
    "\tds_read_b128 v[104:107], %[ao0] offset:26624\n" \
    "\tds_read_b128 v[108:111], %[ao1] offset:26624\n" \
    "\ts_waitcnt lgkmcnt(8)\n" \
    "\tv_and_b32 %[t], 7, %[vd]\n" \
    "\tv_cmp_eq_u32 vcc, 0x2007, %[vd]\n" \
    "\tv_mfma_f32_16x16x128_f8f6f4 v[44:47], v[120:127], v[160:167], v[44:47] cbsz:1 blgp:1\n" \
    "\tv_lshl_add_u32 %[wa0], %[t], 14, %[bo0]\n" \
    "\tv_lshl_add_u32 %[wa1], %[t], 14, %[bo1]\n" \
    "\tv_add_u32 %[dp], 4, %[dp]\n" \
    "\tv_bfi_b32 %[dp], 63, %[dp], %[dbase]\n" \
    "\tv_mfma_f32_16x16x128_f8f6f4 v[48:51], v[128:135], v[160:167], v[48:51] cbsz:1 blgp:1\n" \
    "\ts_cbranch_vccnz 12f\n" \
    "22:\n" \
    "\ts_waitcnt lgkmcnt(0)\n" \
    "\ts_barrier\n" \
    "\tv_mfma_f32_16x16x128_f8f6f4 v[20:23], v[64:71], v[80:87], v[20:23] cbsz:1 blgp:1\n" \
    "\tds_read_b64_tr_b8 v[120:121], %[wa0] offset:0\n" \
    "\tds_read_b64_tr_b8 v[122:123], %[wa0] offset:1024\n" \
    "\tds_read_b64_tr_b8 v[124:125], %[wa0] offset:8192\n" \
    "\tds_read_b64_tr_b8 v[126:127], %[wa0] offset:9216\n" \
    "\tv_mfma_f32_16x16x128_f8f6f4 v[24:27], v[72:79], v[80:87], v[24:27] cbsz:1 blgp:1\n" \
    "\tds_read_b64_tr_b8 v[128:129], %[wa1] offset:0\n" \
    "\tds_read_b64_tr_b8 v[130:131], %[wa1] offset:1024\n" \
    "\tds_read_b64_tr_b8 v[132:133], %[wa1] offset:8192\n" \
    "\tds_read_b64_tr_b8 v[134:135], %[wa1] offset:9216\n" \
    "\tv_mfma_f32_16x16x128_f8f6f4 v[28:31], v[64:71], v[88:95], v[28:31] cbsz:1 blgp:1\n" \
    "\tds_read_b32 %[vd], %[dp]\n" \
    "\tds_read_b128 v[136:139], %[ao0] offset:30720\n" \
    "\tds_read_b128 v[140:143], %[ao1] offset:30720\n" \
    "\tv_mfma_f32_16x16x128_f8f6f4 v[32:35], v[72:79], v[88:95], v[32:35] cbsz:1 blgp:1\n" \
    "\tds_read_b128 v[144:147], %[ao0] offset:32768\n" \
    "\tds_read_b128 v[148:151], %[ao1] offset:32768\n" \
    "\tv_mfma_f32_16x16x128_f8f6f4 v[36:39], v[64:71], v[96:103], v[36:39] cbsz:1 blgp:1\n" \
    "\tds_read_b128 v[152:155], %[ao0] offset:34816\n" \
    "\tds_read_b128 v[156:159], %[ao1] offset:34816\n" \
    "\tv_mfma_f32_16x16x128_f8f6f4 v[40:43], v[72:79], v[96:103], v[40:43] cbsz:1 blgp:1\n" \
    "\tds_read_b128 v[160:163], %[ao0] offset:36864\n" \
    "\tds_read_b128 v[164:167], %[ao1] offset:36864\n" \
    "\ts_waitcnt lgkmcnt(8)\n" \
    "\tv_and_b32 %[t], 7, %[vd]\n" \
    "\tv_cmp_eq_u32 vcc, 0x2007, %[vd]\n" \
    "\tv_mfma_f32_16x16x128_f8f6f4 v[44:47], v[64:71], v[104:111], v[44:47] cbsz:1 blgp:1\n" \
    "\tv_lshl_add_u32 %[wa0], %[t], 14, %[bo0]\n" \
    "\tv_lshl_add_u32 %[wa1], %[t], 14, %[bo1]\n" \
    "\tv_add_u32 %[dp], 4, %[dp]\n" \
    "\tv_bfi_b32 %[dp], 63, %[dp], %[dbase]\n" \
    "\ts_cmp_eq_u32 %[n], 1\n" \
    "\tv_mfma_f32_16x16x128_f8f6f4 v[48:51], v[72:79], v[104:111], v[48:51] cbsz:1 blgp:1\n" \
    "\ts_cbranch_scc1 5f\n" \
    "\ts_cbranch_vccnz 13f\n" \
    "23:\n" \
    "\ts_waitcnt lgkmcnt(0)\n" \
    "\ts_barrier\n" \
    "\tv_mfma_f32_16x16x128_f8f6f4 v[20:23], v[120:127], v[136:143], v[20:23] cbsz:1 blgp:1\n" \
    "\tv_readfirstlane_b32 s92, %[t]\n" \
    "\tds_read_b64_tr_b8 v[64:65], %[wa0] offset:0\n" \
    "\tds_read_b64_tr_b8 v[66:67], %[wa0] offset:1024\n" \
    "\tds_read_b64_tr_b8 v[68:69], %[wa0] offset:8192\n" \
    "\tds_read_b64_tr_b8 v[70:71], %[wa0] offset:9216\n" \
    "\tv_mfma_f32_16x16x128_f8f6f4 v[24:27], v[128:135], v[136:143], v[24:27] cbsz:1 blgp:1\n" \
    "\tds_read_b64_tr_b8 v[72:73], %[wa1] offset:0\n" \
    "\tds_read_b64_tr_b8 v[74:75], %[wa1] offset:1024\n" \
    "\tds_read_b64_tr_b8 v[76:77], %[wa1] offset:8192\n" \
    "\tds_read_b64_tr_b8 v[78:79], %[wa1] offset:9216\n" \
    "\tv_mfma_f32_16x16x128_f8f6f4 v[28:31], v[120:127], v[144:151], v[28:31] cbsz:1 blgp:1\n" \
    "\tds_read_b32 %[vd], %[dp]\n" \
    "\tds_read_b128 v[80:83], %[ao0] offset:0\n" \
    "\tds_read_b128 v[84:87], %[ao1] offset:0\n" \
    "\tv_mfma_f32_16x16x128_f8f6f4 v[32:35], v[128:135], v[144:151], v[32:35] cbsz:1 blgp:1\n" \
    "\tds_read_b128 v[88:91], %[ao0] offset:2048\n" \
    "\tds_read_b128 v[92:95], %[ao1] offset:2048\n" \
    "\tv_mfma_f32_16x16x128_f8f6f4 v[36:39], v[120:127], v[152:159], v[36:39] cbsz:1 blgp:1\n" \
    "\tds_read_b128 v[96:99], %[ao0] offset:4096\n" \
    "\tds_read_b128 v[100:103], %[ao1] offset:4096\n" \
    "\tv_mfma_f32_16x16x128_f8f6f4 v[40:43], v[128:135], v[152:159], v[40:43] cbsz:1 blgp:1\n" \
    "\tds_read_b128 v[104:107], %[ao0] offset:6144\n" \
    "\tds_read_b128 v[108:111], %[ao1] offset:6144\n" \
    "\ts_waitcnt lgkmcnt(8)\n" \
    "\tv_and_b32 %[t], 7, %[vd]\n" \
    "\tv_cmp_eq_u32 vcc, 0x2007, %[vd]\n" \
    "\tv_mfma_f32_16x16x128_f8f6f4 v[44:47], v[120:127], v[160:167], v[44:47] cbsz:1 blgp:1\n" \
    "\tv_lshl_add_u32 %[wa0], %[t], 14, %[bo0]\n" \
    "\tv_lshl_add_u32 %[wa1], %[t], 14, %[bo1]\n" \
    "\tv_add_u32 %[dp], 4, %[dp]\n" \
    "\tv_bfi_b32 %[dp], 63, %[dp], %[dbase]\n" \
    "\ts_sub_u32 %[n], %[n], 1\n" \
    "\ts_cmp_lg_u32 %[n], 2\n" \
    "\tv_mfma_f32_16x16x128_f8f6f4 v[48:51], v[128:135], v[160:167], v[48:51] cbsz:1 blgp:1\n" \
    "\ts_cbranch_scc1 3b\n" \
    "\ts_cbranch_vccnz 14f\n" \
    "24:\n" \
    "\tv_cmp_ne_u32 vcc, 0x200f, %[vd]\n" \
    "\ts_lshl_b32 s93, s92, 14\n" \
    "\ts_cbranch_vccnz 20b\n" \
    "\ts_add_u32 s93, s93, 0x4000\n" \
    "\ts_cmp_eq_u32 s93, 0x1c000\n" \
    "\ts_cselect_b32 s93, 0, s93\n" \
    "\ts_mov_b32 s94, s93\n" \
    "\tv_add_u32 %[wa0], s93, %[bo0]\n" \
    "\tv_add_u32 %[wa1], s93, %[bo1]\n" \
    "\ts_waitcnt lgkmcnt(0)\n" \
    "\ts_barrier\n" \
    "\tv_mfma_f32_16x16x128_f8f6f4 v[20:23], v[64:71], v[80:87], v[20:23] cbsz:1 blgp:1\n" \
    "\tds_read_b64_tr_b8 v[120:121], %[wa0] offset:0\n" \
    "\tds_read_b64_tr_b8 v[122:123], %[wa0] offset:1024\n" \
    "\tds_read_b64_tr_b8 v[124:125], %[wa0] offset:8192\n" \
    "\tds_read_b64_tr_b8 v[126:127], %[wa0] offset:9216\n" \
    "\ts_add_u32 s93, s93, 0x4000\n" \
    "\ts_cmp_eq_u32 s93, 0x1c000\n" \
    "\ts_cselect_b32 s93, 0, s93\n" \
    "\ts_mov_b32 s95, s93\n" \
    "\tv_mfma_f32_16x16x128_f8f6f4 v[24:27], v[72:79], v[80:87], v[24:27] cbsz:1 blgp:1\n" \
    "\tds_read_b64_tr_b8 v[128:129], %[wa1] offset:0\n" \
    "\tds_read_b64_tr_b8 v[130:131], %[wa1] offset:1024\n" \
    "\tds_read_b64_tr_b8 v[132:133], %[wa1] offset:8192\n" \
    "\tds_read_b64_tr_b8 v[134:135], %[wa1] offset:9216\n" \
    "\tv_add_u32 %[wa0], s93, %[bo0]\n" \
    "\tv_add_u32 %[wa1], s93, %[bo1]\n" \
    "\tv_mfma_f32_16x16x128_f8f6f4 v[28:31], v[64:71], v[88:95], v[28:31] cbsz:1 blgp:1\n" \
    "\tds_read_b64_tr_b8 v[136:137], %[wa0] offset:0\n" \
    "\tds_read_b64_tr_b8 v[138:139], %[wa0] offset:1024\n" \
    "\tds_read_b64_tr_b8 v[140:141], %[wa0] offset:8192\n" \
    "\tds_read_b64_tr_b8 v[142:143], %[wa0] offset:9216\n" \
    "\ts_add_u32 s93, s93, 0x4000\n" \
    "\ts_cmp_eq_u32 s93, 0x1c000\n" \
    "\ts_cselect_b32 s93, 0, s93\n" \
    "\ts_mov_b32 s96, s93\n" \
    "\tv_mfma_f32_16x16x128_f8f6f4 v[32:35], v[72:79], v[88:95], v[32:35] cbsz:1 blgp:1\n" \
    "\tds_read_b64_tr_b8 v[144:145], %[wa1] offset:0\n" \
    "\tds_read_b64_tr_b8 v[146:147], %[wa1] offset:1024\n" \
    "\tds_read_b64_tr_b8 v[148:149], %[wa1] offset:8192\n" \
    "\tds_read_b64_tr_b8 v[150:151], %[wa1] offset:9216\n" \
    "\tv_add_u32 %[wa0], s93, %[bo0]\n" \
    "\tv_add_u32 %[wa1], s93, %[bo1]\n" \
    "\tv_mfma_f32_16x16x128_f8f6f4 v[36:39], v[64:71], v[96:103], v[36:39] cbsz:1 blgp:1\n" \
    "\tds_read_b64_tr_b8 v[152:153], %[wa0] offset:0\n" \
    "\tds_read_b64_tr_b8 v[154:155], %[wa0] offset:1024\n" \
    "\tds_read_b64_tr_b8 v[156:157], %[wa0] offset:8192\n" \
    "\tds_read_b64_tr_b8 v[158:159], %[wa0] offset:9216\n" \
    "\ts_add_u32 s93, s93, 0x4000\n" \
    "\ts_cmp_eq_u32 s93, 0x1c000\n" \
    "\ts_cselect_b32 s93, 0, s93\n" \
    "\tv_mfma_f32_16x16x128_f8f6f4 v[40:43], v[72:79], v[96:103], v[40:43] cbsz:1 blgp:1\n" \
    "\tds_read_b64_tr_b8 v[160:161], %[wa1] offset:0\n" \
    "\tds_read_b64_tr_b8 v[162:163], %[wa1] offset:1024\n" \
    "\tds_read_b64_tr_b8 v[164:165], %[wa1] offset:8192\n" \
    "\tds_read_b64_tr_b8 v[166:167], %[wa1] offset:9216\n" \
    "\tv_add_u32 %[wa0], s93, %[bo0]\n" \
    "\tv_add_u32 %[wa1], s93, %[bo1]\n" \
    "\tv_add_u32 %[ew0], s96, %[ew0]\n" \
    "\tv_add_u32 %[ew1], s96, %[ew1]\n" \
    "\tv_add_u32 %[er], s96, %[er]\n" \
    "\tv_subrev_u32 %[ao0], 4096, %[ao0]\n" \
    "\tv_subrev_u32 %[ao1], 4096, %[ao1]\n" \
    "\ts_waitcnt lgkmcnt(0)\n" \
    "\ts_barrier\n" \
    "\tds_read_b128 v[206:209], %[ao0] offset:14336\n" \
    "\tds_read_b128 v[210:213], %[ao1] offset:14336\n" \
    "\tds_read_b128 v[214:217], %[ao0] offset:16384\n" \
    "\tds_read_b128 v[218:221], %[ao1] offset:16384\n" \
    "\tds_read_b128 v[222:225], %[ao0] offset:18432\n" \
    "\tds_read_b128 v[226:229], %[ao1] offset:18432\n" \
    "\tv_mfma_f32_16x16x128_f8f6f4 v[44:47], v[64:71], v[104:111], v[44:47] cbsz:1 blgp:1\n" \
    "\tds_read_b64_tr_b8 v[168:169], %[wa0] offset:0\n" \
    "\tds_read_b64_tr_b8 v[170:171], %[wa0] offset:1024\n" \
    "\tds_read_b64_tr_b8 v[172:173], %[wa0] offset:8192\n" \
    "\tds_read_b64_tr_b8 v[174:175], %[wa0] offset:9216\n" \
    "\tv_mfma_f32_16x16x128_f8f6f4 v[48:51], v[72:79], v[104:111], v[48:51] cbsz:1 blgp:1\n" \
    "\tds_read_b64_tr_b8 v[176:177], %[wa1] offset:0\n" \
    "\tds_read_b64_tr_b8 v[178:179], %[wa1] offset:1024\n" \
    "\tds_read_b64_tr_b8 v[180:181], %[wa1] offset:8192\n" \
    "\tds_read_b64_tr_b8 v[182:183], %[wa1] offset:9216\n" \
    "\ts_add_u32 s93, s93, 0x4000\n" \
    "\ts_cmp_eq_u32 s93, 0x1c000\n" \
    "\ts_cselect_b32 s93, 0, s93\n" \
    "\tv_add_u32 %[wa0], s93, %[bo0]\n" \
    "\tv_add_u32 %[wa1], s93, %[bo1]\n" \
    "\ts_waitcnt lgkmcnt(12)\n" \
    "\tv_mfma_f32_16x16x128_f8f6f4 v[20:23], v[120:127], v[206:213], v[20:23] cbsz:1 blgp:1\n" \
    "\tds_read_b128 v[96:99], %[ao0] offset:20480\n" \
    "\tds_read_b128 v[100:103], %[ao1] offset:20480\n" \
    "\tds_read_b64_tr_b8 v[184:185], %[wa0] offset:0\n" \
    "\tds_read_b64_tr_b8 v[186:187], %[wa0] offset:1024\n" \
    "\tds_read_b64_tr_b8 v[188:189], %[wa0] offset:8192\n" \
    "\tds_read_b64_tr_b8 v[190:191], %[wa0] offset:9216\n" \
    "\tv_mfma_f32_16x16x128_f8f6f4 v[24:27], v[128:135], v[206:213], v[24:27] cbsz:1 blgp:1\n" \
    "\tds_read_b64_tr_b8 v[192:193], %[wa1] offset:0\n" \
    "\tds_read_b64_tr_b8 v[194:195], %[wa1] offset:1024\n" \
    "\tds_read_b64_tr_b8 v[196:197], %[wa1] offset:8192\n" \
    "\tds_read_b64_tr_b8 v[198:199], %[wa1] offset:9216\n" \
    "\ts_waitcnt lgkmcnt(14)\n" \
    "\tv_mfma_f32_16x16x128_f8f6f4 v[20:23], v[136:143], v[214:221], v[20:23] cbsz:1 blgp:1\n" \
    "\tds_read_b128 v[104:107], %[ao0] offset:22528\n" \
    "\tds_read_b128 v[108:111], %[ao1] offset:22528\n" \
    "\tv_mfma_f32_16x16x128_f8f6f4 v[24:27], v[144:151], v[214:221], v[24:27] cbsz:1 blgp:1\n" \
    "\ts_add_u32 s93, s93, 0x4000\n" \
    "\ts_cmp_eq_u32 s93, 0x1c000\n" \
    "\ts_cselect_b32 s93, 0, s93\n" \
    "\tv_add_u32 %[wa0], s93, %[bo0]\n" \
    "\tv_add_u32 %[wa1], s93, %[bo1]\n" \
    "\ts_waitcnt lgkmcnt(14)\n" \
    "\tv_mfma_f32_16x16x128_f8f6f4 v[20:23], v[152:159], v[222:229], v[20:23] cbsz:1 blgp:1\n" \
    "\tds_read_b128 v[112:115], %[ao0] offset:0\n" \
    "\tds_read_b128 v[116:119], %[ao1] offset:0\n" \
    "\tv_mfma_f32_16x16x128_f8f6f4 v[24:27], v[160:167], v[222:229], v[24:27] cbsz:1 blgp:1\n" \
    "\ts_barrier\n" \
    "\ts_waitcnt lgkmcnt(14)\n" \
    "\ts_waitcnt lgkmcnt(12)\n" \
    "\tv_mfma_f32_16x16x128_f8f6f4 v[20:23], v[168:175], v[96:103], v[20:23] cbsz:1 blgp:1\n" \
    "\tds_read_b128 v[206:209], %[ao0] offset:2048\n" \
    "\tds_read_b128 v[210:213], %[ao1] offset:2048\n" \
    "\tds_read_b64_tr_b8 v[64:65], %[wa0] offset:0\n" \
    "\tds_read_b64_tr_b8 v[66:67], %[wa0] offset:1024\n" \
    "\tds_read_b64_tr_b8 v[68:69], %[wa0] offset:8192\n" \
    "\tds_read_b64_tr_b8 v[70:71], %[wa0] offset:9216\n" \
    "\tv_mfma_f32_16x16x128_f8f6f4 v[24:27], v[176:183], v[96:103], v[24:27] cbsz:1 blgp:1\n" \
    "\tds_read_b64_tr_b8 v[72:73], %[wa1] offset:0\n" \
    "\tds_read_b64_tr_b8 v[74:75], %[wa1] offset:1024\n" \
    "\tds_read_b64_tr_b8 v[76:77], %[wa1] offset:8192\n" \
    "\tds_read_b64_tr_b8 v[78:79], %[wa1] offset:9216\n" \
    "\ts_add_u32 s93, s93, 0x4000\n" \
    "\ts_cmp_eq_u32 s93, 0x1c000\n" \
    "\ts_cselect_b32 s93, 0, s93\n" \
    "\tv_add_u32 %[wa0], s93, %[bo0]\n" \
    "\tv_add_u32 %[wa1], s93, %[bo1]\n" \
    "\ts_waitcnt lgkmcnt(14)\n" \
    "\ts_waitcnt lgkmcnt(12)\n" \
    "\tv_mfma_f32_16x16x128_f8f6f4 v[20:23], v[184:191], v[104:111], v[20:23] cbsz:1 blgp:1\n" \
    "\tds_read_b128 v[214:217], %[ao0] offset:24576\n" \
    "\tds_read_b128 v[218:221], %[ao1] offset:24576\n" \
    "\tds_read_b64_tr_b8 v[80:81], %[wa0] offset:0\n" \
    "\tds_read_b64_tr_b8 v[82:83], %[wa0] offset:1024\n" \
    "\tds_read_b64_tr_b8 v[84:85], %[wa0] offset:8192\n" \
    "\tds_read_b64_tr_b8 v[86:87], %[wa0] offset:9216\n" \
    "\tv_mfma_f32_16x16x128_f8f6f4 v[24:27], v[192:199], v[104:111], v[24:27] cbsz:1 blgp:1\n" \
    "\tds_read_b64_tr_b8 v[88:89], %[wa1] offset:0\n" \
    "\tds_read_b64_tr_b8 v[90:91], %[wa1] offset:1024\n" \
    "\tds_read_b64_tr_b8 v[92:93], %[wa1] offset:8192\n" \
    "\tds_read_b64_tr_b8 v[94:95], %[wa1] offset:9216\n" \
    "\ts_waitcnt lgkmcnt(10)\n" \
    "\ts_waitcnt lgkmcnt(14)\n" \
    "\tv_mfma_f32_16x16x128_f8f6f4 v[20:23], v[64:71], v[112:119], v[20:23] cbsz:1 blgp:1\n" \
    "\tds_read_b128 v[222:225], %[ao0] offset:26624\n" \
    "\tds_read_b128 v[226:229], %[ao1] offset:26624\n" \
    "\tv_mfma_f32_16x16x128_f8f6f4 v[24:27], v[72:79], v[112:119], v[24:27] cbsz:1 blgp:1\n" \
    "\ts_barrier\n" \
    "\tds_read_b32 %[vd], %[stat]\n" \
    "\ts_waitcnt lgkmcnt(3)\n" \
    "\ts_waitcnt lgkmcnt(14)\n" \
    "\tv_mfma_f32_16x16x128_f8f6f4 v[20:23], v[80:87], v[206:213], v[20:23] cbsz:1 blgp:1\n" \
    "\tds_read_b128 v[96:99], %[ao0] offset:28672\n" \
    "\tds_read_b128 v[100:103], %[ao1] offset:28672\n" \
    "\tv_mfma_f32_16x16x128_f8f6f4 v[24:27], v[88:95], v[206:213], v[24:27] cbsz:1 blgp:1\n" \
    "\ts_waitcnt lgkmcnt(13)\n" \
    "\tv_mfma_f32_16x16x128_f8f6f4 v[28:31], v[120:127], v[214:221], v[28:31] cbsz:1 blgp:1\n" \
    "\tds_read_b128 v[104:107], %[ao0] offset:30720\n" \
    "\tds_read_b128 v[108:111], %[ao1] offset:30720\n" \
    "\ts_waitcnt lgkmcnt(4)\n" \
    "\tv_cmp_eq_u32 vcc, 0, %[vd]\n" \
    "\tv_mov_b32 v60, 0x7fc00000\n" \
    "\tv_mfma_f32_16x16x128_f8f6f4 v[32:35], v[128:135], v[214:221], v[32:35] cbsz:1 blgp:1\n" \
    "\tv_cndmask_b32 v60, v60, %[sc], vcc\n" \
    "\tv_mov_b32 v61, v60\n" \
    "\ts_waitcnt lgkmcnt(5)\n" \
    "\tv_mfma_f32_16x16x128_f8f6f4 v[28:31], v[136:143], v[222:229], v[28:31] cbsz:1 blgp:1\n" \
    "\tds_read_b128 v[112:115], %[ao0] offset:32768\n" \
    "\tds_read_b128 v[116:119], %[ao1] offset:32768\n" \
    "\tv_mfma_f32_16x16x128_f8f6f4 v[32:35], v[144:151], v[222:229], v[32:35] cbsz:1 blgp:1\n" \
    "\tv_mul_f32 v20, v60, v20\n" \
    "\tv_mul_f32 v21, v60, v21\n" \
    "\tv_mul_f32 v22, v60, v22\n" \
    "\tv_mul_f32 v23, v60, v23\n" \
    "\tds_write_b128 %[ew0], v[20:23]\n" \
    "\ts_waitcnt lgkmcnt(5)\n" \
    "\tv_mfma_f32_16x16x128_f8f6f4 v[28:31], v[152:159], v[96:103], v[28:31] cbsz:1 blgp:1\n" \
    "\tds_read_b128 v[206:209], %[ao0] offset:34816\n" \
    "\tds_read_b128 v[210:213], %[ao1] offset:34816\n" \
    "\tv_mfma_f32_16x16x128_f8f6f4 v[32:35], v[160:167], v[96:103], v[32:35] cbsz:1 blgp:1\n" \
    "\tv_mul_f32 v24, v60, v24\n" \
    "\tv_mul_f32 v25, v60, v25\n" \
    "\tv_mul_f32 v26, v60, v26\n" \
    "\tv_mul_f32 v27, v60, v27\n" \
    "\tds_write_b128 %[ew1], v[24:27]\n" \
    "\ts_waitcnt lgkmcnt(6)\n" \
    "\tv_mfma_f32_16x16x128_f8f6f4 v[28:31], v[168:175], v[104:111], v[28:31] cbsz:1 blgp:1\n" \
    "\tds_read_b128 v[214:217], %[ao0] offset:36864\n" \
    "\tds_read_b128 v[218:221], %[ao1] offset:36864\n" \
    "\tv_mfma_f32_16x16x128_f8f6f4 v[32:35], v[176:183], v[104:111], v[32:35] cbsz:1 blgp:1\n" \
    "\tds_read_b128 v[230:233], %[er] offset:0\n" \
    "\tds_read_b128 v[234:237], %[er] offset:1024\n" \
    "\ts_waitcnt lgkmcnt(8)\n" \
    "\tv_mfma_f32_16x16x128_f8f6f4 v[28:31], v[184:191], v[112:119], v[28:31] cbsz:1 blgp:1\n" \
    "\tds_read_b128 v[222:225], %[ao0] offset:38912\n" \
    "\tds_read_b128 v[226:229], %[ao1] offset:38912\n" \
    "\tv_mfma_f32_16x16x128_f8f6f4 v[32:35], v[192:199], v[112:119], v[32:35] cbsz:1 blgp:1\n" \
    "\ts_waitcnt lgkmcnt(7)\n" \
    "\tv_mfma_f32_16x16x128_f8f6f4 v[28:31], v[64:71], v[206:213], v[28:31] cbsz:1 blgp:1\n" \
    "\tds_read_b128 v[96:99], %[ao0] offset:40960\n" \
    "\tds_read_b128 v[100:103], %[ao1] offset:40960\n" \
    "\tv_mfma_f32_16x16x128_f8f6f4 v[32:35], v[72:79], v[206:213], v[32:35] cbsz:1 blgp:1\n" \
    "\ts_waitcnt lgkmcnt(4)\n" \
    "\tglobal_store_dwordx4 %[oo], v[230:233], s[90:91] sc1\n" \
    "\ts_add_u32 s90, s90, 0x20000\n" \
    "\ts_addc_u32 s91, s91, 0\n" \
    "\ts_waitcnt lgkmcnt(6)\n" \
    "\tv_mfma_f32_16x16x128_f8f6f4 v[28:31], v[80:87], v[214:221], v[28:31] cbsz:1 blgp:1\n" \
    "\tds_read_b128 v[104:107], %[ao0] offset:43008\n" \
    "\tds_read_b128 v[108:111], %[ao1] offset:43008\n" \
    "\tv_mfma_f32_16x16x128_f8f6f4 v[32:35], v[88:95], v[214:221], v[32:35] cbsz:1 blgp:1\n" \
    "\tglobal_store_dwordx4 %[oo], v[234:237], s[90:91] sc1\n" \
    "\ts_add_u32 s90, s90, 0x20000\n" \
    "\ts_addc_u32 s91, s91, 0\n" \
    "\ts_waitcnt lgkmcnt(4)\n" \
    "\tv_mfma_f32_16x16x128_f8f6f4 v[36:39], v[120:127], v[222:229], v[36:39] cbsz:1 blgp:1\n" \
    "\tds_read_b128 v[112:115], %[ao0] offset:4096\n" \
    "\tds_read_b128 v[116:119], %[ao1] offset:4096\n" \
    "\tv_mfma_f32_16x16x128_f8f6f4 v[40:43], v[128:135], v[222:229], v[40:43] cbsz:1 blgp:1\n" \
    "\ts_waitcnt lgkmcnt(4)\n" \
    "\tv_mfma_f32_16x16x128_f8f6f4 v[36:39], v[136:143], v[96:103], v[36:39] cbsz:1 blgp:1\n" \
    "\tds_read_b128 v[206:209], %[ao0] offset:6144\n" \
    "\tds_read_b128 v[210:213], %[ao1] offset:6144\n" \
    "\tv_mfma_f32_16x16x128_f8f6f4 v[40:43], v[144:151], v[96:103], v[40:43] cbsz:1 blgp:1\n" \
    "\tv_mul_f32 v28, v60, v28\n" \
    "\tv_mul_f32 v29, v60, v29\n" \
    "\tv_mul_f32 v30, v60, v30\n" \
    "\tv_mul_f32 v31, v60, v31\n" \
    "\tds_write_b128 %[ew0], v[28:31]\n" \
    "\ts_waitcnt lgkmcnt(5)\n" \
    "\tv_mfma_f32_16x16x128_f8f6f4 v[36:39], v[152:159], v[104:111], v[36:39] cbsz:1 blgp:1\n" \
    "\tds_read_b128 v[214:217], %[ao0] offset:8192\n" \
    "\tds_read_b128 v[218:221], %[ao1] offset:8192\n" \
    "\tv_mfma_f32_16x16x128_f8f6f4 v[40:43], v[160:167], v[104:111], v[40:43] cbsz:1 blgp:1\n" \
    "\tv_mul_f32 v32, v60, v32\n" \
    "\tv_mul_f32 v33, v60, v33\n" \
    "\tv_mul_f32 v34, v60, v34\n" \
    "\tv_mul_f32 v35, v60, v35\n" \
    "\tds_write_b128 %[ew1], v[32:35]\n" \
    "\ts_waitcnt lgkmcnt(6)\n" \
    "\tv_mfma_f32_16x16x128_f8f6f4 v[36:39], v[168:175], v[112:119], v[36:39] cbsz:1 blgp:1\n" \
    "\tds_read_b128 v[222:225], %[ao0] offset:10240\n" \
    "\tds_read_b128 v[226:229], %[ao1] offset:10240\n" \
    "\tv_mfma_f32_16x16x128_f8f6f4 v[40:43], v[176:183], v[112:119], v[40:43] cbsz:1 blgp:1\n" \
    "\tds_read_b128 v[238:241], %[er] offset:0\n" \
    "\tds_read_b128 v[242:245], %[er] offset:1024\n" \
    "\ts_waitcnt lgkmcnt(8)\n" \
    "\tv_mfma_f32_16x16x128_f8f6f4 v[36:39], v[184:191], v[206:213], v[36:39] cbsz:1 blgp:1\n" \
    "\ts_barrier\n" \
    "\ts_sub_u32 s93, s94, %[abase]\n" \
    "\tv_add_u32 %[wa0], s93, %[ao0]\n" \
    "\tv_add_u32 %[wa1], s93, %[ao1]\n" \
    "\tds_read_b128 v[96:99], %[wa0] offset:0\n" \
    "\tds_read_b128 v[100:103], %[wa1] offset:0\n" \
    "\tv_mfma_f32_16x16x128_f8f6f4 v[40:43], v[192:199], v[206:213], v[40:43] cbsz:1 blgp:1\n" \
    "\ts_waitcnt lgkmcnt(7)\n" \
    "\tv_mfma_f32_16x16x128_f8f6f4 v[36:39], v[64:71], v[214:221], v[36:39] cbsz:1 blgp:1\n" \
    "\tds_read_b128 v[104:107], %[wa0] offset:2048\n" \
    "\tds_read_b128 v[108:111], %[wa1] offset:2048\n" \
    "\tv_mfma_f32_16x16x128_f8f6f4 v[40:43], v[72:79], v[214:221], v[40:43] cbsz:1 blgp:1\n" \
    "\ts_waitcnt lgkmcnt(4)\n" \
    "\tglobal_store_dwordx4 %[oo], v[238:241], s[90:91] sc1\n" \
    "\ts_add_u32 s90, s90, 0x20000\n" \
    "\ts_addc_u32 s91, s91, 0\n" \
    "\ts_waitcnt lgkmcnt(6)\n" \
    "\tv_mfma_f32_16x16x128_f8f6f4 v[36:39], v[80:87], v[222:229], v[36:39] cbsz:1 blgp:1\n" \
    "\tds_read_b128 v[112:115], %[wa0] offset:4096\n" \
    "\tds_read_b128 v[116:119], %[wa1] offset:4096\n" \
    "\tv_mfma_f32_16x16x128_f8f6f4 v[40:43], v[88:95], v[222:229], v[40:43] cbsz:1 blgp:1\n" \
    "\tglobal_store_dwordx4 %[oo], v[242:245], s[90:91] sc1\n" \
    "\ts_add_u32 s90, s90, 0x20000\n" \
    "\ts_addc_u32 s91, s91, 0\n" \
    "\ts_waitcnt lgkmcnt(4)\n" \
    "\tv_mfma_f32_16x16x128_f8f6f4 v[44:47], v[120:127], v[96:103], v[44:47] cbsz:1 blgp:1\n" \
    "\tds_read_b128 v[206:209], %[wa0] offset:6144\n" \
    "\tds_read_b128 v[210:213], %[wa1] offset:6144\n" \
    "\tv_mfma_f32_16x16x128_f8f6f4 v[48:51], v[128:135], v[96:103], v[48:51] cbsz:1 blgp:1\n" \
    "\ts_waitcnt lgkmcnt(4)\n" \
    "\tv_mfma_f32_16x16x128_f8f6f4 v[44:47], v[136:143], v[104:111], v[44:47] cbsz:1 blgp:1\n" \
    "\tds_read_b128 v[214:217], %[wa0] offset:8192\n" \
    "\tds_read_b128 v[218:221], %[wa1] offset:8192\n" \
    "\tv_mfma_f32_16x16x128_f8f6f4 v[48:51], v[144:151], v[104:111], v[48:51] cbsz:1 blgp:1\n" \
    "\tv_mul_f32 v36, v60, v36\n" \
    "\tv_mul_f32 v37, v60, v37\n" \
    "\tv_mul_f32 v38, v60, v38\n" \
    "\tv_mul_f32 v39, v60, v39\n" \
    "\tds_write_b128 %[ew0], v[36:39]\n" \
    "\ts_waitcnt lgkmcnt(5)\n" \
    "\tv_mfma_f32_16x16x128_f8f6f4 v[44:47], v[152:159], v[112:119], v[44:47] cbsz:1 blgp:1\n" \
    "\tds_read_b128 v[222:225], %[wa0] offset:10240\n" \
    "\tds_read_b128 v[226:229], %[wa1] offset:10240\n" \
    "\tv_mfma_f32_16x16x128_f8f6f4 v[48:51], v[160:167], v[112:119], v[48:51] cbsz:1 blgp:1\n" \
    "\tv_mul_f32 v40, v60, v40\n" \
    "\tv_mul_f32 v41, v60, v41\n" \
    "\tv_mul_f32 v42, v60, v42\n" \
    "\tv_mul_f32 v43, v60, v43\n" \
    "\tds_write_b128 %[ew1], v[40:43]\n" \
    "\ts_waitcnt lgkmcnt(6)\n" \
    "\tv_mfma_f32_16x16x128_f8f6f4 v[44:47], v[168:175], v[206:213], v[44:47] cbsz:1 blgp:1\n" \
    "\tds_read_b128 v[96:99], %[wa0] offset:12288\n" \
    "\tds_read_b128 v[100:103], %[wa1] offset:12288\n" \
    "\tv_mfma_f32_16x16x128_f8f6f4 v[48:51], v[176:183], v[206:213], v[48:51] cbsz:1 blgp:1\n" \
    "\tds_read_b128 v[230:233], %[er] offset:0\n" \
    "\tds_read_b128 v[234:237], %[er] offset:1024\n" \
    "\ts_waitcnt lgkmcnt(8)\n" \
    "\tv_mfma_f32_16x16x128_f8f6f4 v[44:47], v[184:191], v[214:221], v[44:47] cbsz:1 blgp:1\n" \
    "\tv_mfma_f32_16x16x128_f8f6f4 v[48:51], v[192:199], v[214:221], v[48:51] cbsz:1 blgp:1\n" \
    "\ts_waitcnt lgkmcnt(5)\n" \
    "\tv_mfma_f32_16x16x128_f8f6f4 v[44:47], v[64:71], v[222:229], v[44:47] cbsz:1 blgp:1\n" \
    "\tv_mfma_f32_16x16x128_f8f6f4 v[48:51], v[72:79], v[222:229], v[48:51] cbsz:1 blgp:1\n" \
    "\ts_waitcnt lgkmcnt(0)\n" \
    "\tglobal_store_dwordx4 %[oo], v[230:233], s[90:91] sc1\n" \
    "\ts_add_u32 s90, s90, 0x20000\n" \
    "\ts_addc_u32 s91, s91, 0\n" \
    "\ts_waitcnt lgkmcnt(2)\n" \
    "\tv_mfma_f32_16x16x128_f8f6f4 v[44:47], v[80:87], v[96:103], v[44:47] cbsz:1 blgp:1\n" \
    "\tv_mfma_f32_16x16x128_f8f6f4 v[48:51], v[88:95], v[96:103], v[48:51] cbsz:1 blgp:1\n" \
    "\tglobal_store_dwordx4 %[oo], v[234:237], s[90:91] sc1\n" \
    "\ts_add_u32 s90, s90, 0x20000\n" \
    "\ts_addc_u32 s91, s91, 0\n" \
    "\ts_nop 11\n" \
    "\tv_mul_f32 v44, v60, v44\n" \
    "\tv_mul_f32 v45, v60, v45\n" \
    "\tv_mul_f32 v46, v60, v46\n" \
    "\tv_mul_f32 v47, v60, v47\n" \
    "\tds_write_b128 %[ew0], v[44:47]\n" \
    "\tv_mul_f32 v48, v60, v48\n" \
    "\tv_mul_f32 v49, v60, v49\n" \
    "\tv_mul_f32 v50, v60, v50\n" \
    "\tv_mul_f32 v51, v60, v51\n" \
    "\tds_write_b128 %[ew1], v[48:51]\n" \
    "\tds_read_b128 v[238:241], %[er] offset:0\n" \
    "\tds_read_b128 v[242:245], %[er] offset:1024\n" \
    "\ts_waitcnt lgkmcnt(0)\n" \
    "\tglobal_store_dwordx4 %[oo], v[238:241], s[90:91] sc1\n" \
    "\ts_add_u32 s90, s90, 0x20000\n" \
    "\ts_addc_u32 s91, s91, 0\n" \
    "\tglobal_store_dwordx4 %[oo], v[242:245], s[90:91] sc1\n" \
    "\ts_add_u32 s90, s90, 0x20000\n" \
    "\ts_addc_u32 s91, s91, 0\n" \
    "\ts_endpgm\n" \
    "10:\n" \
    "\ts_waitcnt lgkmcnt(0)\n" \
    "\ts_barrier\n" \
    "\tds_read_b32 %[vd], %[dp]\n" \
    "\tv_add_u32 %[dp], 4, %[dp]\n" \
    "\tv_bfi_b32 %[dp], 63, %[dp], %[dbase]\n" \
    "\ts_waitcnt lgkmcnt(0)\n" \
    "\tv_cmp_eq_u32 vcc, 0x2007, %[vd]\n" \
    "\tv_and_b32 %[t], 7, %[vd]\n" \
    "\tv_lshl_add_u32 %[wa0], %[t], 14, %[bo0]\n" \
    "\tv_lshl_add_u32 %[wa1], %[t], 14, %[bo1]\n" \
    "\ts_cbranch_vccnz 10b\n" \
    "\ts_branch 20b\n" \
    "11:\n" \
    "\ts_waitcnt lgkmcnt(0)\n" \
    "\ts_barrier\n" \
    "\tds_read_b32 %[vd], %[dp]\n" \
    "\tv_add_u32 %[dp], 4, %[dp]\n" \
    "\tv_bfi_b32 %[dp], 63, %[dp], %[dbase]\n" \
    "\ts_waitcnt lgkmcnt(0)\n" \
    "\tv_cmp_eq_u32 vcc, 0x2007, %[vd]\n" \
    "\tv_and_b32 %[t], 7, %[vd]\n" \
    "\tv_lshl_add_u32 %[wa0], %[t], 14, %[bo0]\n" \
    "\tv_lshl_add_u32 %[wa1], %[t], 14, %[bo1]\n" \
    "\ts_cbranch_vccnz 11b\n" \
    "\ts_branch 21b\n" \
    "12:\n" \
    "\ts_waitcnt lgkmcnt(0)\n" \
    "\ts_barrier\n" \
    "\tds_read_b32 %[vd], %[dp]\n" \
    "\tv_add_u32 %[dp], 4, %[dp]\n" \
    "\tv_bfi_b32 %[dp], 63, %[dp], %[dbase]\n" \
    "\ts_waitcnt lgkmcnt(0)\n" \
    "\tv_cmp_eq_u32 vcc, 0x2007, %[vd]\n" \
    "\tv_and_b32 %[t], 7, %[vd]\n" \
    "\tv_lshl_add_u32 %[wa0], %[t], 14, %[bo0]\n" \
    "\tv_lshl_add_u32 %[wa1], %[t], 14, %[bo1]\n" \
    "\ts_cbranch_vccnz 12b\n" \
    "\ts_branch 22b\n" \
    "13:\n" \
    "\ts_waitcnt lgkmcnt(0)\n" \
    "\ts_barrier\n" \
    "\tds_read_b32 %[vd], %[dp]\n" \
    "\tv_add_u32 %[dp], 4, %[dp]\n" \
    "\tv_bfi_b32 %[dp], 63, %[dp], %[dbase]\n" \
    "\ts_waitcnt lgkmcnt(0)\n" \
    "\tv_cmp_eq_u32 vcc, 0x2007, %[vd]\n" \
    "\tv_and_b32 %[t], 7, %[vd]\n" \
    "\tv_lshl_add_u32 %[wa0], %[t], 14, %[bo0]\n" \
    "\tv_lshl_add_u32 %[wa1], %[t], 14, %[bo1]\n" \
    "\ts_cbranch_vccnz 13b\n" \
    "\ts_branch 23b\n" \
    "14:\n" \
    "\ts_waitcnt lgkmcnt(0)\n" \
    "\ts_barrier\n" \
    "\tds_read_b32 %[vd], %[dp]\n" \
    "\tv_add_u32 %[dp], 4, %[dp]\n" \
    "\tv_bfi_b32 %[dp], 63, %[dp], %[dbase]\n" \
    "\ts_waitcnt lgkmcnt(0)\n" \
    "\tv_cmp_eq_u32 vcc, 0x2007, %[vd]\n" \
    "\tv_and_b32 %[t], 7, %[vd]\n" \
    "\tv_lshl_add_u32 %[wa0], %[t], 14, %[bo0]\n" \
    "\tv_lshl_add_u32 %[wa1], %[t], 14, %[bo1]\n" \
    "\ts_cbranch_vccnz 14b\n" \
    "\ts_branch 24b\n" \
    "5:\n" \
    "\tds_read_b32 %[vd], %[stat]\n" \
    "\tv_add_u32 %[ew0], 118784, %[ew0]\n" \
    "\tv_add_u32 %[ew1], 118784, %[ew1]\n" \
    "\tv_add_u32 %[er], 118784, %[er]\n" \
    "\ts_waitcnt lgkmcnt(1)\n" \
    "\tv_mfma_f32_16x16x128_f8f6f4 v[20:23], v[120:127], v[136:143], v[20:23] cbsz:1 blgp:1\n" \
    "\tv_mfma_f32_16x16x128_f8f6f4 v[24:27], v[128:135], v[136:143], v[24:27] cbsz:1 blgp:1\n" \
    "\ts_waitcnt lgkmcnt(0)\n" \
    "\tv_cmp_eq_u32 vcc, 0, %[vd]\n" \
    "\tv_mov_b32 v60, 0x7fc00000\n" \
    "\tv_mfma_f32_16x16x128_f8f6f4 v[28:31], v[120:127], v[144:151], v[28:31] cbsz:1 blgp:1\n" \
    "\tv_cndmask_b32 v60, v60, %[sc], vcc\n" \
    "\tv_mov_b32 v61, v60\n" \
    "\tv_mfma_f32_16x16x128_f8f6f4 v[32:35], v[128:135], v[144:151], v[32:35] cbsz:1 blgp:1\n" \
    "\ts_nop 4\n" \
    "\tv_mul_f32 v20, v60, v20\n" \
    "\tv_mul_f32 v21, v60, v21\n" \
    "\tv_mul_f32 v22, v60, v22\n" \
    "\tv_mul_f32 v23, v60, v23\n" \
    "\tds_write_b128 %[ew0], v[20:23]\n" \
    "\tv_mul_f32 v24, v60, v24\n" \
    "\tv_mul_f32 v25, v60, v25\n" \
    "\tv_mul_f32 v26, v60, v26\n" \
    "\tv_mul_f32 v27, v60, v27\n" \
    "\tds_write_b128 %[ew1], v[24:27]\n" \
    "\tds_read_b128 v[64:67], %[er] offset:0\n" \
    "\tds_read_b128 v[68:71], %[er] offset:1024\n" \
    "\tv_mfma_f32_16x16x128_f8f6f4 v[36:39], v[120:127], v[152:159], v[36:39] cbsz:1 blgp:1\n" \
    "\tv_mfma_f32_16x16x128_f8f6f4 v[40:43], v[128:135], v[152:159], v[40:43] cbsz:1 blgp:1\n" \
    "\tv_mul_f32 v28, v60, v28\n" \
    "\tv_mul_f32 v29, v60, v29\n" \
    "\tv_mul_f32 v30, v60, v30\n" \
    "\tv_mul_f32 v31, v60, v31\n" \
    "\tds_write_b128 %[ew0], v[28:31]\n" \
    "\tv_mul_f32 v32, v60, v32\n" \
    "\tv_mul_f32 v33, v60, v33\n" \
    "\tv_mul_f32 v34, v60, v34\n" \
    "\tv_mul_f32 v35, v60, v35\n" \
    "\tds_write_b128 %[ew1], v[32:35]\n" \
    "\tds_read_b128 v[72:75], %[er] offset:0\n" \
    "\tds_read_b128 v[76:79], %[er] offset:1024\n" \
    "\ts_waitcnt lgkmcnt(4)\n" \
    "\tglobal_store_dwordx4 %[oo], v[64:67], s[90:91] sc1\n" \
    "\ts_add_u32 s90, s90, 0x20000\n" \
    "\ts_addc_u32 s91, s91, 0\n" \
    "\tglobal_store_dwordx4 %[oo], v[68:71], s[90:91] sc1\n" \
    "\ts_add_u32 s90, s90, 0x20000\n" \
    "\ts_addc_u32 s91, s91, 0\n" \
    "\tv_mfma_f32_16x16x128_f8f6f4 v[44:47], v[120:127], v[160:167], v[44:47] cbsz:1 blgp:1\n" \
    "\tv_mfma_f32_16x16x128_f8f6f4 v[48:51], v[128:135], v[160:167], v[48:51] cbsz:1 blgp:1\n" \
    "\tv_mul_f32 v36, v60, v36\n" \
    "\tv_mul_f32 v37, v60, v37\n" \
    "\tv_mul_f32 v38, v60, v38\n" \
    "\tv_mul_f32 v39, v60, v39\n" \
    "\tds_write_b128 %[ew0], v[36:39]\n" \
    "\tv_mul_f32 v40, v60, v40\n" \
    "\tv_mul_f32 v41, v60, v41\n" \
    "\tv_mul_f32 v42, v60, v42\n" \
    "\tv_mul_f32 v43, v60, v43\n" \
    "\tds_write_b128 %[ew1], v[40:43]\n" \
    "\tds_read_b128 v[64:67], %[er] offset:0\n" \
    "\tds_read_b128 v[68:71], %[er] offset:1024\n" \
    "\ts_waitcnt lgkmcnt(4)\n" \
    "\tglobal_store_dwordx4 %[oo], v[72:75], s[90:91] sc1\n" \
    "\ts_add_u32 s90, s90, 0x20000\n" \
    "\ts_addc_u32 s91, s91, 0\n" \
    "\tglobal_store_dwordx4 %[oo], v[76:79], s[90:91] sc1\n" \
    "\ts_add_u32 s90, s90, 0x20000\n" \
    "\ts_addc_u32 s91, s91, 0\n" \
    "\tv_mul_f32 v44, v60, v44\n" \
    "\tv_mul_f32 v45, v60, v45\n" \
    "\tv_mul_f32 v46, v60, v46\n" \
    "\tv_mul_f32 v47, v60, v47\n" \
    "\tds_write_b128 %[ew0], v[44:47]\n" \
    "\tv_mul_f32 v48, v60, v48\n" \
    "\tv_mul_f32 v49, v60, v49\n" \
    "\tv_mul_f32 v50, v60, v50\n" \
    "\tv_mul_f32 v51, v60, v51\n" \
    "\tds_write_b128 %[ew1], v[48:51]\n" \
    "\tds_read_b128 v[72:75], %[er] offset:0\n" \
    "\tds_read_b128 v[76:79], %[er] offset:1024\n" \
    "\ts_waitcnt lgkmcnt(4)\n" \
    "\tglobal_store_dwordx4 %[oo], v[64:67], s[90:91] sc1\n" \
    "\ts_add_u32 s90, s90, 0x20000\n" \
    "\ts_addc_u32 s91, s91, 0\n" \
    "\tglobal_store_dwordx4 %[oo], v[68:71], s[90:91] sc1\n" \
    "\ts_add_u32 s90, s90, 0x20000\n" \
    "\ts_addc_u32 s91, s91, 0\n" \
    "\ts_waitcnt lgkmcnt(0)\n" \
    "\tglobal_store_dwordx4 %[oo], v[72:75], s[90:91] sc1\n" \
    "\ts_add_u32 s90, s90, 0x20000\n" \
    "\ts_addc_u32 s91, s91, 0\n" \
    "\tglobal_store_dwordx4 %[oo], v[76:79], s[90:91] sc1\n" \
    "\ts_add_u32 s90, s90, 0x20000\n" \
    "\ts_addc_u32 s91, s91, 0\n" \
    "\ts_endpgm\n"
#define CONS_ASM5 \
    "\tv_mov_b64 v[20:21], 0\n" \
    "\tv_mov_b64 v[22:23], 0\n" \
    "\tv_mov_b64 v[24:25], 0\n" \
    "\tv_mov_b64 v[26:27], 0\n" \
    "\tv_mov_b64 v[28:29], 0\n" \
    "\tv_mov_b64 v[30:31], 0\n" \
    "\tv_mov_b64 v[32:33], 0\n" \
    "\tv_mov_b64 v[34:35], 0\n" \
    "\tv_mov_b64 v[36:37], 0\n" \
    "\tv_mov_b64 v[38:39], 0\n" \
    "\tv_mov_b64 v[40:41], 0\n" \
    "\tv_mov_b64 v[42:43], 0\n" \
    "\tv_mov_b64 v[44:45], 0\n" \
    "\tv_mov_b64 v[46:47], 0\n" \
    "\tv_mov_b64 v[48:49], 0\n" \
    "\tv_mov_b64 v[50:51], 0\n" \
    "\tv_mov_b64 v[52:53], 0\n" \
    "\tv_mov_b64 v[54:55], 0\n" \
    "\tv_mov_b64 v[56:57], 0\n" \
    "\tv_mov_b64 v[58:59], 0\n" \
    "\ts_mov_b64 s[90:91], %[ob]\n" \
    "\ts_waitcnt lgkmcnt(0)\n" \
    "\ts_barrier\n" \
    "\tds_read_b32 %[vd], %[dp]\n" \
    "\tds_read_b64_tr_b8 v[64:65], %[wa0] offset:0\n" \
    "\tds_read_b64_tr_b8 v[66:67], %[wa0] offset:1024\n" \
    "\tds_read_b64_tr_b8 v[68:69], %[wa0] offset:8192\n" \
    "\tds_read_b64_tr_b8 v[70:71], %[wa0] offset:9216\n" \
    "\tds_read_b64_tr_b8 v[72:73], %[wa1] offset:0\n" \
    "\tds_read_b64_tr_b8 v[74:75], %[wa1] offset:1024\n" \
    "\tds_read_b64_tr_b8 v[76:77], %[wa1] offset:8192\n" \
    "\tds_read_b64_tr_b8 v[78:79], %[wa1] offset:9216\n" \
    "\ts_waitcnt lgkmcnt(8)\n" \
    "\tds_read_b128 v[80:83], %[ao0] offset:0\n" \
    "\tds_read_b128 v[84:87], %[ao1] offset:0\n" \
    "\tds_read_b128 v[88:91], %[ao0] offset:2048\n" \
    "\tds_read_b128 v[92:95], %[ao1] offset:2048\n" \
    "\tds_read_b128 v[96:99], %[ao0] offset:4096\n" \
    "\tds_read_b128 v[100:103], %[ao1] offset:4096\n" \
    "\tds_read_b128 v[104:107], %[ao0] offset:6144\n" \
    "\tds_read_b128 v[108:111], %[ao1] offset:6144\n" \
    "\tds_read_b128 v[112:115], %[ao0] offset:8192\n" \
    "\tds_read_b128 v[116:119], %[ao1] offset:8192\n" \
    "\tv_cmp_eq_u32 vcc, 0x2007, %[vd]\n" \
    "\tv_and_b32 %[t], 7, %[vd]\n" \
    "\tv_lshl_add_u32 %[wa0], %[t], 14, %[bo0]\n" \
    "\tv_lshl_add_u32 %[wa1], %[t], 14, %[bo1]\n" \
    "\tv_add_u32 %[dp], 4, %[dp]\n" \
    "\tv_bfi_b32 %[dp], 63, %[dp], %[dbase]\n" \
    "3:\n" \
    "\ts_cbranch_vccnz 10f\n" \
    "20:\n" \
    "\ts_waitcnt lgkmcnt(0)\n" \
    "\ts_barrier\n" \
    "\tv_mfma_f32_16x16x128_f8f6f4 v[20:23], v[64:71], v[80:87], v[20:23] cbsz:1 blgp:1\n" \
    "\tds_read_b64_tr_b8 v[120:121], %[wa0] offset:0\n" \
    "\tds_read_b64_tr_b8 v[122:123], %[wa0] offset:1024\n" \
    "\tds_read_b64_tr_b8 v[124:125], %[wa0] offset:8192\n" \
    "\tds_read_b64_tr_b8 v[126:127], %[wa0] offset:9216\n" \
    "\tv_mfma_f32_16x16x128_f8f6f4 v[24:27], v[72:79], v[80:87], v[24:27] cbsz:1 blgp:1\n" \
    "\tds_read_b64_tr_b8 v[128:129], %[wa1] offset:0\n" \
    "\tds_read_b64_tr_b8 v[130:131], %[wa1] offset:1024\n" \
    "\tds_read_b64_tr_b8 v[132:133], %[wa1] offset:8192\n" \
    "\tds_read_b64_tr_b8 v[134:135], %[wa1] offset:9216\n" \
    "\tv_mfma_f32_16x16x128_f8f6f4 v[28:31], v[64:71], v[88:95], v[28:31] cbsz:1 blgp:1\n" \
    "\tds_read_b32 %[vd], %[dp]\n" \
    "\tds_read_b128 v[136:139], %[ao0] offset:10240\n" \
    "\tds_read_b128 v[140:143], %[ao1] offset:10240\n" \
    "\tv_mfma_f32_16x16x128_f8f6f4 v[32:35], v[72:79], v[88:95], v[32:35] cbsz:1 blgp:1\n" \
    "\tds_read_b128 v[144:147], %[ao0] offset:12288\n" \
    "\tds_read_b128 v[148:151], %[ao1] offset:12288\n" \
    "\tv_mfma_f32_16x16x128_f8f6f4 v[36:39], v[64:71], v[96:103], v[36:39] cbsz:1 blgp:1\n" \
    "\tds_read_b128 v[152:155], %[ao0] offset:14336\n" \
    "\tds_read_b128 v[156:159], %[ao1] offset:14336\n" \
    "\tv_mfma_f32_16x16x128_f8f6f4 v[40:43], v[72:79], v[96:103], v[40:43] cbsz:1 blgp:1\n" \
    "\tds_read_b128 v[160:163], %[ao0] offset:16384\n" \
    "\tds_read_b128 v[164:167], %[ao1] offset:16384\n" \
    "\tv_mfma_f32_16x16x128_f8f6f4 v[44:47], v[64:71], v[104:111], v[44:47] cbsz:1 blgp:1\n" \
    "\tds_read_b128 v[168:171], %[ao0] offset:18432\n" \
    "\tds_read_b128 v[172:175], %[ao1] offset:18432\n" \
    "\tv_mfma_f32_16x16x128_f8f6f4 v[48:51], v[72:79], v[104:111], v[48:51] cbsz:1 blgp:1\n" \
    "\ts_waitcnt lgkmcnt(10)\n" \
    "\tv_and_b32 %[t], 7, %[vd]\n" \
    "\tv_lshl_add_u32 %[wa0], %[t], 14, %[bo0]\n" \
    "\tv_lshl_add_u32 %[wa1], %[t], 14, %[bo1]\n" \
    "\tv_mfma_f32_16x16x128_f8f6f4 v[52:55], v[64:71], v[112:119], v[52:55] cbsz:1 blgp:1\n" \
    "\tv_cmp_eq_u32 vcc, 0x2007, %[vd]\n" \
    "\tv_add_u32 %[dp], 4, %[dp]\n" \
    "\tv_bfi_b32 %[dp], 63, %[dp], %[dbase]\n" \
    "\tv_mfma_f32_16x16x128_f8f6f4 v[56:59], v[72:79], v[112:119], v[56:59] cbsz:1 blgp:1\n" \
    "\ts_cbranch_vccnz 11f\n" \
    "21:\n" \
    "\ts_waitcnt lgkmcnt(0)\n" \
    "\ts_barrier\n" \
    "\tv_mfma_f32_16x16x128_f8f6f4 v[20:23], v[120:127], v[136:143], v[20:23] cbsz:1 blgp:1\n" \
    "\tds_read_b64_tr_b8 v[64:65], %[wa0] offset:0\n" \
    "\tds_read_b64_tr_b8 v[66:67], %[wa0] offset:1024\n" \
    "\tds_read_b64_tr_b8 v[68:69], %[wa0] offset:8192\n" \
    "\tds_read_b64_tr_b8 v[70:71], %[wa0] offset:9216\n" \
    "\tv_mfma_f32_16x16x128_f8f6f4 v[24:27], v[128:135], v[136:143], v[24:27] cbsz:1 blgp:1\n" \
    "\tds_read_b64_tr_b8 v[72:73], %[wa1] offset:0\n" \
    "\tds_read_b64_tr_b8 v[74:75], %[wa1] offset:1024\n" \
    "\tds_read_b64_tr_b8 v[76:77], %[wa1] offset:8192\n" \
    "\tds_read_b64_tr_b8 v[78:79], %[wa1] offset:9216\n" \
    "\tv_mfma_f32_16x16x128_f8f6f4 v[28:31], v[120:127], v[144:151], v[28:31] cbsz:1 blgp:1\n" \
    "\tds_read_b32 %[vd], %[dp]\n" \
    "\tds_read_b128 v[80:83], %[ao0] offset:20480\n" \
    "\tds_read_b128 v[84:87], %[ao1] offset:20480\n" \
    "\tv_mfma_f32_16x16x128_f8f6f4 v[32:35], v[128:135], v[144:151], v[32:35] cbsz:1 blgp:1\n" \
    "\tds_read_b128 v[88:91], %[ao0] offset:22528\n" \
    "\tds_read_b128 v[92:95], %[ao1] offset:22528\n" \
    "\tv_mfma_f32_16x16x128_f8f6f4 v[36:39], v[120:127], v[152:159], v[36:39] cbsz:1 blgp:1\n" \
    "\tds_read_b128 v[96:99], %[ao0] offset:24576\n" \
    "\tds_read_b128 v[100:103], %[ao1] offset:24576\n" \
    "\tv_mfma_f32_16x16x128_f8f6f4 v[40:43], v[128:135], v[152:159], v[40:43] cbsz:1 blgp:1\n" \
    "\tds_read_b128 v[104:107], %[ao0] offset:26624\n" \
    "\tds_read_b128 v[108:111], %[ao1] offset:26624\n" \
    "\tv_mfma_f32_16x16x128_f8f6f4 v[44:47], v[120:127], v[160:167], v[44:47] cbsz:1 blgp:1\n" \
    "\tds_read_b128 v[112:115], %[ao0] offset:28672\n" \
    "\tds_read_b128 v[116:119], %[ao1] offset:28672\n" \
    "\tv_mfma_f32_16x16x128_f8f6f4 v[48:51], v[128:135], v[160:167], v[48:51] cbsz:1 blgp:1\n" \
    "\ts_waitcnt lgkmcnt(10)\n" \
    "\tv_and_b32 %[t], 7, %[vd]\n" \
    "\tv_lshl_add_u32 %[wa0], %[t], 14, %[bo0]\n" \
    "\tv_lshl_add_u32 %[wa1], %[t], 14, %[bo1]\n" \
    "\tv_mfma_f32_16x16x128_f8f6f4 v[52:55], v[120:127], v[168:175], v[52:55] cbsz:1 blgp:1\n" \
    "\tv_cmp_eq_u32 vcc, 0x2007, %[vd]\n" \
    "\tv_add_u32 %[dp], 4, %[dp]\n" \
    "\tv_bfi_b32 %[dp], 63, %[dp], %[dbase]\n" \
    "\tv_mfma_f32_16x16x128_f8f6f4 v[56:59], v[128:135], v[168:175], v[56:59] cbsz:1 blgp:1\n" \
    "\ts_cbranch_vccnz 12f\n" \
    "22:\n" \
    "\ts_waitcnt lgkmcnt(0)\n" \
    "\ts_barrier\n" \
    "\tv_mfma_f32_16x16x128_f8f6f4 v[20:23], v[64:71], v[80:87], v[20:23] cbsz:1 blgp:1\n" \
    "\tds_read_b64_tr_b8 v[120:121], %[wa0] offset:0\n" \
    "\tds_read_b64_tr_b8 v[122:123], %[wa0] offset:1024\n" \
    "\tds_read_b64_tr_b8 v[124:125], %[wa0] offset:8192\n" \
    "\tds_read_b64_tr_b8 v[126:127], %[wa0] offset:9216\n" \
    "\tv_mfma_f32_16x16x128_f8f6f4 v[24:27], v[72:79], v[80:87], v[24:27] cbsz:1 blgp:1\n" \
    "\tds_read_b64_tr_b8 v[128:129], %[wa1] offset:0\n" \
    "\tds_read_b64_tr_b8 v[130:131], %[wa1] offset:1024\n" \
    "\tds_read_b64_tr_b8 v[132:133], %[wa1] offset:8192\n" \
    "\tds_read_b64_tr_b8 v[134:135], %[wa1] offset:9216\n" \
    "\tv_mfma_f32_16x16x128_f8f6f4 v[28:31], v[64:71], v[88:95], v[28:31] cbsz:1 blgp:1\n" \
    "\tds_read_b32 %[vd], %[dp]\n" \
    "\tds_read_b128 v[136:139], %[ao0] offset:30720\n" \
    "\tds_read_b128 v[140:143], %[ao1] offset:30720\n" \
    "\tv_mfma_f32_16x16x128_f8f6f4 v[32:35], v[72:79], v[88:95], v[32:35] cbsz:1 blgp:1\n" \
    "\tds_read_b128 v[144:147], %[ao0] offset:32768\n" \
    "\tds_read_b128 v[148:151], %[ao1] offset:32768\n" \
    "\tv_mfma_f32_16x16x128_f8f6f4 v[36:39], v[64:71], v[96:103], v[36:39] cbsz:1 blgp:1\n" \
    "\tds_read_b128 v[152:155], %[ao0] offset:34816\n" \
    "\tds_read_b128 v[156:159], %[ao1] offset:34816\n" \
    "\tv_mfma_f32_16x16x128_f8f6f4 v[40:43], v[72:79], v[96:103], v[40:43] cbsz:1 blgp:1\n" \
    "\tds_read_b128 v[160:163], %[ao0] offset:36864\n" \
    "\tds_read_b128 v[164:167], %[ao1] offset:36864\n" \
    "\tv_mfma_f32_16x16x128_f8f6f4 v[44:47], v[64:71], v[104:111], v[44:47] cbsz:1 blgp:1\n" \
    "\tds_read_b128 v[168:171], %[ao0] offset:38912\n" \
    "\tds_read_b128 v[172:175], %[ao1] offset:38912\n" \
    "\tv_mfma_f32_16x16x128_f8f6f4 v[48:51], v[72:79], v[104:111], v[48:51] cbsz:1 blgp:1\n" \
    "\ts_waitcnt lgkmcnt(10)\n" \
    "\tv_and_b32 %[t], 7, %[vd]\n" \
    "\tv_lshl_add_u32 %[wa0], %[t], 14, %[bo0]\n" \
    "\tv_lshl_add_u32 %[wa1], %[t], 14, %[bo1]\n" \
    "\tv_mfma_f32_16x16x128_f8f6f4 v[52:55], v[64:71], v[112:119], v[52:55] cbsz:1 blgp:1\n" \
    "\tv_cmp_eq_u32 vcc, 0x2007, %[vd]\n" \
    "\tv_add_u32 %[dp], 4, %[dp]\n" \
    "\tv_bfi_b32 %[dp], 63, %[dp], %[dbase]\n" \
    "\ts_cmp_eq_u32 %[n], 1\n" \
    "\tv_mfma_f32_16x16x128_f8f6f4 v[56:59], v[72:79], v[112:119], v[56:59] cbsz:1 blgp:1\n" \
    "\ts_cbranch_scc1 5f\n" \
    "\ts_cbranch_vccnz 13f\n" \
    "23:\n" \
    "\ts_waitcnt lgkmcnt(0)\n" \
    "\ts_barrier\n" \
    "\tv_mfma_f32_16x16x128_f8f6f4 v[20:23], v[120:127], v[136:143], v[20:23] cbsz:1 blgp:1\n" \
    "\tv_readfirstlane_b32 s92, %[t]\n" \
    "\tds_read_b64_tr_b8 v[64:65], %[wa0] offset:0\n" \
    "\tds_read_b64_tr_b8 v[66:67], %[wa0] offset:1024\n" \
    "\tds_read_b64_tr_b8 v[68:69], %[wa0] offset:8192\n" \
    "\tds_read_b64_tr_b8 v[70:71], %[wa0] offset:9216\n" \
    "\tv_mfma_f32_16x16x128_f8f6f4 v[24:27], v[128:135], v[136:143], v[24:27] cbsz:1 blgp:1\n" \
    "\tds_read_b64_tr_b8 v[72:73], %[wa1] offset:0\n" \
    "\tds_read_b64_tr_b8 v[74:75], %[wa1] offset:1024\n" \
    "\tds_read_b64_tr_b8 v[76:77], %[wa1] offset:8192\n" \
    "\tds_read_b64_tr_b8 v[78:79], %[wa1] offset:9216\n" \
    "\tv_mfma_f32_16x16x128_f8f6f4 v[28:31], v[120:127], v[144:151], v[28:31] cbsz:1 blgp:1\n" \
    "\tds_read_b32 %[vd], %[dp]\n" \
    "\tds_read_b128 v[80:83], %[ao0] offset:0\n" \
    "\tds_read_b128 v[84:87], %[ao1] offset:0\n" \
    "\tv_mfma_f32_16x16x128_f8f6f4 v[32:35], v[128:135], v[144:151], v[32:35] cbsz:1 blgp:1\n" \
    "\tds_read_b128 v[88:91], %[ao0] offset:2048\n" \
    "\tds_read_b128 v[92:95], %[ao1] offset:2048\n" \
    "\tv_mfma_f32_16x16x128_f8f6f4 v[36:39], v[120:127], v[152:159], v[36:39] cbsz:1 blgp:1\n" \
    "\tds_read_b128 v[96:99], %[ao0] offset:4096\n" \
    "\tds_read_b128 v[100:103], %[ao1] offset:4096\n" \
    "\tv_mfma_f32_16x16x128_f8f6f4 v[40:43], v[128:135], v[152:159], v[40:43] cbsz:1 blgp:1\n" \
    "\tds_read_b128 v[104:107], %[ao0] offset:6144\n" \
    "\tds_read_b128 v[108:111], %[ao1] offset:6144\n" \
    "\tv_mfma_f32_16x16x128_f8f6f4 v[44:47], v[120:127], v[160:167], v[44:47] cbsz:1 blgp:1\n" \
    "\tds_read_b128 v[112:115], %[ao0] offset:8192\n" \
    "\tds_read_b128 v[116:119], %[ao1] offset:8192\n" \
    "\tv_mfma_f32_16x16x128_f8f6f4 v[48:51], v[128:135], v[160:167], v[48:51] cbsz:1 blgp:1\n" \
    "\ts_waitcnt lgkmcnt(10)\n" \
    "\tv_and_b32 %[t], 7, %[vd]\n" \
    "\tv_lshl_add_u32 %[wa0], %[t], 14, %[bo0]\n" \
    "\tv_lshl_add_u32 %[wa1], %[t], 14, %[bo1]\n" \
    "\tv_mfma_f32_16x16x128_f8f6f4 v[52:55], v[120:127], v[168:175], v[52:55] cbsz:1 blgp:1\n" \
    "\tv_cmp_eq_u32 vcc, 0x2007, %[vd]\n" \
    "\tv_add_u32 %[dp], 4, %[dp]\n" \
    "\tv_bfi_b32 %[dp], 63, %[dp], %[dbase]\n" \
    "\ts_sub_u32 %[n], %[n], 1\n" \
    "\ts_cmp_lg_u32 %[n], 2\n" \
    "\tv_mfma_f32_16x16x128_f8f6f4 v[56:59], v[128:135], v[168:175], v[56:59] cbsz:1 blgp:1\n" \
    "\ts_cbranch_scc1 3b\n" \
    "\ts_cbranch_vccnz 14f\n" \
    "24:\n" \
    "\tv_cmp_ne_u32 vcc, 0x200f, %[vd]\n" \
    "\ts_lshl_b32 s93, s92, 14\n" \
    "\ts_cbranch_vccnz 20b\n" \
    "\ts_add_u32 s93, s93, 0x4000\n" \
    "\ts_cmp_eq_u32 s93, 0x1c000\n" \
    "\ts_cselect_b32 s93, 0, s93\n" \
    "\ts_mov_b32 s94, s93\n" \
    "\tv_add_u32 %[wa0], s93, %[bo0]\n" \
    "\tv_add_u32 %[wa1], s93, %[bo1]\n" \
    "\ts_waitcnt lgkmcnt(0)\n" \
    "\ts_barrier\n" \
    "\tv_mfma_f32_16x16x128_f8f6f4 v[20:23], v[64:71], v[80:87], v[20:23] cbsz:1 blgp:1\n" \
    "\tds_read_b64_tr_b8 v[120:121], %[wa0] offset:0\n" \
    "\tds_read_b64_tr_b8 v[122:123], %[wa0] offset:1024\n" \
    "\tds_read_b64_tr_b8 v[124:125], %[wa0] offset:8192\n" \
    "\tds_read_b64_tr_b8 v[126:127], %[wa0] offset:9216\n" \
    "\ts_add_u32 s93, s93, 0x4000\n" \
    "\ts_cmp_eq_u32 s93, 0x1c000\n" \
    "\ts_cselect_b32 s93, 0, s93\n" \
    "\ts_mov_b32 s95, s93\n" \
    "\tv_mfma_f32_16x16x128_f8f6f4 v[24:27], v[72:79], v[80:87], v[24:27] cbsz:1 blgp:1\n" \
    "\tds_read_b64_tr_b8 v[128:129], %[wa1] offset:0\n" \
    "\tds_read_b64_tr_b8 v[130:131], %[wa1] offset:1024\n" \
    "\tds_read_b64_tr_b8 v[132:133], %[wa1] offset:8192\n" \
    "\tds_read_b64_tr_b8 v[134:135], %[wa1] offset:9216\n" \
    "\tv_add_u32 %[wa0], s93, %[bo0]\n" \
    "\tv_add_u32 %[wa1], s93, %[bo1]\n" \
    "\tv_mfma_f32_16x16x128_f8f6f4 v[28:31], v[64:71], v[88:95], v[28:31] cbsz:1 blgp:1\n" \
    "\tds_read_b64_tr_b8 v[136:137], %[wa0] offset:0\n" \
    "\tds_read_b64_tr_b8 v[138:139], %[wa0] offset:1024\n" \
    "\tds_read_b64_tr_b8 v[140:141], %[wa0] offset:8192\n" \
    "\tds_read_b64_tr_b8 v[142:143], %[wa0] offset:9216\n" \
    "\ts_add_u32 s93, s93, 0x4000\n" \
    "\ts_cmp_eq_u32 s93, 0x1c000\n" \
    "\ts_cselect_b32 s93, 0, s93\n" \
    "\ts_mov_b32 s96, s93\n" \
    "\tv_mfma_f32_16x16x128_f8f6f4 v[32:35], v[72:79], v[88:95], v[32:35] cbsz:1 blgp:1\n" \
    "\tds_read_b64_tr_b8 v[144:145], %[wa1] offset:0\n" \
    "\tds_read_b64_tr_b8 v[146:147], %[wa1] offset:1024\n" \
    "\tds_read_b64_tr_b8 v[148:149], %[wa1] offset:8192\n" \
    "\tds_read_b64_tr_b8 v[150:151], %[wa1] offset:9216\n" \
    "\tv_add_u32 %[wa0], s93, %[bo0]\n" \
    "\tv_add_u32 %[wa1], s93, %[bo1]\n" \
    "\tv_mfma_f32_16x16x128_f8f6f4 v[36:39], v[64:71], v[96:103], v[36:39] cbsz:1 blgp:1\n" \
    "\tds_read_b64_tr_b8 v[152:153], %[wa0] offset:0\n" \
    "\tds_read_b64_tr_b8 v[154:155], %[wa0] offset:1024\n" \
    "\tds_read_b64_tr_b8 v[156:157], %[wa0] offset:8192\n" \
    "\tds_read_b64_tr_b8 v[158:159], %[wa0] offset:9216\n" \
    "\ts_add_u32 s93, s93, 0x4000\n" \
    "\ts_cmp_eq_u32 s93, 0x1c000\n" \
    "\ts_cselect_b32 s93, 0, s93\n" \
    "\tv_mfma_f32_16x16x128_f8f6f4 v[40:43], v[72:79], v[96:103], v[40:43] cbsz:1 blgp:1\n" \
    "\tds_read_b64_tr_b8 v[160:161], %[wa1] offset:0\n" \
    "\tds_read_b64_tr_b8 v[162:163], %[wa1] offset:1024\n" \
    "\tds_read_b64_tr_b8 v[164:165], %[wa1] offset:8192\n" \
    "\tds_read_b64_tr_b8 v[166:167], %[wa1] offset:9216\n" \
    "\tv_add_u32 %[wa0], s93, %[bo0]\n" \
    "\tv_add_u32 %[wa1], s93, %[bo1]\n" \
    "\tv_mfma_f32_16x16x128_f8f6f4 v[44:47], v[64:71], v[104:111], v[44:47] cbsz:1 blgp:1\n" \
    "\tv_add_u32 %[ew0], s96, %[ew0]\n" \
    "\tv_add_u32 %[ew1], s96, %[ew1]\n" \
    "\tv_add_u32 %[er], s96, %[er]\n" \
    "\tv_subrev_u32 %[ao0], 4096, %[ao0]\n" \
    "\tv_subrev_u32 %[ao1], 4096, %[ao1]\n" \
    "\tv_mfma_f32_16x16x128_f8f6f4 v[48:51], v[72:79], v[104:111], v[48:51] cbsz:1 blgp:1\n" \
    "\ts_waitcnt lgkmcnt(0)\n" \
    "\ts_barrier\n" \
    "\tds_read_b128 v[206:209], %[ao0] offset:14336\n" \
    "\tds_read_b128 v[210:213], %[ao1] offset:14336\n" \
    "\tds_read_b128 v[214:217], %[ao0] offset:16384\n" \
    "\tds_read_b128 v[218:221], %[ao1] offset:16384\n" \
    "\tds_read_b128 v[222:225], %[ao0] offset:18432\n" \
    "\tds_read_b128 v[226:229], %[ao1] offset:18432\n" \
    "\tv_mfma_f32_16x16x128_f8f6f4 v[52:55], v[64:71], v[112:119], v[52:55] cbsz:1 blgp:1\n" \
    "\tds_read_b64_tr_b8 v[168:169], %[wa0] offset:0\n" \
    "\tds_read_b64_tr_b8 v[170:171], %[wa0] offset:1024\n" \
    "\tds_read_b64_tr_b8 v[172:173], %[wa0] offset:8192\n" \
    "\tds_read_b64_tr_b8 v[174:175], %[wa0] offset:9216\n" \
    "\tv_mfma_f32_16x16x128_f8f6f4 v[56:59], v[72:79], v[112:119], v[56:59] cbsz:1 blgp:1\n" \
    "\tds_read_b64_tr_b8 v[176:177], %[wa1] offset:0\n" \
    "\tds_read_b64_tr_b8 v[178:179], %[wa1] offset:1024\n" \
    "\tds_read_b64_tr_b8 v[180:181], %[wa1] offset:8192\n" \
    "\tds_read_b64_tr_b8 v[182:183], %[wa1] offset:9216\n" \
    "\ts_add_u32 s93, s93, 0x4000\n" \
    "\ts_cmp_eq_u32 s93, 0x1c000\n" \
    "\ts_cselect_b32 s93, 0, s93\n" \
    "\tv_add_u32 %[wa0], s93, %[bo0]\n" \
    "\tv_add_u32 %[wa1], s93, %[bo1]\n" \
    "\ts_waitcnt lgkmcnt(12)\n" \
    "\tv_mfma_f32_16x16x128_f8f6f4 v[20:23], v[120:127], v[206:213], v[20:23] cbsz:1 blgp:1\n" \
    "\tds_read_b128 v[96:99], %[ao0] offset:20480\n" \
    "\tds_read_b128 v[100:103], %[ao1] offset:20480\n" \
    "\tds_read_b64_tr_b8 v[184:185], %[wa0] offset:0\n" \
    "\tds_read_b64_tr_b8 v[186:187], %[wa0] offset:1024\n" \
    "\tds_read_b64_tr_b8 v[188:189], %[wa0] offset:8192\n" \
    "\tds_read_b64_tr_b8 v[190:191], %[wa0] offset:9216\n" \
    "\tv_mfma_f32_16x16x128_f8f6f4 v[24:27], v[128:135], v[206:213], v[24:27] cbsz:1 blgp:1\n" \
    "\tds_read_b64_tr_b8 v[192:193], %[wa1] offset:0\n" \
    "\tds_read_b64_tr_b8 v[194:195], %[wa1] offset:1024\n" \
    "\tds_read_b64_tr_b8 v[196:197], %[wa1] offset:8192\n" \
    "\tds_read_b64_tr_b8 v[198:199], %[wa1] offset:9216\n" \
    "\ts_waitcnt lgkmcnt(14)\n" \
    "\tv_mfma_f32_16x16x128_f8f6f4 v[20:23], v[136:143], v[214:221], v[20:23] cbsz:1 blgp:1\n" \
    "\tds_read_b128 v[104:107], %[ao0] offset:22528\n" \
    "\tds_read_b128 v[108:111], %[ao1] offset:22528\n" \
    "\tv_mfma_f32_16x16x128_f8f6f4 v[24:27], v[144:151], v[214:221], v[24:27] cbsz:1 blgp:1\n" \
    "\ts_add_u32 s93, s93, 0x4000\n" \
    "\ts_cmp_eq_u32 s93, 0x1c000\n" \
    "\ts_cselect_b32 s93, 0, s93\n" \
    "\tv_add_u32 %[wa0], s93, %[bo0]\n" \
    "\tv_add_u32 %[wa1], s93, %[bo1]\n" \
    "\ts_waitcnt lgkmcnt(14)\n" \
    "\tv_mfma_f32_16x16x128_f8f6f4 v[20:23], v[152:159], v[222:229], v[20:23] cbsz:1 blgp:1\n" \
    "\tds_read_b128 v[112:115], %[ao0] offset:0\n" \
    "\tds_read_b128 v[116:119], %[ao1] offset:0\n" \
    "\tv_mfma_f32_16x16x128_f8f6f4 v[24:27], v[160:167], v[222:229], v[24:27] cbsz:1 blgp:1\n" \
    "\ts_barrier\n" \
    "\ts_waitcnt lgkmcnt(14)\n" \
    "\ts_waitcnt lgkmcnt(12)\n" \
    "\tv_mfma_f32_16x16x128_f8f6f4 v[20:23], v[168:175], v[96:103], v[20:23] cbsz:1 blgp:1\n" \
    "\tds_read_b128 v[206:209], %[ao0] offset:2048\n" \
    "\tds_read_b128 v[210:213], %[ao1] offset:2048\n" \
    "\tds_read_b64_tr_b8 v[64:65], %[wa0] offset:0\n" \
    "\tds_read_b64_tr_b8 v[66:67], %[wa0] offset:1024\n" \
    "\tds_read_b64_tr_b8 v[68:69], %[wa0] offset:8192\n" \
    "\tds_read_b64_tr_b8 v[70:71], %[wa0] offset:9216\n" \
    "\tv_mfma_f32_16x16x128_f8f6f4 v[24:27], v[176:183], v[96:103], v[24:27] cbsz:1 blgp:1\n" \
    "\tds_read_b64_tr_b8 v[72:73], %[wa1] offset:0\n" \
    "\tds_read_b64_tr_b8 v[74:75], %[wa1] offset:1024\n" \
    "\tds_read_b64_tr_b8 v[76:77], %[wa1] offset:8192\n" \
    "\tds_read_b64_tr_b8 v[78:79], %[wa1] offset:9216\n" \
    "\ts_add_u32 s93, s93, 0x4000\n" \
    "\ts_cmp_eq_u32 s93, 0x1c000\n" \
    "\ts_cselect_b32 s93, 0, s93\n" \
    "\tv_add_u32 %[wa0], s93, %[bo0]\n" \
    "\tv_add_u32 %[wa1], s93, %[bo1]\n" \
    "\ts_waitcnt lgkmcnt(14)\n" \
    "\ts_waitcnt lgkmcnt(12)\n" \
    "\tv_mfma_f32_16x16x128_f8f6f4 v[20:23], v[184:191], v[104:111], v[20:23] cbsz:1 blgp:1\n" \
    "\tds_read_b128 v[214:217], %[ao0] offset:24576\n" \
    "\tds_read_b128 v[218:221], %[ao1] offset:24576\n" \
    "\tds_read_b64_tr_b8 v[80:81], %[wa0] offset:0\n" \
    "\tds_read_b64_tr_b8 v[82:83], %[wa0] offset:1024\n" \
    "\tds_read_b64_tr_b8 v[84:85], %[wa0] offset:8192\n" \
    "\tds_read_b64_tr_b8 v[86:87], %[wa0] offset:9216\n" \
    "\tv_mfma_f32_16x16x128_f8f6f4 v[24:27], v[192:199], v[104:111], v[24:27] cbsz:1 blgp:1\n" \
    "\tds_read_b64_tr_b8 v[88:89], %[wa1] offset:0\n" \
    "\tds_read_b64_tr_b8 v[90:91], %[wa1] offset:1024\n" \
    "\tds_read_b64_tr_b8 v[92:93], %[wa1] offset:8192\n" \
    "\tds_read_b64_tr_b8 v[94:95], %[wa1] offset:9216\n" \
    "\ts_waitcnt lgkmcnt(10)\n" \
    "\ts_waitcnt lgkmcnt(14)\n" \
    "\tv_mfma_f32_16x16x128_f8f6f4 v[20:23], v[64:71], v[112:119], v[20:23] cbsz:1 blgp:1\n" \
    "\tds_read_b128 v[222:225], %[ao0] offset:26624\n" \
    "\tds_read_b128 v[226:229], %[ao1] offset:26624\n" \
    "\tv_mfma_f32_16x16x128_f8f6f4 v[24:27], v[72:79], v[112:119], v[24:27] cbsz:1 blgp:1\n" \
    "\ts_barrier\n" \
    "\tds_read_b32 %[vd], %[stat]\n" \
    "\ts_waitcnt lgkmcnt(3)\n" \
    "\ts_waitcnt lgkmcnt(14)\n" \
    "\tv_mfma_f32_16x16x128_f8f6f4 v[20:23], v[80:87], v[206:213], v[20:23] cbsz:1 blgp:1\n" \
    "\tds_read_b128 v[96:99], %[ao0] offset:28672\n" \
    "\tds_read_b128 v[100:103], %[ao1] offset:28672\n" \
    "\tv_mfma_f32_16x16x128_f8f6f4 v[24:27], v[88:95], v[206:213], v[24:27] cbsz:1 blgp:1\n" \
    "\ts_waitcnt lgkmcnt(13)\n" \
    "\tv_mfma_f32_16x16x128_f8f6f4 v[28:31], v[120:127], v[214:221], v[28:31] cbsz:1 blgp:1\n" \
    "\tds_read_b128 v[104:107], %[ao0] offset:30720\n" \
    "\tds_read_b128 v[108:111], %[ao1] offset:30720\n" \
    "\ts_waitcnt lgkmcnt(4)\n" \
    "\tv_cmp_eq_u32 vcc, 0, %[vd]\n" \
    "\tv_mov_b32 v60, 0x7fc00000\n" \
    "\tv_mfma_f32_16x16x128_f8f6f4 v[32:35], v[128:135], v[214:221], v[32:35] cbsz:1 blgp:1\n" \
    "\tv_cndmask_b32 v60, v60, %[sc], vcc\n" \
    "\tv_mov_b32 v61, v60\n" \
    "\ts_waitcnt lgkmcnt(5)\n" \
    "\tv_mfma_f32_16x16x128_f8f6f4 v[28:31], v[136:143], v[222:229], v[28:31] cbsz:1 blgp:1\n" \
    "\tds_read_b128 v[112:115], %[ao0] offset:32768\n" \
    "\tds_read_b128 v[116:119], %[ao1] offset:32768\n" \
    "\tv_mfma_f32_16x16x128_f8f6f4 v[32:35], v[144:151], v[222:229], v[32:35] cbsz:1 blgp:1\n" \
    "\tv_mul_f32 v20, v60, v20\n" \
    "\tv_mul_f32 v21, v60, v21\n" \
    "\tv_mul_f32 v22, v60, v22\n" \
    "\tv_mul_f32 v23, v60, v23\n" \
    "\tds_write_b128 %[ew0], v[20:23]\n" \
    "\ts_waitcnt lgkmcnt(5)\n" \
    "\tv_mfma_f32_16x16x128_f8f6f4 v[28:31], v[152:159], v[96:103], v[28:31] cbsz:1 blgp:1\n" \
    "\tds_read_b128 v[206:209], %[ao0] offset:34816\n" \
    "\tds_read_b128 v[210:213], %[ao1] offset:34816\n" \
    "\tv_mfma_f32_16x16x128_f8f6f4 v[32:35], v[160:167], v[96:103], v[32:35] cbsz:1 blgp:1\n" \
    "\tv_mul_f32 v24, v60, v24\n" \
    "\tv_mul_f32 v25, v60, v25\n" \
    "\tv_mul_f32 v26, v60, v26\n" \
    "\tv_mul_f32 v27, v60, v27\n" \
    "\tds_write_b128 %[ew1], v[24:27]\n" \
    "\ts_waitcnt lgkmcnt(6)\n" \
    "\tv_mfma_f32_16x16x128_f8f6f4 v[28:31], v[168:175], v[104:111], v[28:31] cbsz:1 blgp:1\n" \
    "\tds_read_b128 v[214:217], %[ao0] offset:36864\n" \
    "\tds_read_b128 v[218:221], %[ao1] offset:36864\n" \
    "\tv_mfma_f32_16x16x128_f8f6f4 v[32:35], v[176:183], v[104:111], v[32:35] cbsz:1 blgp:1\n" \
    "\tds_read_b128 v[230:233], %[er] offset:0\n" \
    "\tds_read_b128 v[234:237], %[er] offset:1024\n" \
    "\ts_waitcnt lgkmcnt(8)\n" \
    "\tv_mfma_f32_16x16x128_f8f6f4 v[28:31], v[184:191], v[112:119], v[28:31] cbsz:1 blgp:1\n" \
    "\tds_read_b128 v[222:225], %[ao0] offset:38912\n" \
    "\tds_read_b128 v[226:229], %[ao1] offset:38912\n" \
    "\tv_mfma_f32_16x16x128_f8f6f4 v[32:35], v[192:199], v[112:119], v[32:35] cbsz:1 blgp:1\n" \
    "\ts_waitcnt lgkmcnt(7)\n" \
    "\tv_mfma_f32_16x16x128_f8f6f4 v[28:31], v[64:71], v[206:213], v[28:31] cbsz:1 blgp:1\n" \
    "\tds_read_b128 v[96:99], %[ao0] offset:40960\n" \
    "\tds_read_b128 v[100:103], %[ao1] offset:40960\n" \
    "\tv_mfma_f32_16x16x128_f8f6f4 v[32:35], v[72:79], v[206:213], v[32:35] cbsz:1 blgp:1\n" \
    "\ts_waitcnt lgkmcnt(4)\n" \
    "\tglobal_store_dwordx4 %[oo], v[230:233], s[90:91] sc1\n" \
    "\ts_add_u32 s90, s90, 0x20000\n" \
    "\ts_addc_u32 s91, s91, 0\n" \
    "\ts_waitcnt lgkmcnt(6)\n" \
    "\tv_mfma_f32_16x16x128_f8f6f4 v[28:31], v[80:87], v[214:221], v[28:31] cbsz:1 blgp:1\n" \
    "\tds_read_b128 v[104:107], %[ao0] offset:43008\n" \
    "\tds_read_b128 v[108:111], %[ao1] offset:43008\n" \
    "\tv_mfma_f32_16x16x128_f8f6f4 v[32:35], v[88:95], v[214:221], v[32:35] cbsz:1 blgp:1\n" \
    "\tglobal_store_dwordx4 %[oo], v[234:237], s[90:91] sc1\n" \
    "\ts_add_u32 s90, s90, 0x20000\n" \
    "\ts_addc_u32 s91, s91, 0\n" \
    "\ts_waitcnt lgkmcnt(4)\n" \
    "\tv_mfma_f32_16x16x128_f8f6f4 v[36:39], v[120:127], v[222:229], v[36:39] cbsz:1 blgp:1\n" \
    "\tds_read_b128 v[112:115], %[ao0] offset:4096\n" \
    "\tds_read_b128 v[116:119], %[ao1] offset:4096\n" \
    "\tv_mfma_f32_16x16x128_f8f6f4 v[40:43], v[128:135], v[222:229], v[40:43] cbsz:1 blgp:1\n" \
    "\ts_waitcnt lgkmcnt(4)\n" \
    "\tv_mfma_f32_16x16x128_f8f6f4 v[36:39], v[136:143], v[96:103], v[36:39] cbsz:1 blgp:1\n" \
    "\tds_read_b128 v[206:209], %[ao0] offset:6144\n" \
    "\tds_read_b128 v[210:213], %[ao1] offset:6144\n" \
    "\tv_mfma_f32_16x16x128_f8f6f4 v[40:43], v[144:151], v[96:103], v[40:43] cbsz:1 blgp:1\n" \
    "\tv_mul_f32 v28, v60, v28\n" \
    "\tv_mul_f32 v29, v60, v29\n" \
    "\tv_mul_f32 v30, v60, v30\n" \
    "\tv_mul_f32 v31, v60, v31\n" \
    "\tds_write_b128 %[ew0], v[28:31]\n" \
    "\ts_waitcnt lgkmcnt(5)\n" \
    "\tv_mfma_f32_16x16x128_f8f6f4 v[36:39], v[152:159], v[104:111], v[36:39] cbsz:1 blgp:1\n" \
    "\tds_read_b128 v[214:217], %[ao0] offset:8192\n" \
    "\tds_read_b128 v[218:221], %[ao1] offset:8192\n" \
    "\tv_mfma_f32_16x16x128_f8f6f4 v[40:43], v[160:167], v[104:111], v[40:43] cbsz:1 blgp:1\n" \
    "\tv_mul_f32 v32, v60, v32\n" \
    "\tv_mul_f32 v33, v60, v33\n" \
    "\tv_mul_f32 v34, v60, v34\n" \
    "\tv_mul_f32 v35, v60, v35\n" \
    "\tds_write_b128 %[ew1], v[32:35]\n" \
    "\ts_waitcnt lgkmcnt(6)\n" \
    "\tv_mfma_f32_16x16x128_f8f6f4 v[36:39], v[168:175], v[112:119], v[36:39] cbsz:1 blgp:1\n" \
    "\tds_read_b128 v[222:225], %[ao0] offset:10240\n" \
    "\tds_read_b128 v[226:229], %[ao1] offset:10240\n" \
    "\tv_mfma_f32_16x16x128_f8f6f4 v[40:43], v[176:183], v[112:119], v[40:43] cbsz:1 blgp:1\n" \
    "\tds_read_b128 v[238:241], %[er] offset:0\n" \
    "\tds_read_b128 v[242:245], %[er] offset:1024\n" \
    "\ts_waitcnt lgkmcnt(8)\n" \
    "\tv_mfma_f32_16x16x128_f8f6f4 v[36:39], v[184:191], v[206:213], v[36:39] cbsz:1 blgp:1\n" \
    "\ts_barrier\n" \
    "\ts_sub_u32 s93, s94, %[abase]\n" \
    "\tv_add_u32 %[wa0], s93, %[ao0]\n" \
    "\tv_add_u32 %[wa1], s93, %[ao1]\n" \
    "\tds_read_b128 v[96:99], %[wa0] offset:0\n" \
    "\tds_read_b128 v[100:103], %[wa1] offset:0\n" \
    "\tv_mfma_f32_16x16x128_f8f6f4 v[40:43], v[192:199], v[206:213], v[40:43] cbsz:1 blgp:1\n" \
    "\ts_waitcnt lgkmcnt(7)\n" \
    "\tv_mfma_f32_16x16x128_f8f6f4 v[36:39], v[64:71], v[214:221], v[36:39] cbsz:1 blgp:1\n" \
    "\tds_read_b128 v[104:107], %[wa0] offset:2048\n" \
    "\tds_read_b128 v[108:111], %[wa1] offset:2048\n" \
    "\tv_mfma_f32_16x16x128_f8f6f4 v[40:43], v[72:79], v[214:221], v[40:43] cbsz:1 blgp:1\n" \
    "\ts_waitcnt lgkmcnt(4)\n" \
    "\tglobal_store_dwordx4 %[oo], v[238:241], s[90:91] sc1\n" \
    "\ts_add_u32 s90, s90, 0x20000\n" \
    "\ts_addc_u32 s91, s91, 0\n" \
    "\ts_waitcnt lgkmcnt(6)\n" \
    "\tv_mfma_f32_16x16x128_f8f6f4 v[36:39], v[80:87], v[222:229], v[36:39] cbsz:1 blgp:1\n" \
    "\tds_read_b128 v[112:115], %[wa0] offset:4096\n" \
    "\tds_read_b128 v[116:119], %[wa1] offset:4096\n" \
    "\tv_mfma_f32_16x16x128_f8f6f4 v[40:43], v[88:95], v[222:229], v[40:43] cbsz:1 blgp:1\n" \
    "\tglobal_store_dwordx4 %[oo], v[242:245], s[90:91] sc1\n" \
    "\ts_add_u32 s90, s90, 0x20000\n" \
    "\ts_addc_u32 s91, s91, 0\n" \
    "\ts_waitcnt lgkmcnt(4)\n" \
    "\tv_mfma_f32_16x16x128_f8f6f4 v[44:47], v[120:127], v[96:103], v[44:47] cbsz:1 blgp:1\n" \
    "\tds_read_b128 v[206:209], %[wa0] offset:6144\n" \
    "\tds_read_b128 v[210:213], %[wa1] offset:6144\n" \
    "\tv_mfma_f32_16x16x128_f8f6f4 v[48:51], v[128:135], v[96:103], v[48:51] cbsz:1 blgp:1\n" \
    "\ts_waitcnt lgkmcnt(4)\n" \
    "\tv_mfma_f32_16x16x128_f8f6f4 v[44:47], v[136:143], v[104:111], v[44:47] cbsz:1 blgp:1\n" \
    "\tds_read_b128 v[214:217], %[wa0] offset:8192\n" \
    "\tds_read_b128 v[218:221], %[wa1] offset:8192\n" \
    "\tv_mfma_f32_16x16x128_f8f6f4 v[48:51], v[144:151], v[104:111], v[48:51] cbsz:1 blgp:1\n" \
    "\tv_mul_f32 v36, v60, v36\n" \
    "\tv_mul_f32 v37, v60, v37\n" \
    "\tv_mul_f32 v38, v60, v38\n" \
    "\tv_mul_f32 v39, v60, v39\n" \
    "\tds_write_b128 %[ew0], v[36:39]\n" \
    "\ts_waitcnt lgkmcnt(5)\n" \
    "\tv_mfma_f32_16x16x128_f8f6f4 v[44:47], v[152:159], v[112:119], v[44:47] cbsz:1 blgp:1\n" \
    "\tds_read_b128 v[222:225], %[wa0] offset:10240\n" \
    "\tds_read_b128 v[226:229], %[wa1] offset:10240\n" \
    "\tv_mfma_f32_16x16x128_f8f6f4 v[48:51], v[160:167], v[112:119], v[48:51] cbsz:1 blgp:1\n" \
    "\tv_mul_f32 v40, v60, v40\n" \
    "\tv_mul_f32 v41, v60, v41\n" \
    "\tv_mul_f32 v42, v60, v42\n" \
    "\tv_mul_f32 v43, v60, v43\n" \
    "\tds_write_b128 %[ew1], v[40:43]\n" \
    "\ts_waitcnt lgkmcnt(6)\n" \
    "\tv_mfma_f32_16x16x128_f8f6f4 v[44:47], v[168:175], v[206:213], v[44:47] cbsz:1 blgp:1\n" \
    "\tds_read_b128 v[96:99], %[wa0] offset:12288\n" \
    "\tds_read_b128 v[100:103], %[wa1] offset:12288\n" \
    "\tv_mfma_f32_16x16x128_f8f6f4 v[48:51], v[176:183], v[206:213], v[48:51] cbsz:1 blgp:1\n" \
    "\tds_read_b128 v[230:233], %[er] offset:0\n" \
    "\tds_read_b128 v[234:237], %[er] offset:1024\n" \
    "\ts_waitcnt lgkmcnt(8)\n" \
    "\tv_mfma_f32_16x16x128_f8f6f4 v[44:47], v[184:191], v[214:221], v[44:47] cbsz:1 blgp:1\n" \
    "\ts_sub_u32 s93, s95, %[abase]\n" \
    "\tv_add_u32 %[wa0], s93, %[ao0]\n" \
    "\tv_add_u32 %[wa1], s93, %[ao1]\n" \
    "\tds_read_b128 v[104:107], %[wa0] offset:0\n" \
    "\tds_read_b128 v[108:111], %[wa1] offset:0\n" \
    "\tv_mfma_f32_16x16x128_f8f6f4 v[48:51], v[192:199], v[214:221], v[48:51] cbsz:1 blgp:1\n" \
    "\ts_waitcnt lgkmcnt(7)\n" \
    "\tv_mfma_f32_16x16x128_f8f6f4 v[44:47], v[64:71], v[222:229], v[44:47] cbsz:1 blgp:1\n" \
    "\tds_read_b128 v[112:115], %[wa0] offset:2048\n" \
    "\tds_read_b128 v[116:119], %[wa1] offset:2048\n" \
    "\tv_mfma_f32_16x16x128_f8f6f4 v[48:51], v[72:79], v[222:229], v[48:51] cbsz:1 blgp:1\n" \
    "\ts_waitcnt lgkmcnt(4)\n" \
    "\tglobal_store_dwordx4 %[oo], v[230:233], s[90:91] sc1\n" \
    "\ts_add_u32 s90, s90, 0x20000\n" \
    "\ts_addc_u32 s91, s91, 0\n" \
    "\ts_waitcnt lgkmcnt(6)\n" \
    "\tv_mfma_f32_16x16x128_f8f6f4 v[44:47], v[80:87], v[96:103], v[44:47] cbsz:1 blgp:1\n" \
    "\tds_read_b128 v[206:209], %[wa0] offset:4096\n" \
    "\tds_read_b128 v[210:213], %[wa1] offset:4096\n" \
    "\tv_mfma_f32_16x16x128_f8f6f4 v[48:51], v[88:95], v[96:103], v[48:51] cbsz:1 blgp:1\n" \
    "\tglobal_store_dwordx4 %[oo], v[234:237], s[90:91] sc1\n" \
    "\ts_add_u32 s90, s90, 0x20000\n" \
    "\ts_addc_u32 s91, s91, 0\n" \
    "\ts_waitcnt lgkmcnt(4)\n" \
    "\tv_mfma_f32_16x16x128_f8f6f4 v[52:55], v[120:127], v[104:111], v[52:55] cbsz:1 blgp:1\n" \
    "\tds_read_b128 v[214:217], %[wa0] offset:6144\n" \
    "\tds_read_b128 v[218:221], %[wa1] offset:6144\n" \
    "\tv_mfma_f32_16x16x128_f8f6f4 v[56:59], v[128:135], v[104:111], v[56:59] cbsz:1 blgp:1\n" \
    "\ts_waitcnt lgkmcnt(4)\n" \
    "\tv_mfma_f32_16x16x128_f8f6f4 v[52:55], v[136:143], v[112:119], v[52:55] cbsz:1 blgp:1\n" \
    "\tds_read_b128 v[222:225], %[wa0] offset:8192\n" \
    "\tds_read_b128 v[226:229], %[wa1] offset:8192\n" \
    "\tv_mfma_f32_16x16x128_f8f6f4 v[56:59], v[144:151], v[112:119], v[56:59] cbsz:1 blgp:1\n" \
    "\tv_mul_f32 v44, v60, v44\n" \
    "\tv_mul_f32 v45, v60, v45\n" \
    "\tv_mul_f32 v46, v60, v46\n" \
    "\tv_mul_f32 v47, v60, v47\n" \
    "\tds_write_b128 %[ew0], v[44:47]\n" \
    "\ts_waitcnt lgkmcnt(5)\n" \
    "\tv_mfma_f32_16x16x128_f8f6f4 v[52:55], v[152:159], v[206:213], v[52:55] cbsz:1 blgp:1\n" \
    "\tds_read_b128 v[96:99], %[wa0] offset:10240\n" \
    "\tds_read_b128 v[100:103], %[wa1] offset:10240\n" \
    "\tv_mfma_f32_16x16x128_f8f6f4 v[56:59], v[160:167], v[206:213], v[56:59] cbsz:1 blgp:1\n" \
    "\tv_mul_f32 v48, v60, v48\n" \
    "\tv_mul_f32 v49, v60, v49\n" \
    "\tv_mul_f32 v50, v60, v50\n" \
    "\tv_mul_f32 v51, v60, v51\n" \
    "\tds_write_b128 %[ew1], v[48:51]\n" \
    "\ts_waitcnt lgkmcnt(6)\n" \
    "\tv_mfma_f32_16x16x128_f8f6f4 v[52:55], v[168:175], v[214:221], v[52:55] cbsz:1 blgp:1\n" \
    "\tds_read_b128 v[104:107], %[wa0] offset:12288\n" \
    "\tds_read_b128 v[108:111], %[wa1] offset:12288\n" \
    "\tv_mfma_f32_16x16x128_f8f6f4 v[56:59], v[176:183], v[214:221], v[56:59] cbsz:1 blgp:1\n" \
    "\tds_read_b128 v[238:241], %[er] offset:0\n" \
    "\tds_read_b128 v[242:245], %[er] offset:1024\n" \
    "\ts_waitcnt lgkmcnt(8)\n" \
    "\tv_mfma_f32_16x16x128_f8f6f4 v[52:55], v[184:191], v[222:229], v[52:55] cbsz:1 blgp:1\n" \
    "\tv_mfma_f32_16x16x128_f8f6f4 v[56:59], v[192:199], v[222:229], v[56:59] cbsz:1 blgp:1\n" \
    "\ts_waitcnt lgkmcnt(5)\n" \
    "\tv_mfma_f32_16x16x128_f8f6f4 v[52:55], v[64:71], v[96:103], v[52:55] cbsz:1 blgp:1\n" \
    "\tv_mfma_f32_16x16x128_f8f6f4 v[56:59], v[72:79], v[96:103], v[56:59] cbsz:1 blgp:1\n" \
    "\ts_waitcnt lgkmcnt(0)\n" \
    "\tglobal_store_dwordx4 %[oo], v[238:241], s[90:91] sc1\n" \
    "\ts_add_u32 s90, s90, 0x20000\n" \
    "\ts_addc_u32 s91, s91, 0\n" \
    "\ts_waitcnt lgkmcnt(2)\n" \
    "\tv_mfma_f32_16x16x128_f8f6f4 v[52:55], v[80:87], v[104:111], v[52:55] cbsz:1 blgp:1\n" \
    "\tv_mfma_f32_16x16x128_f8f6f4 v[56:59], v[88:95], v[104:111], v[56:59] cbsz:1 blgp:1\n" \
    "\tglobal_store_dwordx4 %[oo], v[242:245], s[90:91] sc1\n" \
    "\ts_add_u32 s90, s90, 0x20000\n" \
    "\ts_addc_u32 s91, s91, 0\n" \
    "\ts_nop 11\n" \
    "\tv_mul_f32 v52, v60, v52\n" \
    "\tv_mul_f32 v53, v60, v53\n" \
    "\tv_mul_f32 v54, v60, v54\n" \
    "\tv_mul_f32 v55, v60, v55\n" \
    "\tds_write_b128 %[ew0], v[52:55]\n" \
    "\tv_mul_f32 v56, v60, v56\n" \
    "\tv_mul_f32 v57, v60, v57\n" \
    "\tv_mul_f32 v58, v60, v58\n" \
    "\tv_mul_f32 v59, v60, v59\n" \
    "\tds_write_b128 %[ew1], v[56:59]\n" \
    "\tds_read_b128 v[230:233], %[er] offset:0\n" \
    "\tds_read_b128 v[234:237], %[er] offset:1024\n" \
    "\ts_waitcnt lgkmcnt(0)\n" \
    "\tglobal_store_dwordx4 %[oo], v[230:233], s[90:91] sc1\n" \
    "\ts_add_u32 s90, s90, 0x20000\n" \
    "\ts_addc_u32 s91, s91, 0\n" \
    "\tglobal_store_dwordx4 %[oo], v[234:237], s[90:91] sc1\n" \
    "\ts_add_u32 s90, s90, 0x20000\n" \
    "\ts_addc_u32 s91, s91, 0\n" \
    "\ts_endpgm\n" \
    "10:\n" \
    "\ts_waitcnt lgkmcnt(0)\n" \
    "\ts_barrier\n" \
    "\tds_read_b32 %[vd], %[dp]\n" \
    "\tv_add_u32 %[dp], 4, %[dp]\n" \
    "\tv_bfi_b32 %[dp], 63, %[dp], %[dbase]\n" \
    "\ts_waitcnt lgkmcnt(0)\n" \
    "\tv_cmp_eq_u32 vcc, 0x2007, %[vd]\n" \
    "\tv_and_b32 %[t], 7, %[vd]\n" \
    "\tv_lshl_add_u32 %[wa0], %[t], 14, %[bo0]\n" \
    "\tv_lshl_add_u32 %[wa1], %[t], 14, %[bo1]\n" \
    "\ts_cbranch_vccnz 10b\n" \
    "\ts_branch 20b\n" \
    "11:\n" \
    "\ts_waitcnt lgkmcnt(0)\n" \
    "\ts_barrier\n" \
    "\tds_read_b32 %[vd], %[dp]\n" \
    "\tv_add_u32 %[dp], 4, %[dp]\n" \
    "\tv_bfi_b32 %[dp], 63, %[dp], %[dbase]\n" \
    "\ts_waitcnt lgkmcnt(0)\n" \
    "\tv_cmp_eq_u32 vcc, 0x2007, %[vd]\n" \
    "\tv_and_b32 %[t], 7, %[vd]\n" \
    "\tv_lshl_add_u32 %[wa0], %[t], 14, %[bo0]\n" \
    "\tv_lshl_add_u32 %[wa1], %[t], 14, %[bo1]\n" \
    "\ts_cbranch_vccnz 11b\n" \
    "\ts_branch 21b\n" \
    "12:\n" \
    "\ts_waitcnt lgkmcnt(0)\n" \
    "\ts_barrier\n" \
    "\tds_read_b32 %[vd], %[dp]\n" \
    "\tv_add_u32 %[dp], 4, %[dp]\n" \
    "\tv_bfi_b32 %[dp], 63, %[dp], %[dbase]\n" \
    "\ts_waitcnt lgkmcnt(0)\n" \
    "\tv_cmp_eq_u32 vcc, 0x2007, %[vd]\n" \
    "\tv_and_b32 %[t], 7, %[vd]\n" \
    "\tv_lshl_add_u32 %[wa0], %[t], 14, %[bo0]\n" \
    "\tv_lshl_add_u32 %[wa1], %[t], 14, %[bo1]\n" \
    "\ts_cbranch_vccnz 12b\n" \
    "\ts_branch 22b\n" \
    "13:\n" \
    "\ts_waitcnt lgkmcnt(0)\n" \
    "\ts_barrier\n" \
    "\tds_read_b32 %[vd], %[dp]\n" \
    "\tv_add_u32 %[dp], 4, %[dp]\n" \
    "\tv_bfi_b32 %[dp], 63, %[dp], %[dbase]\n" \
    "\ts_waitcnt lgkmcnt(0)\n" \
    "\tv_cmp_eq_u32 vcc, 0x2007, %[vd]\n" \
    "\tv_and_b32 %[t], 7, %[vd]\n" \
    "\tv_lshl_add_u32 %[wa0], %[t], 14, %[bo0]\n" \
    "\tv_lshl_add_u32 %[wa1], %[t], 14, %[bo1]\n" \
    "\ts_cbranch_vccnz 13b\n" \
    "\ts_branch 23b\n" \
    "14:\n" \
    "\ts_waitcnt lgkmcnt(0)\n" \
    "\ts_barrier\n" \
    "\tds_read_b32 %[vd], %[dp]\n" \
    "\tv_add_u32 %[dp], 4, %[dp]\n" \
    "\tv_bfi_b32 %[dp], 63, %[dp], %[dbase]\n" \
    "\ts_waitcnt lgkmcnt(0)\n" \
    "\tv_cmp_eq_u32 vcc, 0x2007, %[vd]\n" \
    "\tv_and_b32 %[t], 7, %[vd]\n" \
    "\tv_lshl_add_u32 %[wa0], %[t], 14, %[bo0]\n" \
    "\tv_lshl_add_u32 %[wa1], %[t], 14, %[bo1]\n" \
    "\ts_cbranch_vccnz 14b\n" \
    "\ts_branch 24b\n" \
    "5:\n" \
    "\tds_read_b32 %[vd], %[stat]\n" \
    "\tv_add_u32 %[ew0], 118784, %[ew0]\n" \
    "\tv_add_u32 %[ew1], 118784, %[ew1]\n" \
    "\tv_add_u32 %[er], 118784, %[er]\n" \
    "\ts_waitcnt lgkmcnt(1)\n" \
    "\tv_mfma_f32_16x16x128_f8f6f4 v[20:23], v[120:127], v[136:143], v[20:23] cbsz:1 blgp:1\n" \
    "\tv_mfma_f32_16x16x128_f8f6f4 v[24:27], v[128:135], v[136:143], v[24:27] cbsz:1 blgp:1\n" \
    "\ts_waitcnt lgkmcnt(0)\n" \
    "\tv_cmp_eq_u32 vcc, 0, %[vd]\n" \
    "\tv_mov_b32 v60, 0x7fc00000\n" \
    "\tv_mfma_f32_16x16x128_f8f6f4 v[28:31], v[120:127], v[144:151], v[28:31] cbsz:1 blgp:1\n" \
    "\tv_cndmask_b32 v60, v60, %[sc], vcc\n" \
    "\tv_mov_b32 v61, v60\n" \
    "\tv_mfma_f32_16x16x128_f8f6f4 v[32:35], v[128:135], v[144:151], v[32:35] cbsz:1 blgp:1\n" \
    "\ts_nop 4\n" \
    "\tv_mul_f32 v20, v60, v20\n" \
    "\tv_mul_f32 v21, v60, v21\n" \
    "\tv_mul_f32 v22, v60, v22\n" \
    "\tv_mul_f32 v23, v60, v23\n" \
    "\tds_write_b128 %[ew0], v[20:23]\n" \
    "\tv_mul_f32 v24, v60, v24\n" \
    "\tv_mul_f32 v25, v60, v25\n" \
    "\tv_mul_f32 v26, v60, v26\n" \
    "\tv_mul_f32 v27, v60, v27\n" \
    "\tds_write_b128 %[ew1], v[24:27]\n" \
    "\tds_read_b128 v[64:67], %[er] offset:0\n" \
    "\tds_read_b128 v[68:71], %[er] offset:1024\n" \
    "\tv_mfma_f32_16x16x128_f8f6f4 v[36:39], v[120:127], v[152:159], v[36:39] cbsz:1 blgp:1\n" \
    "\tv_mfma_f32_16x16x128_f8f6f4 v[40:43], v[128:135], v[152:159], v[40:43] cbsz:1 blgp:1\n" \
    "\tv_mul_f32 v28, v60, v28\n" \
    "\tv_mul_f32 v29, v60, v29\n" \
    "\tv_mul_f32 v30, v60, v30\n" \
    "\tv_mul_f32 v31, v60, v31\n" \
    "\tds_write_b128 %[ew0], v[28:31]\n" \
    "\tv_mul_f32 v32, v60, v32\n" \
    "\tv_mul_f32 v33, v60, v33\n" \
    "\tv_mul_f32 v34, v60, v34\n" \
    "\tv_mul_f32 v35, v60, v35\n" \
    "\tds_write_b128 %[ew1], v[32:35]\n" \
    "\tds_read_b128 v[72:75], %[er] offset:0\n" \
    "\tds_read_b128 v[76:79], %[er] offset:1024\n" \
    "\ts_waitcnt lgkmcnt(4)\n" \
    "\tglobal_store_dwordx4 %[oo], v[64:67], s[90:91] sc1\n" \
    "\ts_add_u32 s90, s90, 0x20000\n" \
    "\ts_addc_u32 s91, s91, 0\n" \
    "\tglobal_store_dwordx4 %[oo], v[68:71], s[90:91] sc1\n" \
    "\ts_add_u32 s90, s90, 0x20000\n" \
    "\ts_addc_u32 s91, s91, 0\n" \
    "\tv_mfma_f32_16x16x128_f8f6f4 v[44:47], v[120:127], v[160:167], v[44:47] cbsz:1 blgp:1\n" \
    "\tv_mfma_f32_16x16x128_f8f6f4 v[48:51], v[128:135], v[160:167], v[48:51] cbsz:1 blgp:1\n" \
    "\tv_mul_f32 v36, v60, v36\n" \
    "\tv_mul_f32 v37, v60, v37\n" \
    "\tv_mul_f32 v38, v60, v38\n" \
    "\tv_mul_f32 v39, v60, v39\n" \
    "\tds_write_b128 %[ew0], v[36:39]\n" \
    "\tv_mul_f32 v40, v60, v40\n" \
    "\tv_mul_f32 v41, v60, v41\n" \
    "\tv_mul_f32 v42, v60, v42\n" \
    "\tv_mul_f32 v43, v60, v43\n" \
    "\tds_write_b128 %[ew1], v[40:43]\n" \
    "\tds_read_b128 v[64:67], %[er] offset:0\n" \
    "\tds_read_b128 v[68:71], %[er] offset:1024\n" \
    "\ts_waitcnt lgkmcnt(4)\n" \
    "\tglobal_store_dwordx4 %[oo], v[72:75], s[90:91] sc1\n" \
    "\ts_add_u32 s90, s90, 0x20000\n" \
    "\ts_addc_u32 s91, s91, 0\n" \
    "\tglobal_store_dwordx4 %[oo], v[76:79], s[90:91] sc1\n" \
    "\ts_add_u32 s90, s90, 0x20000\n" \
    "\ts_addc_u32 s91, s91, 0\n" \
    "\tv_mfma_f32_16x16x128_f8f6f4 v[52:55], v[120:127], v[168:175], v[52:55] cbsz:1 blgp:1\n" \
    "\tv_mfma_f32_16x16x128_f8f6f4 v[56:59], v[128:135], v[168:175], v[56:59] cbsz:1 blgp:1\n" \
    "\tv_mul_f32 v44, v60, v44\n" \
    "\tv_mul_f32 v45, v60, v45\n" \
    "\tv_mul_f32 v46, v60, v46\n" \
    "\tv_mul_f32 v47, v60, v47\n" \
    "\tds_write_b128 %[ew0], v[44:47]\n" \
    "\tv_mul_f32 v48, v60, v48\n" \
    "\tv_mul_f32 v49, v60, v49\n" \
    "\tv_mul_f32 v50, v60, v50\n" \
    "\tv_mul_f32 v51, v60, v51\n" \
    "\tds_write_b128 %[ew1], v[48:51]\n" \
    "\tds_read_b128 v[72:75], %[er] offset:0\n" \
    "\tds_read_b128 v[76:79], %[er] offset:1024\n" \
    "\ts_waitcnt lgkmcnt(4)\n" \
    "\tglobal_store_dwordx4 %[oo], v[64:67], s[90:91] sc1\n" \
    "\ts_add_u32 s90, s90, 0x20000\n" \
    "\ts_addc_u32 s91, s91, 0\n" \
    "\tglobal_store_dwordx4 %[oo], v[68:71], s[90:91] sc1\n" \
    "\ts_add_u32 s90, s90, 0x20000\n" \
    "\ts_addc_u32 s91, s91, 0\n" \
    "\tv_mul_f32 v52, v60, v52\n" \
    "\tv_mul_f32 v53, v60, v53\n" \
    "\tv_mul_f32 v54, v60, v54\n" \
    "\tv_mul_f32 v55, v60, v55\n" \
    "\tds_write_b128 %[ew0], v[52:55]\n" \
    "\tv_mul_f32 v56, v60, v56\n" \
    "\tv_mul_f32 v57, v60, v57\n" \
    "\tv_mul_f32 v58, v60, v58\n" \
    "\tv_mul_f32 v59, v60, v59\n" \
    "\tds_write_b128 %[ew1], v[56:59]\n" \
    "\tds_read_b128 v[64:67], %[er] offset:0\n" \
    "\tds_read_b128 v[68:71], %[er] offset:1024\n" \
    "\ts_waitcnt lgkmcnt(4)\n" \
    "\tglobal_store_dwordx4 %[oo], v[72:75], s[90:91] sc1\n" \
    "\ts_add_u32 s90, s90, 0x20000\n" \
    "\ts_addc_u32 s91, s91, 0\n" \
    "\tglobal_store_dwordx4 %[oo], v[76:79], s[90:91] sc1\n" \
    "\ts_add_u32 s90, s90, 0x20000\n" \
    "\ts_addc_u32 s91, s91, 0\n" \
    "\ts_waitcnt lgkmcnt(0)\n" \
    "\tglobal_store_dwordx4 %[oo], v[64:67], s[90:91] sc1\n" \
    "\ts_add_u32 s90, s90, 0x20000\n" \
    "\ts_addc_u32 s91, s91, 0\n" \
    "\tglobal_store_dwordx4 %[oo], v[68:71], s[90:91] sc1\n" \
    "\ts_add_u32 s90, s90, 0x20000\n" \
    "\ts_addc_u32 s91, s91, 0\n" \
    "\ts_endpgm\n" \
    ".La2a_end:\n"
#define V10(a) "v" #a "0", "v" #a "1", "v" #a "2", "v" #a "3", "v" #a "4", "v" #a "5", "v" #a "6", "v" #a "7", "v" #a "8", "v" #a "9"
#define CONS_CLOB "memory", "scc", "vcc", "s84", "s85", "s86", "s88", "s89", "s90", "s91", "s92", "s93", "s94", "s95", "s96", \
  V10(2), V10(3), V10(4), V10(5), V10(6), V10(7), V10(8), V10(9), V10(10), V10(11), V10(12), V10(13), V10(14), V10(15), V10(16), \
  V10(17), V10(18), V10(19), V10(20), V10(21), V10(22), V10(23), "v240", "v241", "v242", "v243", "v244", "v245"
template <int RT>
__device__ __forceinline__ void consumer(const Args& p, u32 lds0, int wave, int lane, int mt, int n0) {
  const u32 ctl = lds0 + LDS_CTL;
  if (wave == 1 && lane >= 32) *(volatile LAS u32*)(ctl + lane * 4) = (lane >= 48 && lane < 52) ? (u32)(lane - 45) * 0x101u : 0u;
  const float scale = *p.sx * *p.sw;
  const int g = lane >> 4, t = lane & 15;
  u32 aoff[2], boff[2];
  #pragma unroll
  for (int i = 0; i < 2; ++i) {
    aoff[i] = lds0 + LDS_A + 128 * t + 16 * ((g + 4 * i) ^ (t >> 1));
    boff[i] = lds0 + LDS_W + 128 * (16 * g + (t >> 1)) + 16 * ((2 * wave + i) ^ ((t >> 2) | ((g & 1) << 2))) + 8 * (t & 1);
  }
  u32 dp = ctl + C_DESC + 4, wa0 = boff[0] + 3 * 16384u, wa1 = boff[1] + 3 * 16384u, n = 8, vd, tmp;
  const u32 sb = lds0 + LDS_W + wave * 2048;
  u32 ew0 = sb + 128 * t + 16 * (g ^ (t & 7)), ew1 = ew0 ^ 64u, er = sb + 128 * (lane >> 3) + 16 * ((lane & 7) ^ (lane >> 3));
  const u32 oo = (u32)((lane >> 3) * GN + 4 * (lane & 7)) * 4;
  const float* ob = p.out + (size_t)row0_of(mt) * GN + n0 + wave * 32;
#define CONS_IN [bo0] "v"(boff[0]), [bo1] "v"(boff[1]), [stat] "v"(ctl + C_STAT), [dbase] "s"(ctl + C_DESC), \
                [oo] "v"(oo), [ob] "s"(ob), [sc] "v"(scale), [abase] "i"(LDS_PAD - LDS_W)
#define CONS_OUT [dp] "+&v"(dp), [wa0] "+&v"(wa0), [wa1] "+&v"(wa1), [n] "+&s"(n), [vd] "=&v"(vd), [t] "=&v"(tmp), \
                 [ao0] "+&v"(aoff[0]), [ao1] "+&v"(aoff[1]), [ew0] "+&v"(ew0), [ew1] "+&v"(ew1), [er] "+&v"(er)
  if constexpr (RT == 5) asm volatile(CONS_ASM5 : CONS_OUT : CONS_IN : CONS_CLOB);
  else asm volatile(CONS_ASM4 : CONS_OUT : CONS_IN : CONS_CLOB);
}

template <int RT>
__device__ __forceinline__ void aloader(const Args& p, u32 lds0, int lane, int b, int mt, int rot, int lr) {
  constexpr int NI = 2 * RT;
  const int rank = p.rank;
  char* const myws = p.myws;
  const u32 ctl = lds0 + LDS_CTL;
  u32* cntp = (u32*)(myws + OFF_CNT + (size_t)b * 128) + 6;
  asm volatile("global_load_dword v205, %0, off sc0 sc1" :: "v"(cntp) : "memory", "v205");
  const char* dtab;
  asm volatile("s_getpc_b64 %0\n\ts_branch 1f\n\t"
               ".long 0x800, 0x901, 0xa02, 0xb03, 0xc04, 0xd05, 0xe06, 0x2007\n\t"
               ".long 0x1000, 0x1101, 0x1202, 0x1303, 0x1404, 0x1505, 0x1606, 0x200f\n\t"
               ".long 0x1800, 0x1901, 0x1a02, 0x1b03, 0x1c04, 0x1d05, 0x1e06, 0x1f07\n\t"
               ".long 0x2000, 0x2101, 0x2202, 0x2303, 0x2404, 0x2505, 0x2606, 0x2707\n\t"
               "1:" : "=s"(dtab));
  dtab += 4;
  u32 off[10];
  #pragma unroll
  for (int q = 0; q < 10; ++q) off[q] = (u32)(8 * q + (lane >> 3)) * KS + 16 * ((lane & 7) ^ ((4 * q + (lane >> 4)) & 7)) - (q & 3) * 1024;
  const unsigned char* xl = p.x + (size_t)(rank * 512 + row0_of(mt)) * KS;
  const u32 d0 = lds0 + LDS_A;
  #pragma unroll
  for (int s = 0; s < 2; ++s) dma<NI>(off, xl + ((s + lr) & 3) * 128, d0 + s * PIECE);
  const int lq = lane < 32 ? lane & 7 : (lane - 32) % 7, lj = lane < 32 ? lane >> 3 : 3, lsrc = src_of(lq < 7 ? lq : 0, rank, rot);
  const u32 tab = (u32)(lsrc * 4 + lj) * PIECE;
  const u32 or0 = (u32)lane * 16, or1 = or0 + 4096, or2 = or0 + 8192;
  u32 c;
  asm volatile("s_waitcnt vmcnt(%1)\n\tv_readfirstlane_b32 %0, v205" : "=s"(c) : "i"(NI) : "memory", "v205");
  const u32 call = c + 1;
  const char* slot = myws + OFF_SLOT + (size_t)(call & 1) * SLOT_BYTES + (size_t)mt * 32 * PIECE;
  u32* tmo = (u32*)(myws + OFF_TMO);
  const u32 snap = ctl + C_FLAGS + lq * 16 + lj * 4;
  const u32 fo = (lq < 7 ? (u32)OFF_FLAG + (u32)(mt * 8 + lsrc) * FSTR : (u32)OFF_TMO) + lj * 4;
  const u32 t0 = (u32)__builtin_amdgcn_s_memrealtime();
  u32 mask = 0u;
  u32 next = 0x7Fu;
  u32 l1 = 0x40u;
  u32 dst = d0, dpa = ctl + C_DESC + 16, bad = 0;
  u32 slo = (u32)(u64)slot, shi = (u32)((u64)slot >> 32);
  u32 c_, x_, o_, nl_, nn_, lx, vD, vF = 0;
  bar();
  #pragma unroll
  for (int s = 2; s < 4; ++s) dma<NI>(off, xl + ((s + lr) & 3) * 128, d0 + s * PIECE);
  waitvm<2 * NI>();
  u32 l2 = 0x20u, l3 = 0x10u;
  u32 idle = 0;
  u32 lim = 0xFFFFFFu;
#define SHIFTL "s_mov_b32 %[l3], %[l2]\n\ts_mov_b32 %[l2], %[l1]\n\t"
#define SKEWED   \
    "s_and_b32 %[o], %[next], 0xffff\n\t" \
    "s_cselect_b32 %[o], %[next], 0\n\t" \
    "s_lshr_b32 %[o], %[o], 24\n\t" \
    "s_bcnt1_i32_b32 %[o], %[o]\n\t" \
    "s_cmp_ge_u32 %[o], 3\n\t" \
    "s_cselect_b32 %[lim], -1, %[lim]\n\t"
#define GL(V, OFF) "global_load_lds_dwordx4 " V ", s[80:81]" OFF " sc0 sc1\n\t"
#define PICK \
    "s_andn2_b32 %[c], %[mask], %[l1]\n\t" \
    "s_and_b32 %[c], %[c], %[next]\n\t" \
    "s_and_b32 %[c], %[c], %[lim]\n\t" \
    "s_or_b32 %[x], %[l2], %[l3]\n\t" \
    "s_andn2_b32 %[x], %[c], %[x]\n\t" \
    "s_cselect_b32 %[c], %[x], %[c]\n\t" \
    "s_ff1_i32_b32 %[x], %[c]\n\t" \
    "v_readlane_b32 %[o], %[tab], %[x]\n\t"
#define PICK2 \
    "s_lshl_b32 %[nl], 0x100, %[x]\n\t" \
    "s_lshl_b32 %[nn], 0x101, %[x]\n\t" \
    "v_lshlrev_b32 %[vD], 2, %[x]\n\t" \
    "s_add_u32 s82, %[slo], %[o]\n\t" \
    "s_addc_u32 s83, %[shi], 0\n\t"
#define NEXTDPA \
    "s_add_u32 %[dpa], %[dpa], 4\n\t" \
    "s_or_b32 %[dpa], %[dpa], 0xc0\n\t" \
    "s_bitset0_b32 %[dpa], 8\n\t"
#define NEXTDST \
    "s_add_u32 %[dst], %[dst], %[piece]\n\t" \
    "s_cmp_eq_u32 %[dst], %[dend]\n\t" \
    "s_cselect_b32 %[dst], %[d0], %[dst]\n\t" \
    "s_cmp_lg_u32 %[c], 0\n\t"
#define ARM \
    "s_mov_b64 s[80:81], s[82:83]\n\t" \
    "s_mov_b32 m0, %[dpa]\n\t" \
    "s_mov_b64 exec, 1\n\t"
#define ALOOP(TAILDMA) asm volatile( \
    "s_mov_b64 vcc, 0\n\t" \
    "s_branch 3f\n" \
    "1:\n\t" \
    "s_waitcnt vmcnt(%[n2])\n\t" \
    "s_barrier\n" \
    "2:\n\t" \
    "global_load_lds_dword %[vD], %[dtab]\n\t" \
    "s_mov_b32 m0, %[dst]\n\t" \
    "s_mov_b64 exec, -1\n\t" \
    GL("%[o0]", "") \
    "s_waitcnt lgkmcnt(0)\n\t" \
    "v_cmp_le_u32 s[84:85], %[call], %[vF]\n\t" \
    SHIFTL \
    "s_mov_b32 %[l1], %[nl]\n\t" \
    "s_xor_b32 %[next], %[next], %[nn]\n\t" \
    "s_or_b32 %[mask], %[mask], s84\n\t" \
    GL("%[o0]", " offset:1024") \
    "s_mov_b32 %[lx], %[x]\n\t" \
    "ds_read_b32 %[vF], %[snap]\n\t" \
    GL("%[o0]", " offset:2048") \
    PICK \
    GL("%[o0]", " offset:3072") \
    "s_add_u32 m0, m0, 0x1000\n\t" \
    PICK2 \
    GL("%[o1]", "") \
    NEXTDPA \
    GL("%[o1]", " offset:1024") \
    SKEWED \
    GL("%[o1]", " offset:2048") \
    TAILDMA \
    ARM \
    "s_cbranch_vccnz 5f\n" \
    "6:\n\t" \
    "s_cbranch_scc1 1b\n\t" \
    "s_and_b32 %[c], %[next], %[mask]\n\t" \
    "s_cmp_eq_u32 %[c], 0x7f000000\n\t" \
    "v_mov_b32 %[vD], 60\n\t" \
    "s_waitcnt vmcnt(%[n2])\n\t" \
    "s_cbranch_scc1 11f\n\t" \
    "s_mov_b64 exec, -1\n" \
    "3:\n\t"                        \
    "s_barrier\n\t" \
    "ds_read_b32 %[vF], %[snap]\n\t" \
    "s_memrealtime s[86:87]\n\t" \
    "s_waitcnt lgkmcnt(0)\n\t" \
    "v_cmp_le_u32 s[84:85], %[call], %[vF]\n\t" \
    "s_or_b32 %[mask], %[mask], s84\n" \
    "14:\n\t" \
    "s_cmp_eq_u32 %[next], 0\n\t" \
    "s_cbranch_scc1 13f\n\t" \
    "s_and_b32 %[c], %[next], %[mask]\n\t" \
    "s_cmp_eq_u32 %[c], 0x7f000000\n\t"            \
    "s_cbranch_scc1 9f\n\t" \
    "s_and_b32 %[x], %[c], 0xffffff\n\t"           \
    "s_cselect_b32 %[c], 0, %[c]\n\t" \
    "s_and_b32 %[x], %[idle], vcc_lo\n\t" \
    "s_cmp_ge_u32 %[x], 1\n\t" \
    "s_cselect_b32 %[c], %[c], 0\n\t" \
    "s_bcnt1_i32_b32 %[c], %[c]\n\t" \
    "s_cmp_ge_u32 %[c], 2\n\t" \
    "s_cselect_b32 %[lim], -1, %[lim]\n" \
    "4:\n\t" \
    PICK PICK2 ARM \
    "s_cmp_lg_u32 %[c], 0\n\t" \
    "s_cbranch_scc1 2b\n\t" \
    "s_sub_u32 s86, s86, %[t0]\n\t" \
    "s_cmp_gt_u32 s86, %[dl]\n\t" \
    "s_cbranch_scc1 8f\n\t" \
    "s_bitcmp1_b32 %[mask], 7\n\t" \
    "s_cbranch_scc1 8f\n\t" \
    "s_and_b32 %[x], %[idle], vcc_lo\n\t" \
    "s_cmp_ge_u32 %[x], 64\n\t" \
    "s_cbranch_scc1 15f\n" \
    "7:\n\t" \
    "v_mov_b32 %[vD], 28\n\t" \
    SHIFTL \
    "s_mov_b32 %[l1], 0\n\t" \
    "global_load_lds_dword %[vD], %[dtab]\n\t" \
    "s_mov_b64 exec, -1\n\t" \
    NEXTDPA \
    "s_waitcnt vmcnt(%[n1])\n\t" \
    "s_cbranch_vccz 10f\n\t" \
    "s_waitcnt vmcnt(1)\n" \
    "10:\n\t" \
    "s_and_b32 %[idle], %[idle], vcc_lo\n\t"        \
    "s_add_u32 %[idle], %[idle], 1\n\t" \
    "s_mov_b64 vcc, -1\n\t" \
    "s_branch 3b\n" \
    "15:\n\t"                       \
    "s_mov_b64 exec, 0xffffffff\n\t" \
    "s_sleep 2\n\t" \
    "global_load_dword %[vF], %[fo], %[ws] sc0 sc1\n\t" \
    "s_memrealtime s[86:87]\n\t" \
    "s_waitcnt vmcnt(0) lgkmcnt(0)\n\t" \
    "v_cmp_le_u32 s[84:85], %[call], %[vF]\n\t" \
    "s_mov_b64 exec, -1\n\t" \
    "s_or_b32 %[mask], %[mask], s84\n\t" \
    "s_branch 14b\n" \
    "5:\n\t"                        \
    "s_waitcnt vmcnt(%[n1])\n\t" \
    "s_mov_b64 vcc, 0\n\t" \
    "s_branch 6b\n" \
    "8:\n\t"                        \
    "s_cmp_lg_u32 %[bad], 0\n\t" \
    "s_cbranch_scc1 7b\n\t" \
    "s_mov_b32 %[bad], 1\n\t" \
    "v_mov_b32 %[vD], 0\n\t" \
    "v_mov_b32 %[vF], %[call]\n\t" \
    "global_store_dword %[vD], %[vF], %[tmo] sc1\n\t" \
    "s_mov_b64 exec, -1\n\t" \
    "s_mov_b32 %[mask], -1\n\t" \
    "s_mov_b32 %[slo], %[xlo]\n\t" \
    "s_mov_b32 %[shi], %[xhi]\n\t" \
    "s_waitcnt vmcnt(0)\n\t" \
    "s_branch 4b\n" \
    "9:\n\t" \
    "s_mov_b32 m0, %[dpa]\n\t" \
    "v_mov_b32 %[vD], 60\n\t" \
    "s_mov_b64 exec, 1\n\t" \
    "s_branch 12f\n" \
    "11:\n\t" \
    "s_barrier\n" \
    "12:\n\t" \
    "global_load_lds_dword %[vD], %[dtab]\n\t" \
    "s_mov_b64 exec, -1\n\t" \
    "s_mov_b32 %[c], vcc_lo\n" \
    "13:\n\t" \
    "s_waitcnt lgkmcnt(0)" \
    : [mask] "+&s"(mask), [next] "+&s"(next), [l1] "+&s"(l1), [l2] "+&s"(l2), [l3] "+&s"(l3), [lim] "+&s"(lim), [idle] "+&s"(idle), [dst] "+&s"(dst), [dpa] "+&s"(dpa), [bad] "+&s"(bad), [slo] "+&s"(slo), [shi] "+&s"(shi), \
      [c] "=&s"(c_), [x] "=&s"(x_), [o] "=&s"(o_), [nl] "=&s"(nl_), [nn] "=&s"(nn_), [lx] "=&s"(lx), [vD] "=&v"(vD), [vF] "+&v"(vF) \
    : [o0] "v"(or0), [o1] "v"(or1), [o2] "v"(or2), [tab] "v"(tab), [snap] "v"(snap), [call] "s"(call), [t0] "s"(t0), [fo] "v"(fo), [ws] "s"(myws), \
      [d0] "s"(d0), [dend] "s"(d0 + NAS * PIECE), [dtab] "s"(dtab), [tmo] "s"(tmo), [xlo] "s"((u32)(u64)p.x), [xhi] "s"((u32)((u64)p.x >> 32)), \
      [piece] "i"(PIECE), [n2] "i"(2 * NI + 1), [n1] "i"(NI + 1), [dl] "i"((u32)DEADLINE) \
    : "memory", "scc", "vcc", "s80", "s81", "s82", "s83", "s84", "s85", "s86", "s87")
  if constexpr (NI == 10)
    ALOOP(GL("%[o1]", " offset:3072") "s_add_u32 m0, m0, 0x1000\n\t" NEXTDST GL("%[o2]", "") GL("%[o2]", " offset:1024"));
  else
    ALOOP(NEXTDST GL("%[o1]", " offset:3072"));
#undef GL
  next = rfl(next); slo = rfl(slo); shi = rfl(shi); lx = rfl(lx); bad = rfl(bad); c_ = rfl(c_);
  if (next == 0) {
    u32 tv;
    asm volatile("global_load_dword %0, %1, %2 sc1\n\ts_waitcnt vmcnt(%3)\n\ts_barrier\n\ts_waitcnt vmcnt(0)"
                 : "=&v"(tv) : "v"(0u), "s"(tmo), "i"(NI + 1) : "memory");
    if (bad || rfl(tv) == call) { *(volatile LAS u32*)(ctl + C_STAT) = 1u; asm volatile("s_waitcnt lgkmcnt(0)" ::: "memory"); }
    bar();
    if (lane == 0) __hip_atomic_store(cntp, call, __ATOMIC_RELAXED, __HIP_MEMORY_SCOPE_SYSTEM);
    return;
  }
  const char* kb[7];
  const u32 ql = lx & 7u;
  const u32 pad = lds0 + LDS_PAD;
  auto blk = [&](u32 r, int k, u32 ub, int uo) __attribute__((always_inline)) {
    asm volatile("s_add_u32 m0, %1, %3\n\ts_nop 0\n\tglobal_load_lds_dwordx4 %0, %2 sc0 sc1\n\tglobal_load_lds_dwordx4 %0, %2 offset:1024 sc0 sc1"
      :: "v"(or0 + r * 2048), "s"(ub), "s"(kb[k]), "i"(uo) : "memory", "scc");
  };
  #pragma unroll
  for (int k = 0; k < 7; ++k) {
    kb[k] = (const char*)(((u64)shi << 32) | slo) + (u32)__builtin_amdgcn_readlane(tab, 33 + k + ql);
    blk(0, k, pad, k < 5 ? 4096 + PIECE + 2048 * k : 2048 * (k - 5));
    __builtin_amdgcn_sched_barrier(0);
  }
  const u32 q0 = ql == 6 ? 0 : ql + 1, q1 = q0 == 6 ? 0 : q0 + 1;
  waitvm<15 + NI>();
  if (c_) waitvm<15>();
  bar();
  #pragma unroll
  for (int k = 0; k < 5; ++k) blk(1, k, pad, 4096 + 2 * PIECE + 2048 * k);
  waitvm<24>(); bar();
  #pragma unroll
  for (int k = 5; k < 10; ++k) blk(1 + k / 7, k % 7, pad, 4096 + 2 * PIECE + 2048 * k);
  bar();
  asm volatile("global_load_dword v205, %0, %1 sc1" :: "v"(0u), "s"(tmo) : "memory", "v205");
  #pragma unroll
  for (int k = 3; k < 7; ++k) blk(2, k, pad, 4096 + 2048 * (k - 3));
  waitvm<29>(); bar();
  #pragma unroll
  for (int k = 0; k < 7; ++k) blk(3, k, lds0 + LDS_W + (q0 << 14), 2048 * k);
  waitvm<29>(); bar();
  if (RT == 5) {
    #pragma unroll
    for (int k = 0; k < 7; ++k) blk(4, k, lds0 + LDS_W + (q1 << 14), 2048 * k);
  }
  u32 tv;
  asm volatile("s_waitcnt vmcnt(%1)\n\tv_readfirstlane_b32 %0, v205" : "=s"(tv) : "i"(RT == 5 ? 28 : 14) : "memory", "v205");
  if (bad || tv == call) { *(volatile LAS u32*)(ctl + C_STAT) = 1u; asm volatile("s_waitcnt lgkmcnt(0)" ::: "memory"); }
  bar();
  waitvm<0>(); bar();
  if (lane == 0) __hip_atomic_store(cntp, call, __ATOMIC_RELAXED, __HIP_MEMORY_SCOPE_SYSTEM);
}
}

__global__ __launch_bounds__(512, 2) void a2a_gemm_kernel(Args p) {
  extern __shared__ __attribute__((aligned(16))) char smem[];
  {
    u32 co, cs;
    asm volatile("v_readfirstlane_b32 %1, %2\n\ts_cmp_lg_u32 %1, 64\n\ts_cbranch_scc1 1f\n\ts_lshr_b32 %1, %3, 3\n\ts_sub_u32 %1, %1, 4\n\t"
                 "s_cmp_gt_u32 %1, 2\n\ts_cbranch_scc1 1f\n\ts_sub_u32 %1, %1, 1\n\ts_lshl_b32 %1, %1, 13\n\t"
                 "s_getpc_b64 s[92:93]\n9:\n\tv_lshl_add_u32 %0, %2, 7, %1\n\tv_min_u32 %0, .La2a_end - 9b - 4, %0\n\tglobal_load_dword v204, %0, s[92:93]\n1:"
                 : "=&v"(co), "=&s"(cs) : "v"(threadIdx.x), "s"(blockIdx.x) : "memory", "scc", "s92", "s93", "v204");
  }
  const u32 lds0 = (u32)(size_t)(LAS char*)smem;
  const int b = blockIdx.x, lane = threadIdx.x & 63;
  const int wave = rfl(threadIdx.x >> 6);
  const int rank = p.rank;
  char* const myws = p.myws;
  const unsigned char* const px = p.x;
  const int cg = (b & 3) * 2 + ((b >> 2) & 1);

  if (__builtin_expect(b < 32, 1)) {
    const u32 sub = (u32)b >> 3, link = ((u32)b + 7u - sub) & 7u;
    const u64 t0 = __builtin_amdgcn_s_memrealtime();
    const u32 ps = (u32)wave * 4 + sub, pj = (ps * 37) >> 8, pk = ps - pj * 7 + link, pmt = pk >= 7 ? pk - 7 : pk;
    const u32 r0 = 64 * pmt + 16 * min(pmt, 4u);
    const bool big = pmt < 4;
    u32 o[10];
    o[0] = (r0 + (u32)(lane >> 3)) * KS + pj * 128 + 16 * ((lane & 7) ^ (lane >> 4));
    o[1] = (o[0] ^ 64u) + 8 * KS;
    #pragma unroll
    for (int i = 2; i < 10; ++i) o[i] = o[i & 1] + (i < 8 || big ? i >> 1 : 3) * 16 * KS;
    const u32 cnto = (u32)OFF_CNT + (u32)b * 128 + (u32)wave * 4;
    asm volatile("" :: "v"(o[0]), "v"(o[1]), "v"(o[2]), "v"(o[3]), "v"(o[4]), "v"(o[5]), "v"(o[6]), "v"(o[7]), "v"(o[8]), "v"(o[9]), "v"(cnto));
    if (wave == 7) {
      while (__builtin_amdgcn_s_memrealtime() < t0 + (link < 7u ? CPFT : 300)) __builtin_amdgcn_s_sleep(8);
      const u32 cb = (3u - (u32)((size_t)p.w >> 7) - (u32)cg * 4) & 7u;
      #pragma unroll 1
      for (u32 i = (cb >= 4u ? 12 * PFR : 3 * PFS + 9 * PFR) + sub; i < 168; i += 4) touch_w(p.w + cg * 512, rank, i, lane);
      waitvm<0>();
      return;
    }
    if (link >= 7u) return;
    for (int z = 1; z < wave; ++z) __builtin_amdgcn_s_sleep(LDSLEEP);
    const unsigned char* xs = px + (size_t)(((u32)rank + 1 + link) & 7u) * 512 * KS;
    u32x4 v[10];
    u32 c;
    asm volatile(
      "global_load_dwordx4 %0, %13, %23\n\t"
      "global_load_dwordx4 %1, %14, %23\n\t"
      "global_load_dwordx4 %2, %15, %23\n\t"
      "global_load_dwordx4 %3, %16, %23\n\t"
      "global_load_dwordx4 %4, %17, %23\n\t"
      "global_load_dwordx4 %5, %18, %23\n\t"
      "global_load_dwordx4 %6, %19, %23\n\t"
      "global_load_dwordx4 %7, %20, %23\n\t"
      "global_load_dwordx4 %8, %21, %23\n\t"
      "global_load_dwordx4 %9, %22, %23\n\t"
      "global_load_dword %10, %11, %12 sc0 sc1"
      : "=&v"(v[0]), "=&v"(v[1]), "=&v"(v[2]), "=&v"(v[3]), "=&v"(v[4]), "=&v"(v[5]), "=&v"(v[6]), "=&v"(v[7]), "=&v"(v[8]), "=&v"(v[9]), "=&v"(c)
      : "v"(cnto), "s"(myws), "v"(o[0]), "v"(o[1]), "v"(o[2]), "v"(o[3]), "v"(o[4]), "v"(o[5]), "v"(o[6]), "v"(o[7]), "v"(o[8]), "v"(o[9]),
        "s"(xs) : "memory");
    char* dws = p.peer[link];
    const char* d = dws + OFF_SLOT + (size_t)((pmt * 8 + rank) * 4 + pj) * PIECE;
    u32* fp = (u32*)(dws + OFF_FLAG + (size_t)(pmt * 8 + rank) * FSTR) + pj;
    asm volatile(
      "1:\n\t"
      "s_memrealtime s[84:85]\n\t"
      "s_waitcnt lgkmcnt(0)\n\t"
      "s_sub_u32 s84, s84, %11\n\t"
      "s_cmp_ge_i32 s84, 0\n\t"
      "s_cbranch_scc1 2f\n\t"
      "s_sleep 1\n\t"
      "s_branch 1b\n"
      "2:\n\t"
      "s_waitcnt vmcnt(0)"
      : "+v"(v[0]), "+v"(v[1]), "+v"(v[2]), "+v"(v[3]), "+v"(v[4]), "+v"(v[5]), "+v"(v[6]), "+v"(v[7]), "+v"(v[8]), "+v"(v[9]), "+v"(c)
      : "s"((u32)t0 + (TSTART + ps * SLOTNS) / 10), "s"(d), "s"(d + 4096), "s"(d + 8192), "s"(fp), "v"((u32)lane * 16) : "memory", "scc", "s84", "s85");
    const u32 call = rfl(c) + 1;
    const u32 so = ((c + 1) & 1u) * (u32)SLOT_BYTES + (u32)lane * 16;
#define PUSH(I, B, OFF) asm volatile("global_store_dwordx4 %0, %1, %2 offset:" #OFF " sc0 sc1\n\ts_nop 1" :: "v"(so), "v"(v[I]), "s"(d + (B)) : "memory")
    PUSH(0, 0, 0); PUSH(1, 0, 1024); PUSH(2, 0, 2048); PUSH(3, 0, 3072);
    PUSH(4, 4096, 0); PUSH(5, 4096, 1024); PUSH(6, 4096, 2048); PUSH(7, 4096, 3072);
    if (big) { PUSH(8, 8192, 0); PUSH(9, 8192, 1024); }
#undef PUSH
    waitvm<0>();
    if (lane == 0) {
      __hip_atomic_store(fp, call, __ATOMIC_RELAXED, __HIP_MEMORY_SCOPE_SYSTEM);
      __hip_atomic_store((u32*)(myws + cnto), call, __ATOMIC_RELAXED, __HIP_MEMORY_SCOPE_SYSTEM);
    }
    return;
  }

  const int gi = (b >> 3) - 4;
  if (wave == 0) {
    const u32 sub = (u32)gi & 3u, link = ((u32)b + 7u - sub) & 7u;
    const u32 pj = ((u32)gi * 37) >> 8, pk = (u32)gi - pj * 7 + link, pmt = pk >= 7 ? pk - 7 : pk;
    const u32 xo = (64 * pmt + 16 * min(pmt, 4u) + (u32)lane) * KS + pj * 128;
    if (link < 7u) {
      const unsigned char* xs = px + (size_t)(((u32)rank + 1 + link) & 7u) * 512 * KS;
      touch(xo, xs);
      if (pmt < 4) touch(xo + (64 - (u32)(lane & 48)) * KS, xs);
    }
  }
  const int ntl = (gi * 37) >> 8, mt = gi - ntl * 7;
  const int n0 = (cg * 4 + ntl) * 128;
  const int rot = 6 - mt;
  const int lr = (mt + ntl) & 3;
  if (wave >= 4) __builtin_amdgcn_s_sleep(GDELAY);
  if (wave < 4) {
    if (mt < 4) consumer<5>(p, lds0, wave, lane, mt, n0); else consumer<4>(p, lds0, wave, lane, mt, n0);
  } else if (wave < 6) {
    const int h = wave - 4;
    u32 off[10];
    #pragma unroll
    for (int q = 0; q < 8; ++q) off[q] = (u32)(8 * q + (lane >> 3)) * GN + 16 * ((lane & 7) ^ ((lane >> 4) | ((q & 2) << 1))) - (q & 3) * 1024;
    off[8] = off[9] = 0;
    const unsigned char* wb = p.w + (size_t)(64 * h) * GN + n0;
    const u32 d0 = lds0 + LDS_W + h * 8192u;
    const unsigned char* wl = wb + (size_t)(rank * KS) * GN;
    auto local = [&](int s) { dma<8>(off, wl + (size_t)(((s + lr) & 3) * 128) * GN, d0 + (3 + s) * 16384u); };
    local(0);
    dma<8>(off, wb + (size_t)(src_of(0, rank, rot) * KS) * GN, d0);
    local(1);
    const u32 tab = (u32)(src_of((lane & 7) < 7 ? (lane & 7) : 0, rank, rot) * KS + ((lane >> 3) & 3) * 128) * GN;
    const unsigned char* w1 = wb + (u32)__builtin_amdgcn_readlane(tab, 1);
    const unsigned char* w2 = wb + (u32)__builtin_amdgcn_readlane(tab, 2);
    const unsigned char* w3 = wb + (u32)__builtin_amdgcn_readlane(tab, 3);
    const unsigned char* w4 = wb + (u32)__builtin_amdgcn_readlane(tab, 4);
    asm volatile("" :: "s"(w1), "s"(w2), "s"(w3), "s"(w4));
    waitvm<16>(); bar();
    local(2); local(3);
    waitvm<16>(); bar();
    dma<8>(off, w1, d0 + 1 * 16384u);
    dma<8>(off, w2, d0 + 2 * 16384u);
    waitvm<24>(); bar();
    dma<8>(off, w3, d0 + 3 * 16384u);
    dma<8>(off, w4, d0 + 4 * 16384u);
    waitvm<32>(); bar();
    u32 vdp = lds0 + LDS_CTL + C_DESC + 16, vd;
    u32 d1 = 0x606u;
    u32 left = 23, q_, m_, nm_, un_ = 0;
    const unsigned char* b0 = wb + (u32)__builtin_amdgcn_readlane(tab, 5);
#define GL(V, OFF) "global_load_lds_dwordx4 " V ", s[80:81]" OFF "\n\t"
#define RDDESC \
      "ds_read_b32 %[vd], %[vdp]\n\t" \
      "v_add_u32 %[vdp], 4, %[vdp]\n\t" \
      "v_bfi_b32 %[vdp], 63, %[vdp], %[dbase]\n\t"
#define UNSAFE   \
      "s_lshr_b32 %[un], s84, 16\n\t" \
      "s_or_b32 %[un], %[un], s84\n\t" \
      "s_lshr_b32 %[q], %[un], 8\n\t" \
      "s_or_b32 %[un], %[un], %[q]\n\t" \
      "s_or_b32 %[un], %[un], s85\n\t" \
      "s_bitset1_b32 %[un], 15\n\t"
#define NEXTREF   \
      "s_lshr_b32 %[m], %[d1], 8\n\t" \
      "v_readlane_b32 %[q], %[tab], %[m]\n\t" \
      "s_and_b32 %[nm], %[d1], 7\n\t" \
      "s_lshl_b64 s[86:87], s[84:85], 8\n\t" \
      "s_bitset1_b32 s86, %[nm]\n\t" \
      "s_lshl_b32 %[nm], %[nm], 14\n\t"
#define NEXTREF2 \
      "s_add_u32 %[nm], %[nm], %[d0]\n\t" \
      "s_add_u32 s82, %[wlo], %[q]\n\t" \
      "s_addc_u32 s83, %[whi], 0\n\t"
    asm volatile(
      "s_mov_b64 s[80:81], %[b0]\n\t"
      "s_mov_b32 s86, 0x04081020\n\t"
      "s_mov_b32 s87, 0x02\n\t"
      "s_add_u32 m0, %[d0], 0x14000\n\t"
      "s_branch 2f\n"
      "1:\n\t"
      "s_waitcnt vmcnt(40)\n"
      "6:\n\t"
      "s_cmp_lt_u32 %[m], 32\n\t"
      "s_barrier\n\t"
      "s_cbranch_scc0 3f\n"
      "2:\n\t"
      GL("%[o0]", "")
      "s_mov_b64 s[84:85], s[86:87]\n\t"
      RDDESC
      GL("%[o1]", " offset:1024")
      "s_sub_u32 %[left], %[left], 1\n\t"
      UNSAFE
      GL("%[o2]", " offset:2048")
      NEXTREF
      GL("%[o3]", " offset:3072")
      "s_add_u32 m0, m0, 0x1000\n\t"
      NEXTREF2
      GL("%[o4]", "")
      GL("%[o5]", " offset:1024")
      GL("%[o6]", " offset:2048")
      GL("%[o7]", " offset:3072")
      "4:\n\t"
      "s_waitcnt lgkmcnt(0)\n\t"
      "v_readfirstlane_b32 %[d1], %[vd]\n\t"
      "s_mov_b32 m0, %[nm]\n\t"
      "s_mov_b64 s[80:81], s[82:83]\n\t"
      "s_bitcmp1_b32 %[un], %[d1]\n\t"
      "s_cbranch_scc0 1b\n\t"
      "s_cmp_eq_u32 %[d1], 0x200f\n\t"
      "s_cbranch_scc1 7f\n\t"
      "s_and_b32 %[q], %[d1], 7\n\t"
      "s_lshl_b32 %[q], 0x01010101, %[q]\n\t"
      "s_waitcnt vmcnt(32)\n\t"
      "s_and_b32 %[q], %[q], s84\n\t"
      "s_cbranch_scc0 6b\n\t"
      "s_waitcnt vmcnt(24)\n\t"
      "s_and_b32 %[q], %[q], 0xffffff\n\t"
      "s_cbranch_scc0 6b\n\t"
      "s_waitcnt vmcnt(16)\n\t"
      "s_and_b32 %[q], %[q], 0xffff\n\t"
      "s_cbranch_scc0 6b\n\t"
      "s_waitcnt vmcnt(8)\n\t"
      "s_and_b32 %[q], %[q], 0xff\n\t"
      "s_cbranch_scc0 6b\n\t"
      "s_waitcnt vmcnt(0)\n\t"
      "s_cmp_eq_u32 %[left], 0\n\t"
      "s_cbranch_scc0 6b\n\t"
      "s_branch 10f\n"
      "3:\n\t"
      RDDESC NEXTREF NEXTREF2
      "s_branch 4b\n"
      "7:\n\t"
      "s_cmp_eq_u32 %[left], 0\n\t"
      "s_cbranch_scc1 10f\n\t"
      "s_and_b32 %[m], s86, 0xff\n\t"
      "s_mul_i32 %[q], %[m], 14\n\t"
      "s_lshr_b32 %[nm], %[q], 7\n\t"
      "s_or_b32 %[q], %[q], %[nm]\n\t"
      "s_and_b32 %[q], %[q], 0x7f\n\t"
      "s_mul_i32 %[q], %[q], 0x010101\n\t"
      "s_and_b32 %[q], %[q], s84\n\t"
      "s_waitcnt vmcnt(24)\n\t"
      "s_cbranch_scc0 8f\n\t"
      "s_waitcnt vmcnt(0)\n"
      "8:\n\t"
      "s_barrier\n\t"
      GL("%[o0]", "") GL("%[o1]", " offset:1024") GL("%[o2]", " offset:2048") GL("%[o3]", " offset:3072")
      "s_add_u32 m0, m0, 0x1000\n\t"
      "s_lshr_b32 %[q], %[m], 1\n\t"
      "s_cselect_b32 %[q], %[q], 0x40\n\t"
      "s_and_b32 %[q], %[q], s84\n\t"
      GL("%[o4]", "") GL("%[o5]", " offset:1024") GL("%[o6]", " offset:2048") GL("%[o7]", " offset:3072")
      "s_waitcnt vmcnt(16)\n\t"
      "s_cbranch_scc1 9f\n\t"
      "s_waitcnt vmcnt(8)\n"
      "9:\n\t"
      "s_barrier\n"
      "10:\n\t"
      "s_waitcnt vmcnt(0)\n\t"
      "s_barrier\n\t"
      "s_endpgm"
      : [vdp] "+&v"(vdp), [vd] "=&v"(vd), [left] "+&s"(left), [d1] "+&s"(d1), [q] "=&s"(q_), [m] "=&s"(m_), [nm] "=&s"(nm_), [un] "+&s"(un_)
      : [o0] "v"(off[0]), [o1] "v"(off[1]), [o2] "v"(off[2]), [o3] "v"(off[3]), [o4] "v"(off[4]), [o5] "v"(off[5]), [o6] "v"(off[6]), [o7] "v"(off[7]),
        [tab] "v"(tab), [wlo] "s"((u32)(u64)wb), [whi] "s"((u32)((u64)wb >> 32)), [b0] "s"(b0), [d0] "s"(d0), [dbase] "s"(lds0 + LDS_CTL + C_DESC)
      : "memory", "scc", "s80", "s81", "s82", "s83", "s84", "s85", "s86", "s87");
#undef GL
  } else if (wave == 6) {
    if (mt < 4) aloader<5>(p, lds0, lane, b, mt, rot, lr); else aloader<4>(p, lds0, lane, b, mt, rot, lr);
  } else {
    u32* cntp = (u32*)(myws + OFF_CNT + (size_t)b * 128) + 6;
    const u32 dst = lds0 + LDS_CTL + C_FLAGS;
    *(volatile LAS u32*)(dst + (lane & 31) * 4) = 0u;
    asm volatile("global_load_dword v205, %0, off sc0 sc1" :: "v"(cntp) : "memory", "v205");
    const u32 cb = (3u - (u32)((size_t)p.w >> 7) - (u32)cg * 4) & 7u;
    touch(cb < 4u ? (u32)(lane & 31) * GN + cb * 128 : 0u, p.w + (size_t)(((rank + 1 + (gi >> 2)) & 7) * KS + (gi & 3) * 32) * GN + cg * 512);
    const u32 voff = (lane & 7) < 7 ? (u32)OFF_FLAG + (u32)(mt * 8 + src_of(lane & 7, rank, rot)) * FSTR : (u32)OFF_TMO;
    u32 c;
    asm volatile("s_waitcnt vmcnt(1) lgkmcnt(0)\n\tv_readfirstlane_b32 %0, v205" : "=s"(c) :: "memory", "v205");
    const u32 call = c + 1;
    if (mt >= 4) {
      const u32 hi = (u32)((mt - 4) * 4 + ntl), hn = (u32)(mt - 4) * 3 + (u32)ntl - ((u32)ntl > cb ? 1u : 0u);
      const u32 nr = (u32)ntl == cb ? PFS : PFR;
      #pragma unroll 1
      for (u32 r = 0; r < nr; ++r) touch_w(p.w + cg * 512, rank, (cb >= 4u || r < PFS) ? 12 * r + hi : 12 * PFS + 9 * (r - PFS) + hn, lane);
    }
    u32 need = 0xFFu, seen = 0, credit = 0, gap = 0, t0_, vf = 0;
    asm volatile(
      "s_mov_b32 m0, %[dst]\n\t"
      "s_mov_b32 exec_hi, 0\n\t"
      "s_cmp_eq_u32 0, 0\n"
      "1:\n\t"
      "s_mov_b32 exec_lo, %[need]\n\t"
      "s_barrier\n\t"
      "s_cbranch_scc0 2f\n\t"
      "global_load_lds_dwordx4 %[voff], %[fb] sc0 sc1\n\t"
      "s_mov_b32 %[gap], 0\n"
      "2:\n\t"
      "s_mov_b32 exec_lo, -1\n\t"
      "s_waitcnt lgkmcnt(0)\n\t"
      "v_cmp_le_u32 vcc, %[call], %[vf]\n\t"
      "ds_read_b32 %[vf], %[sn]\n\t"
      "s_andn2_b32 %[t0], vcc_lo, %[seen]\n\t"
      "s_and_b32 %[t0], %[t0], 0x77777777\n\t"
      "s_bcnt1_i32_b32 %[t0], %[t0]\n\t"
      "s_or_b32 %[seen], %[seen], vcc_lo\n\t"
      "s_add_u32 %[credit], %[credit], %[t0]\n\t"
      "s_max_u32 %[credit], %[credit], 1\n\t"
      "s_sub_u32 %[credit], %[credit], 1\n\t"
      "s_not_b32 %[t0], %[seen]\n\t"
      "s_quadmask_b32 %[need], %[t0]\n\t"
      "s_bitcmp1_b32 %[seen], 28\n\t"
      "s_cselect_b32 %[need], 0x80, %[need]\n\t"
      "s_cmp_eq_u32 %[need], 0x80\n\t"
      "s_cbranch_scc1 3f\n\t"
      "s_add_u32 %[gap], %[gap], 1\n\t"
      "s_cmp_ge_u32 %[credit], %[c1]\n\t"
      "s_cselect_b32 %[t0], 2, 1\n\t"
      "s_cmp_ge_u32 %[credit], %[c2]\n\t"
      "s_cselect_b32 %[t0], 99, %[t0]\n\t"
      "s_waitcnt vmcnt(24)\n\t"
      "s_cmp_ge_u32 %[gap], %[t0]\n\t"
      "s_branch 1b\n"
      "3:\n\t"
      "s_mov_b64 exec, -1\n\t"
      "s_barrier\n\ts_barrier\n\ts_barrier\n\ts_barrier\n\t"
      "s_waitcnt vmcnt(0) lgkmcnt(0)\n\t"
      "s_endpgm"
      : [need] "+&s"(need), [seen] "+&s"(seen), [credit] "+&s"(credit), [gap] "+&s"(gap), [t0] "=&s"(t0_), [vf] "+&v"(vf)
      : [voff] "v"(voff), [fb] "s"(myws), [dst] "s"(dst), [call] "s"(call), [sn] "v"(dst + (lane & 31) * 4), [c1] "i"(PC1), [c2] "i"(PC2)
      : "memory", "scc", "vcc", "v204");
  }
}

extern "C" void kernel_launch(int rank, int nranks, ncclComm_t comm, hipStream_t stream,
                              void* const* d_in, const int* in_sizes, int n_in,
                              void* d_out, int out_size, const int* in_dtypes, int out_dtype,
                              void* const* d_out_all, void* const* d_ws_all, size_t ws_bytes) {
  if (nranks != NRANK || (unsigned)rank >= (unsigned)NRANK || ws_bytes < WS_NEED || n_in < 4) {
    fprintf(stderr, "unexpected configuration\n");
    return;
  }
  Args a;
  a.x = (const unsigned char*)d_in[0]; a.w = (const unsigned char*)d_in[1];
  a.sx = (const float*)d_in[2]; a.sw = (const float*)d_in[3]; a.out = (float*)d_out;
  for (int q = 0; q < NRANK; ++q) a.peer[q] = (char*)d_ws_all[(rank + 1 + q) % NRANK];
  a.myws = (char*)d_ws_all[rank]; a.rank = rank; a.pad = 0;
  HIP_CHECK(hipFuncSetAttribute((const void*)a2a_gemm_kernel, hipFuncAttributeMaxDynamicSharedMemorySize, LDS_BYTES));
  a2a_gemm_kernel<<<256, 512, LDS_BYTES, stream>>>(a);
  HIP_CHECK(hipGetLastError());
}
```
